```python
import numpy as np

kernel_name = "gpu_ag_gemm_m4096_k4096_n16384_bf16_relu"


ABSMAX_RTOL = 0.008

COMPUTE_DTYPE = "bfloat16"
EPILOGUE = "relu"
IN_QUANT = None
OUT_QUANT = None
IN_DTYPES = {"x": "bfloat16", "w_mat": "bfloat16"}
WORLD_SIZE = 8

def _to_bf16(x):
    u = np.ascontiguousarray(x, dtype=np.float32).view(np.uint32)
    return ((u + 0x7FFF + ((u >> 16) & 1)) & 0xFFFF0000).view(np.float32)

def setup_inputs():

    x = _to_bf16(np.random.randn(4096, 4096).astype(np.float32))
    w_mat = _to_bf16((np.random.randn(4096, 16384) / np.sqrt(4096)).astype(np.float32))
    return {"x": x, "w_mat": w_mat}

def _epilogue(y):
    return np.maximum(y, 0.0).astype(np.float32)

def reference(x, w_mat):
    y = (x @ w_mat).astype(np.float32)
    return _epilogue(y)
```

```cpp
#include <hip/hip_runtime.h>
#include <rccl/rccl.h>
#include <cstdio>

#define OUT_POL " sc1"
#define CONS_ASM_9 \
  "s_setprio 3\n\t" \
  "v_mov_b32 v112, 0\n\t" \
  "v_mov_b32 v113, 0\n\t" \
  "v_mov_b32 v114, 0\n\t" \
  "v_mov_b32 v115, 0\n\t" \
  "v_mov_b32 v116, 0\n\t" \
  "v_mov_b32 v117, 0\n\t" \
  "v_mov_b32 v118, 0\n\t" \
  "v_mov_b32 v119, 0\n\t" \
  "v_mov_b32 v120, 0\n\t" \
  "v_mov_b32 v121, 0\n\t" \
  "v_mov_b32 v122, 0\n\t" \
  "v_mov_b32 v123, 0\n\t" \
  "v_mov_b32 v124, 0\n\t" \
  "v_mov_b32 v125, 0\n\t" \
  "v_mov_b32 v126, 0\n\t" \
  "v_mov_b32 v127, 0\n\t" \
  "v_mov_b32 v128, 0\n\t" \
  "v_mov_b32 v129, 0\n\t" \
  "v_mov_b32 v130, 0\n\t" \
  "v_mov_b32 v131, 0\n\t" \
  "v_mov_b32 v132, 0\n\t" \
  "v_mov_b32 v133, 0\n\t" \
  "v_mov_b32 v134, 0\n\t" \
  "v_mov_b32 v135, 0\n\t" \
  "v_mov_b32 v136, 0\n\t" \
  "v_mov_b32 v137, 0\n\t" \
  "v_mov_b32 v138, 0\n\t" \
  "v_mov_b32 v139, 0\n\t" \
  "v_mov_b32 v140, 0\n\t" \
  "v_mov_b32 v141, 0\n\t" \
  "v_mov_b32 v142, 0\n\t" \
  "v_mov_b32 v143, 0\n\t" \
  "v_mov_b32 v144, 0\n\t" \
  "v_mov_b32 v145, 0\n\t" \
  "v_mov_b32 v146, 0\n\t" \
  "v_mov_b32 v147, 0\n\t" \
  "v_mov_b32 v148, 0\n\t" \
  "v_mov_b32 v149, 0\n\t" \
  "v_mov_b32 v150, 0\n\t" \
  "v_mov_b32 v151, 0\n\t" \
  "v_mov_b32 v152, 0\n\t" \
  "v_mov_b32 v153, 0\n\t" \
  "v_mov_b32 v154, 0\n\t" \
  "v_mov_b32 v155, 0\n\t" \
  "v_mov_b32 v156, 0\n\t" \
  "v_mov_b32 v157, 0\n\t" \
  "v_mov_b32 v158, 0\n\t" \
  "v_mov_b32 v159, 0\n\t" \
  "v_mov_b32 v160, 0\n\t" \
  "v_mov_b32 v161, 0\n\t" \
  "v_mov_b32 v162, 0\n\t" \
  "v_mov_b32 v163, 0\n\t" \
  "v_mov_b32 v164, 0\n\t" \
  "v_mov_b32 v165, 0\n\t" \
  "v_mov_b32 v166, 0\n\t" \
  "v_mov_b32 v167, 0\n\t" \
  "v_mov_b32 v168, 0\n\t" \
  "v_mov_b32 v169, 0\n\t" \
  "v_mov_b32 v170, 0\n\t" \
  "v_mov_b32 v171, 0\n\t" \
  "v_mov_b32 v172, 0\n\t" \
  "v_mov_b32 v173, 0\n\t" \
  "v_mov_b32 v174, 0\n\t" \
  "v_mov_b32 v175, 0\n\t" \
  "v_mov_b32 v176, 0\n\t" \
  "v_mov_b32 v177, 0\n\t" \
  "v_mov_b32 v178, 0\n\t" \
  "v_mov_b32 v179, 0\n\t" \
  "v_mov_b32 v180, 0\n\t" \
  "v_mov_b32 v181, 0\n\t" \
  "v_mov_b32 v182, 0\n\t" \
  "v_mov_b32 v183, 0\n\t" \
  "v_mov_b32 v184, 0\n\t" \
  "v_mov_b32 v185, 0\n\t" \
  "v_mov_b32 v186, 0\n\t" \
  "v_mov_b32 v187, 0\n\t" \
  "v_mov_b32 v188, 0\n\t" \
  "v_mov_b32 v189, 0\n\t" \
  "v_mov_b32 v190, 0\n\t" \
  "v_mov_b32 v191, 0\n\t" \
  "v_mov_b32 v192, 0\n\t" \
  "v_mov_b32 v193, 0\n\t" \
  "v_mov_b32 v194, 0\n\t" \
  "v_mov_b32 v195, 0\n\t" \
  "v_mov_b32 v196, 0\n\t" \
  "v_mov_b32 v197, 0\n\t" \
  "v_mov_b32 v198, 0\n\t" \
  "v_mov_b32 v199, 0\n\t" \
  "v_mov_b32 v200, 0\n\t" \
  "v_mov_b32 v201, 0\n\t" \
  "v_mov_b32 v202, 0\n\t" \
  "v_mov_b32 v203, 0\n\t" \
  "v_mov_b32 v204, 0\n\t" \
  "v_mov_b32 v205, 0\n\t" \
  "v_mov_b32 v206, 0\n\t" \
  "v_mov_b32 v207, 0\n\t" \
  "v_mov_b32 v208, 0\n\t" \
  "v_mov_b32 v209, 0\n\t" \
  "v_mov_b32 v210, 0\n\t" \
  "v_mov_b32 v211, 0\n\t" \
  "v_mov_b32 v212, 0\n\t" \
  "v_mov_b32 v213, 0\n\t" \
  "v_mov_b32 v214, 0\n\t" \
  "v_mov_b32 v215, 0\n\t" \
  "v_mov_b32 v216, 0\n\t" \
  "v_mov_b32 v217, 0\n\t" \
  "v_mov_b32 v218, 0\n\t" \
  "v_mov_b32 v219, 0\n\t" \
  "v_mov_b32 v220, 0\n\t" \
  "v_mov_b32 v221, 0\n\t" \
  "v_mov_b32 v222, 0\n\t" \
  "v_mov_b32 v223, 0\n\t" \
  "v_mov_b32 v224, 0\n\t" \
  "v_mov_b32 v225, 0\n\t" \
  "v_mov_b32 v226, 0\n\t" \
  "v_mov_b32 v227, 0\n\t" \
  "v_mov_b32 v228, 0\n\t" \
  "v_mov_b32 v229, 0\n\t" \
  "v_mov_b32 v230, 0\n\t" \
  "v_mov_b32 v231, 0\n\t" \
  "v_mov_b32 v232, 0\n\t" \
  "v_mov_b32 v233, 0\n\t" \
  "v_mov_b32 v234, 0\n\t" \
  "v_mov_b32 v235, 0\n\t" \
  "v_mov_b32 v236, 0\n\t" \
  "v_mov_b32 v237, 0\n\t" \
  "v_mov_b32 v238, 0\n\t" \
  "v_mov_b32 v239, 0\n\t" \
  "v_mov_b32 v240, 0\n\t" \
  "v_mov_b32 v241, 0\n\t" \
  "v_mov_b32 v242, 0\n\t" \
  "v_mov_b32 v243, 0\n\t" \
  "v_mov_b32 v244, 0\n\t" \
  "v_mov_b32 v245, 0\n\t" \
  "v_mov_b32 v246, 0\n\t" \
  "v_mov_b32 v247, 0\n\t" \
  "v_mov_b32 v248, 0\n\t" \
  "v_mov_b32 v249, 0\n\t" \
  "v_mov_b32 v250, 0\n\t" \
  "v_mov_b32 v251, 0\n\t" \
  "v_mov_b32 v252, 0\n\t" \
  "v_mov_b32 v253, 0\n\t" \
  "v_mov_b32 v254, 0\n\t" \
  "v_mov_b32 v255, 0\n\t" \
  "v_xor_b32 v8, 32, v7\n\t" \
  "v_xor_b32 v9, 64, v7\n\t" \
  "v_xor_b32 v10, 96, v7\n\t" \
  "s_mov_b32 %[st], 0\n\t" \
  "s_mov_b32 %[dp], 53248\n\t" \
  "s_mov_b32 %[dn], 4294860800\n\t" \
  "s_mov_b32 %[cnt], 30\n\t" \
  "v_mov_b32 v5, v11\n\t" \
  "s_barrier\n\t" \
  "ds_read_b64_tr_b16 v[48:49], v7 offset:0\n\t" \
  "ds_read_b64_tr_b16 v[50:51], v7 offset:2048\n\t" \
  "ds_read_b64_tr_b16 v[64:65], v7 offset:16384\n\t" \
  "ds_read_b64_tr_b16 v[66:67], v7 offset:18432\n\t" \
  "ds_read_b64_tr_b16 v[52:53], v8 offset:0\n\t" \
  "ds_read_b64_tr_b16 v[54:55], v8 offset:2048\n\t" \
  "ds_read_b64_tr_b16 v[68:69], v8 offset:16384\n\t" \
  "ds_read_b64_tr_b16 v[70:71], v8 offset:18432\n\t" \
  "ds_read_b64_tr_b16 v[56:57], v9 offset:0\n\t" \
  "ds_read_b64_tr_b16 v[58:59], v9 offset:2048\n\t" \
  "ds_read_b64_tr_b16 v[72:73], v9 offset:16384\n\t" \
  "ds_read_b64_tr_b16 v[74:75], v9 offset:18432\n\t" \
  "ds_read_b64_tr_b16 v[60:61], v10 offset:0\n\t" \
  "ds_read_b64_tr_b16 v[62:63], v10 offset:2048\n\t" \
  "ds_read_b64_tr_b16 v[76:77], v10 offset:16384\n\t" \
  "ds_read_b64_tr_b16 v[78:79], v10 offset:18432\n\t" \
  "v_add_u32 v7, %[dn], v7\n\t" \
  "v_add_u32 v8, %[dn], v8\n\t" \
  "v_add_u32 v9, %[dn], v9\n\t" \
  "v_add_u32 v10, %[dn], v10\n\t" \
  "s_waitcnt lgkmcnt(0)\n\t" \
  "s_barrier\n\t" \
  "ds_read_b128 v[12:15], v11 offset:0\n\t" \
  "ds_read_b128 v[16:19], v11 offset:10240\n\t" \
  "ds_read_b128 v[20:23], v11 offset:1024\n\t" \
  "ds_read_b128 v[24:27], v11 offset:11264\n\t" \
  ".p2align 6\n\t" \
  "L_cons_loop9_%=:\n\t" \
  "s_waitcnt lgkmcnt(3)\n\t" \
  "v_mfma_f32_16x16x32_bf16 v[112:115], v[48:51], v[12:15], v[112:115]\n\t" \
  "s_waitcnt lgkmcnt(2)\n\t" \
  "v_mfma_f32_16x16x32_bf16 v[112:115], v[64:67], v[16:19], v[112:115]\n\t" \
  "ds_read_b128 v[28:31], v11 offset:2048\n\t" \
  "v_mfma_f32_16x16x32_bf16 v[116:119], v[52:55], v[12:15], v[116:119]\n\t" \
  "ds_read_b64_tr_b16 v[80:81], v7 offset:0\n\t" \
  "v_mfma_f32_16x16x32_bf16 v[116:119], v[68:71], v[16:19], v[116:119]\n\t" \
  "s_add_u32 %[st], %[st], 1\n\t" \
  "v_mfma_f32_16x16x32_bf16 v[120:123], v[56:59], v[12:15], v[120:123]\n\t" \
  "ds_read_b64_tr_b16 v[82:83], v7 offset:2048\n\t" \
  "v_mfma_f32_16x16x32_bf16 v[120:123], v[72:75], v[16:19], v[120:123]\n\t" \
  "s_cmp_eq_u32 %[st], 3\n\t" \
  "v_mfma_f32_16x16x32_bf16 v[124:127], v[60:63], v[12:15], v[124:127]\n\t" \
  "ds_read_b128 v[32:35], v11 offset:12288\n\t" \
  "v_mfma_f32_16x16x32_bf16 v[124:127], v[76:79], v[16:19], v[124:127]\n\t" \
  "s_waitcnt lgkmcnt(5)\n\t" \
  "v_mfma_f32_16x16x32_bf16 v[128:131], v[48:51], v[20:23], v[128:131]\n\t" \
  "s_waitcnt lgkmcnt(4)\n\t" \
  "v_mfma_f32_16x16x32_bf16 v[128:131], v[64:67], v[24:27], v[128:131]\n\t" \
  "ds_read_b128 v[12:15], v11 offset:3072\n\t" \
  "v_mfma_f32_16x16x32_bf16 v[132:135], v[52:55], v[20:23], v[132:135]\n\t" \
  "ds_read_b64_tr_b16 v[96:97], v7 offset:16384\n\t" \
  "v_mfma_f32_16x16x32_bf16 v[132:135], v[68:71], v[24:27], v[132:135]\n\t" \
  "s_cselect_b32 %[dl], %[dn], %[dp]\n\t" \
  "v_mfma_f32_16x16x32_bf16 v[136:139], v[56:59], v[20:23], v[136:139]\n\t" \
  "ds_read_b64_tr_b16 v[98:99], v7 offset:18432\n\t" \
  "v_mfma_f32_16x16x32_bf16 v[136:139], v[72:75], v[24:27], v[136:139]\n\t" \
  "s_cselect_b32 %[st], 0, %[st]\n\t" \
  "v_mfma_f32_16x16x32_bf16 v[140:143], v[60:63], v[20:23], v[140:143]\n\t" \
  "ds_read_b128 v[16:19], v11 offset:13312\n\t" \
  "v_mfma_f32_16x16x32_bf16 v[140:143], v[76:79], v[24:27], v[140:143]\n\t" \
  "s_waitcnt lgkmcnt(7)\n\t" \
  "v_mfma_f32_16x16x32_bf16 v[144:147], v[48:51], v[28:31], v[144:147]\n\t" \
  "s_waitcnt lgkmcnt(4)\n\t" \
  "v_mfma_f32_16x16x32_bf16 v[144:147], v[64:67], v[32:35], v[144:147]\n\t" \
  "ds_read_b128 v[20:23], v11 offset:4096\n\t" \
  "v_mfma_f32_16x16x32_bf16 v[148:151], v[52:55], v[28:31], v[148:151]\n\t" \
  "ds_read_b64_tr_b16 v[84:85], v8 offset:0\n\t" \
  "v_mfma_f32_16x16x32_bf16 v[148:151], v[68:71], v[32:35], v[148:151]\n\t" \
  "v_mfma_f32_16x16x32_bf16 v[152:155], v[56:59], v[28:31], v[152:155]\n\t" \
  "ds_read_b64_tr_b16 v[86:87], v8 offset:2048\n\t" \
  "v_mfma_f32_16x16x32_bf16 v[152:155], v[72:75], v[32:35], v[152:155]\n\t" \
  "ds_read_b64_tr_b16 v[100:101], v8 offset:16384\n\t" \
  "v_mfma_f32_16x16x32_bf16 v[156:159], v[60:63], v[28:31], v[156:159]\n\t" \
  "ds_read_b128 v[24:27], v11 offset:14336\n\t" \
  "v_mfma_f32_16x16x32_bf16 v[156:159], v[76:79], v[32:35], v[156:159]\n\t" \
  "s_waitcnt lgkmcnt(8)\n\t" \
  "v_mfma_f32_16x16x32_bf16 v[160:163], v[48:51], v[12:15], v[160:163]\n\t" \
  "s_waitcnt lgkmcnt(5)\n\t" \
  "v_mfma_f32_16x16x32_bf16 v[160:163], v[64:67], v[16:19], v[160:163]\n\t" \
  "ds_read_b128 v[28:31], v11 offset:5120\n\t" \
  "v_mfma_f32_16x16x32_bf16 v[164:167], v[52:55], v[12:15], v[164:167]\n\t" \
  "ds_read_b64_tr_b16 v[102:103], v8 offset:18432\n\t" \
  "v_mfma_f32_16x16x32_bf16 v[164:167], v[68:71], v[16:19], v[164:167]\n\t" \
  "v_mfma_f32_16x16x32_bf16 v[168:171], v[56:59], v[12:15], v[168:171]\n\t" \
  "ds_read_b64_tr_b16 v[88:89], v9 offset:0\n\t" \
  "v_mfma_f32_16x16x32_bf16 v[168:171], v[72:75], v[16:19], v[168:171]\n\t" \
  "ds_read_b64_tr_b16 v[90:91], v9 offset:2048\n\t" \
  "v_mfma_f32_16x16x32_bf16 v[172:175], v[60:63], v[12:15], v[172:175]\n\t" \
  "ds_read_b128 v[32:35], v11 offset:15360\n\t" \
  "v_mfma_f32_16x16x32_bf16 v[172:175], v[76:79], v[16:19], v[172:175]\n\t" \
  "s_waitcnt lgkmcnt(9)\n\t" \
  "v_mfma_f32_16x16x32_bf16 v[176:179], v[48:51], v[20:23], v[176:179]\n\t" \
  "s_waitcnt lgkmcnt(5)\n\t" \
  "v_mfma_f32_16x16x32_bf16 v[176:179], v[64:67], v[24:27], v[176:179]\n\t" \
  "ds_read_b128 v[12:15], v11 offset:6144\n\t" \
  "v_mfma_f32_16x16x32_bf16 v[180:183], v[52:55], v[20:23], v[180:183]\n\t" \
  "ds_read_b64_tr_b16 v[104:105], v9 offset:16384\n\t" \
  "v_mfma_f32_16x16x32_bf16 v[180:183], v[68:71], v[24:27], v[180:183]\n\t" \
  "v_mfma_f32_16x16x32_bf16 v[184:187], v[56:59], v[20:23], v[184:187]\n\t" \
  "ds_read_b64_tr_b16 v[106:107], v9 offset:18432\n\t" \
  "v_mfma_f32_16x16x32_bf16 v[184:187], v[72:75], v[24:27], v[184:187]\n\t" \
  "v_mfma_f32_16x16x32_bf16 v[188:191], v[60:63], v[20:23], v[188:191]\n\t" \
  "ds_read_b128 v[16:19], v11 offset:16384\n\t" \
  "v_mfma_f32_16x16x32_bf16 v[188:191], v[76:79], v[24:27], v[188:191]\n\t" \
  "s_waitcnt lgkmcnt(8)\n\t" \
  "v_mfma_f32_16x16x32_bf16 v[192:195], v[48:51], v[28:31], v[192:195]\n\t" \
  "s_waitcnt lgkmcnt(4)\n\t" \
  "v_mfma_f32_16x16x32_bf16 v[192:195], v[64:67], v[32:35], v[192:195]\n\t" \
  "ds_read_b128 v[20:23], v11 offset:7168\n\t" \
  "v_mfma_f32_16x16x32_bf16 v[196:199], v[52:55], v[28:31], v[196:199]\n\t" \
  "ds_read_b64_tr_b16 v[92:93], v10 offset:0\n\t" \
  "v_mfma_f32_16x16x32_bf16 v[196:199], v[68:71], v[32:35], v[196:199]\n\t" \
  "v_mfma_f32_16x16x32_bf16 v[200:203], v[56:59], v[28:31], v[200:203]\n\t" \
  "ds_read_b64_tr_b16 v[94:95], v10 offset:2048\n\t" \
  "v_mfma_f32_16x16x32_bf16 v[200:203], v[72:75], v[32:35], v[200:203]\n\t" \
  "v_mfma_f32_16x16x32_bf16 v[204:207], v[60:63], v[28:31], v[204:207]\n\t" \
  "ds_read_b128 v[24:27], v11 offset:17408\n\t" \
  "v_mfma_f32_16x16x32_bf16 v[204:207], v[76:79], v[32:35], v[204:207]\n\t" \
  "s_waitcnt lgkmcnt(7)\n\t" \
  "v_mfma_f32_16x16x32_bf16 v[208:211], v[48:51], v[12:15], v[208:211]\n\t" \
  "s_waitcnt lgkmcnt(4)\n\t" \
  "v_mfma_f32_16x16x32_bf16 v[208:211], v[64:67], v[16:19], v[208:211]\n\t" \
  "ds_read_b128 v[28:31], v11 offset:8192\n\t" \
  "v_mfma_f32_16x16x32_bf16 v[212:215], v[52:55], v[12:15], v[212:215]\n\t" \
  "ds_read_b64_tr_b16 v[108:109], v10 offset:16384\n\t" \
  "v_mfma_f32_16x16x32_bf16 v[212:215], v[68:71], v[16:19], v[212:215]\n\t" \
  "v_mfma_f32_16x16x32_bf16 v[216:219], v[56:59], v[12:15], v[216:219]\n\t" \
  "ds_read_b64_tr_b16 v[110:111], v10 offset:18432\n\t" \
  "v_mfma_f32_16x16x32_bf16 v[216:219], v[72:75], v[16:19], v[216:219]\n\t" \
  "v_mfma_f32_16x16x32_bf16 v[220:223], v[60:63], v[12:15], v[220:223]\n\t" \
  "ds_read_b128 v[32:35], v11 offset:18432\n\t" \
  "v_mfma_f32_16x16x32_bf16 v[220:223], v[76:79], v[16:19], v[220:223]\n\t" \
  "s_waitcnt lgkmcnt(7)\n\t" \
  "v_mfma_f32_16x16x32_bf16 v[224:227], v[48:51], v[20:23], v[224:227]\n\t" \
  "s_waitcnt lgkmcnt(4)\n\t" \
  "v_mfma_f32_16x16x32_bf16 v[224:227], v[64:67], v[24:27], v[224:227]\n\t" \
  "v_add_u32 v7, %[dl], v7\n\t" \
  "v_mfma_f32_16x16x32_bf16 v[228:231], v[52:55], v[20:23], v[228:231]\n\t" \
  "v_add_u32 v8, %[dl], v8\n\t" \
  "v_mfma_f32_16x16x32_bf16 v[228:231], v[68:71], v[24:27], v[228:231]\n\t" \
  "v_add_u32 v9, %[dl], v9\n\t" \
  "v_mfma_f32_16x16x32_bf16 v[232:235], v[56:59], v[20:23], v[232:235]\n\t" \
  "v_add_u32 v10, %[dl], v10\n\t" \
  "v_mfma_f32_16x16x32_bf16 v[232:235], v[72:75], v[24:27], v[232:235]\n\t" \
  "v_add_u32 v11, %[dl], v11\n\t" \
  "v_mfma_f32_16x16x32_bf16 v[236:239], v[60:63], v[20:23], v[236:239]\n\t" \
  "v_mfma_f32_16x16x32_bf16 v[236:239], v[76:79], v[24:27], v[236:239]\n\t" \
  "s_waitcnt lgkmcnt(0)\n\t" \
  "s_barrier\n\t" \
  "ds_read_b128 v[12:15], v11 offset:0\n\t" \
  "v_mfma_f32_16x16x32_bf16 v[240:243], v[48:51], v[28:31], v[240:243]\n\t" \
  "ds_read_b128 v[16:19], v11 offset:10240\n\t" \
  "v_mfma_f32_16x16x32_bf16 v[240:243], v[64:67], v[32:35], v[240:243]\n\t" \
  "ds_read_b128 v[20:23], v11 offset:1024\n\t" \
  "v_mfma_f32_16x16x32_bf16 v[244:247], v[52:55], v[28:31], v[244:247]\n\t" \
  "v_mfma_f32_16x16x32_bf16 v[244:247], v[68:71], v[32:35], v[244:247]\n\t" \
  "v_mfma_f32_16x16x32_bf16 v[248:251], v[56:59], v[28:31], v[248:251]\n\t" \
  "v_mfma_f32_16x16x32_bf16 v[248:251], v[72:75], v[32:35], v[248:251]\n\t" \
  "v_mfma_f32_16x16x32_bf16 v[252:255], v[60:63], v[28:31], v[252:255]\n\t" \
  "ds_read_b128 v[24:27], v11 offset:11264\n\t" \
  "v_mfma_f32_16x16x32_bf16 v[252:255], v[76:79], v[32:35], v[252:255]\n\t" \
  "s_waitcnt lgkmcnt(3)\n\t" \
  "v_mfma_f32_16x16x32_bf16 v[112:115], v[80:83], v[12:15], v[112:115]\n\t" \
  "s_waitcnt lgkmcnt(2)\n\t" \
  "v_mfma_f32_16x16x32_bf16 v[112:115], v[96:99], v[16:19], v[112:115]\n\t" \
  "ds_read_b128 v[28:31], v11 offset:2048\n\t" \
  "v_mfma_f32_16x16x32_bf16 v[116:119], v[84:87], v[12:15], v[116:119]\n\t" \
  "ds_read_b64_tr_b16 v[48:49], v7 offset:0\n\t" \
  "v_mfma_f32_16x16x32_bf16 v[116:119], v[100:103], v[16:19], v[116:119]\n\t" \
  "s_add_u32 %[st], %[st], 1\n\t" \
  "v_mfma_f32_16x16x32_bf16 v[120:123], v[88:91], v[12:15], v[120:123]\n\t" \
  "ds_read_b64_tr_b16 v[50:51], v7 offset:2048\n\t" \
  "v_mfma_f32_16x16x32_bf16 v[120:123], v[104:107], v[16:19], v[120:123]\n\t" \
  "s_cmp_eq_u32 %[st], 3\n\t" \
  "v_mfma_f32_16x16x32_bf16 v[124:127], v[92:95], v[12:15], v[124:127]\n\t" \
  "ds_read_b128 v[32:35], v11 offset:12288\n\t" \
  "v_mfma_f32_16x16x32_bf16 v[124:127], v[108:111], v[16:19], v[124:127]\n\t" \
  "s_waitcnt lgkmcnt(5)\n\t" \
  "v_mfma_f32_16x16x32_bf16 v[128:131], v[80:83], v[20:23], v[128:131]\n\t" \
  "s_waitcnt lgkmcnt(4)\n\t" \
  "v_mfma_f32_16x16x32_bf16 v[128:131], v[96:99], v[24:27], v[128:131]\n\t" \
  "ds_read_b128 v[12:15], v11 offset:3072\n\t" \
  "v_mfma_f32_16x16x32_bf16 v[132:135], v[84:87], v[20:23], v[132:135]\n\t" \
  "ds_read_b64_tr_b16 v[64:65], v7 offset:16384\n\t" \
  "v_mfma_f32_16x16x32_bf16 v[132:135], v[100:103], v[24:27], v[132:135]\n\t" \
  "s_cselect_b32 %[dl], %[dn], %[dp]\n\t" \
  "v_mfma_f32_16x16x32_bf16 v[136:139], v[88:91], v[20:23], v[136:139]\n\t" \
  "ds_read_b64_tr_b16 v[66:67], v7 offset:18432\n\t" \
  "v_mfma_f32_16x16x32_bf16 v[136:139], v[104:107], v[24:27], v[136:139]\n\t" \
  "s_cselect_b32 %[st], 0, %[st]\n\t" \
  "v_mfma_f32_16x16x32_bf16 v[140:143], v[92:95], v[20:23], v[140:143]\n\t" \
  "ds_read_b128 v[16:19], v11 offset:13312\n\t" \
  "v_mfma_f32_16x16x32_bf16 v[140:143], v[108:111], v[24:27], v[140:143]\n\t" \
  "s_waitcnt lgkmcnt(7)\n\t" \
  "v_mfma_f32_16x16x32_bf16 v[144:147], v[80:83], v[28:31], v[144:147]\n\t" \
  "s_waitcnt lgkmcnt(4)\n\t" \
  "v_mfma_f32_16x16x32_bf16 v[144:147], v[96:99], v[32:35], v[144:147]\n\t" \
  "ds_read_b128 v[20:23], v11 offset:4096\n\t" \
  "v_mfma_f32_16x16x32_bf16 v[148:151], v[84:87], v[28:31], v[148:151]\n\t" \
  "ds_read_b64_tr_b16 v[52:53], v8 offset:0\n\t" \
  "v_mfma_f32_16x16x32_bf16 v[148:151], v[100:103], v[32:35], v[148:151]\n\t" \
  "v_mfma_f32_16x16x32_bf16 v[152:155], v[88:91], v[28:31], v[152:155]\n\t" \
  "ds_read_b64_tr_b16 v[54:55], v8 offset:2048\n\t" \
  "v_mfma_f32_16x16x32_bf16 v[152:155], v[104:107], v[32:35], v[152:155]\n\t" \
  "ds_read_b64_tr_b16 v[68:69], v8 offset:16384\n\t" \
  "v_mfma_f32_16x16x32_bf16 v[156:159], v[92:95], v[28:31], v[156:159]\n\t" \
  "ds_read_b128 v[24:27], v11 offset:14336\n\t" \
  "v_mfma_f32_16x16x32_bf16 v[156:159], v[108:111], v[32:35], v[156:159]\n\t" \
  "s_waitcnt lgkmcnt(8)\n\t" \
  "v_mfma_f32_16x16x32_bf16 v[160:163], v[80:83], v[12:15], v[160:163]\n\t" \
  "s_waitcnt lgkmcnt(5)\n\t" \
  "v_mfma_f32_16x16x32_bf16 v[160:163], v[96:99], v[16:19], v[160:163]\n\t" \
  "ds_read_b128 v[28:31], v11 offset:5120\n\t" \
  "v_mfma_f32_16x16x32_bf16 v[164:167], v[84:87], v[12:15], v[164:167]\n\t" \
  "ds_read_b64_tr_b16 v[70:71], v8 offset:18432\n\t" \
  "v_mfma_f32_16x16x32_bf16 v[164:167], v[100:103], v[16:19], v[164:167]\n\t" \
  "v_mfma_f32_16x16x32_bf16 v[168:171], v[88:91], v[12:15], v[168:171]\n\t" \
  "ds_read_b64_tr_b16 v[56:57], v9 offset:0\n\t" \
  "v_mfma_f32_16x16x32_bf16 v[168:171], v[104:107], v[16:19], v[168:171]\n\t" \
  "ds_read_b64_tr_b16 v[58:59], v9 offset:2048\n\t" \
  "v_mfma_f32_16x16x32_bf16 v[172:175], v[92:95], v[12:15], v[172:175]\n\t" \
  "ds_read_b128 v[32:35], v11 offset:15360\n\t" \
  "v_mfma_f32_16x16x32_bf16 v[172:175], v[108:111], v[16:19], v[172:175]\n\t" \
  "s_waitcnt lgkmcnt(9)\n\t" \
  "v_mfma_f32_16x16x32_bf16 v[176:179], v[80:83], v[20:23], v[176:179]\n\t" \
  "s_waitcnt lgkmcnt(5)\n\t" \
  "v_mfma_f32_16x16x32_bf16 v[176:179], v[96:99], v[24:27], v[176:179]\n\t" \
  "ds_read_b128 v[12:15], v11 offset:6144\n\t" \
  "v_mfma_f32_16x16x32_bf16 v[180:183], v[84:87], v[20:23], v[180:183]\n\t" \
  "ds_read_b64_tr_b16 v[72:73], v9 offset:16384\n\t" \
  "v_mfma_f32_16x16x32_bf16 v[180:183], v[100:103], v[24:27], v[180:183]\n\t" \
  "v_mfma_f32_16x16x32_bf16 v[184:187], v[88:91], v[20:23], v[184:187]\n\t" \
  "ds_read_b64_tr_b16 v[74:75], v9 offset:18432\n\t" \
  "v_mfma_f32_16x16x32_bf16 v[184:187], v[104:107], v[24:27], v[184:187]\n\t" \
  "v_mfma_f32_16x16x32_bf16 v[188:191], v[92:95], v[20:23], v[188:191]\n\t" \
  "ds_read_b128 v[16:19], v11 offset:16384\n\t" \
  "v_mfma_f32_16x16x32_bf16 v[188:191], v[108:111], v[24:27], v[188:191]\n\t" \
  "s_waitcnt lgkmcnt(8)\n\t" \
  "v_mfma_f32_16x16x32_bf16 v[192:195], v[80:83], v[28:31], v[192:195]\n\t" \
  "s_waitcnt lgkmcnt(4)\n\t" \
  "v_mfma_f32_16x16x32_bf16 v[192:195], v[96:99], v[32:35], v[192:195]\n\t" \
  "ds_read_b128 v[20:23], v11 offset:7168\n\t" \
  "v_mfma_f32_16x16x32_bf16 v[196:199], v[84:87], v[28:31], v[196:199]\n\t" \
  "ds_read_b64_tr_b16 v[60:61], v10 offset:0\n\t" \
  "v_mfma_f32_16x16x32_bf16 v[196:199], v[100:103], v[32:35], v[196:199]\n\t" \
  "v_mfma_f32_16x16x32_bf16 v[200:203], v[88:91], v[28:31], v[200:203]\n\t" \
  "ds_read_b64_tr_b16 v[62:63], v10 offset:2048\n\t" \
  "v_mfma_f32_16x16x32_bf16 v[200:203], v[104:107], v[32:35], v[200:203]\n\t" \
  "v_mfma_f32_16x16x32_bf16 v[204:207], v[92:95], v[28:31], v[204:207]\n\t" \
  "ds_read_b128 v[24:27], v11 offset:17408\n\t" \
  "v_mfma_f32_16x16x32_bf16 v[204:207], v[108:111], v[32:35], v[204:207]\n\t" \
  "s_waitcnt lgkmcnt(7)\n\t" \
  "v_mfma_f32_16x16x32_bf16 v[208:211], v[80:83], v[12:15], v[208:211]\n\t" \
  "s_waitcnt lgkmcnt(4)\n\t" \
  "v_mfma_f32_16x16x32_bf16 v[208:211], v[96:99], v[16:19], v[208:211]\n\t" \
  "ds_read_b128 v[28:31], v11 offset:8192\n\t" \
  "v_mfma_f32_16x16x32_bf16 v[212:215], v[84:87], v[12:15], v[212:215]\n\t" \
  "ds_read_b64_tr_b16 v[76:77], v10 offset:16384\n\t" \
  "v_mfma_f32_16x16x32_bf16 v[212:215], v[100:103], v[16:19], v[212:215]\n\t" \
  "v_mfma_f32_16x16x32_bf16 v[216:219], v[88:91], v[12:15], v[216:219]\n\t" \
  "ds_read_b64_tr_b16 v[78:79], v10 offset:18432\n\t" \
  "v_mfma_f32_16x16x32_bf16 v[216:219], v[104:107], v[16:19], v[216:219]\n\t" \
  "v_mfma_f32_16x16x32_bf16 v[220:223], v[92:95], v[12:15], v[220:223]\n\t" \
  "ds_read_b128 v[32:35], v11 offset:18432\n\t" \
  "v_mfma_f32_16x16x32_bf16 v[220:223], v[108:111], v[16:19], v[220:223]\n\t" \
  "s_waitcnt lgkmcnt(7)\n\t" \
  "v_mfma_f32_16x16x32_bf16 v[224:227], v[80:83], v[20:23], v[224:227]\n\t" \
  "s_waitcnt lgkmcnt(4)\n\t" \
  "v_mfma_f32_16x16x32_bf16 v[224:227], v[96:99], v[24:27], v[224:227]\n\t" \
  "v_add_u32 v7, %[dl], v7\n\t" \
  "v_mfma_f32_16x16x32_bf16 v[228:231], v[84:87], v[20:23], v[228:231]\n\t" \
  "v_add_u32 v8, %[dl], v8\n\t" \
  "v_mfma_f32_16x16x32_bf16 v[228:231], v[100:103], v[24:27], v[228:231]\n\t" \
  "v_add_u32 v9, %[dl], v9\n\t" \
  "v_mfma_f32_16x16x32_bf16 v[232:235], v[88:91], v[20:23], v[232:235]\n\t" \
  "v_add_u32 v10, %[dl], v10\n\t" \
  "v_mfma_f32_16x16x32_bf16 v[232:235], v[104:107], v[24:27], v[232:235]\n\t" \
  "v_add_u32 v11, %[dl], v11\n\t" \
  "v_mfma_f32_16x16x32_bf16 v[236:239], v[92:95], v[20:23], v[236:239]\n\t" \
  "v_mfma_f32_16x16x32_bf16 v[236:239], v[108:111], v[24:27], v[236:239]\n\t" \
  "s_waitcnt lgkmcnt(0)\n\t" \
  "s_barrier\n\t" \
  "ds_read_b128 v[12:15], v11 offset:0\n\t" \
  "v_mfma_f32_16x16x32_bf16 v[240:243], v[80:83], v[28:31], v[240:243]\n\t" \
  "ds_read_b128 v[16:19], v11 offset:10240\n\t" \
  "v_mfma_f32_16x16x32_bf16 v[240:243], v[96:99], v[32:35], v[240:243]\n\t" \
  "ds_read_b128 v[20:23], v11 offset:1024\n\t" \
  "v_mfma_f32_16x16x32_bf16 v[244:247], v[84:87], v[28:31], v[244:247]\n\t" \
  "v_mfma_f32_16x16x32_bf16 v[244:247], v[100:103], v[32:35], v[244:247]\n\t" \
  "v_mfma_f32_16x16x32_bf16 v[248:251], v[88:91], v[28:31], v[248:251]\n\t" \
  "s_sub_u32 %[cnt], %[cnt], 1\n\t" \
  "v_mfma_f32_16x16x32_bf16 v[248:251], v[104:107], v[32:35], v[248:251]\n\t" \
  "s_cmp_lg_u32 %[cnt], 0\n\t" \
  "v_mfma_f32_16x16x32_bf16 v[252:255], v[92:95], v[28:31], v[252:255]\n\t" \
  "ds_read_b128 v[24:27], v11 offset:11264\n\t" \
  "v_mfma_f32_16x16x32_bf16 v[252:255], v[108:111], v[32:35], v[252:255]\n\t" \
  "s_cbranch_scc1 L_cons_loop9_%=\n\t" \
  "s_waitcnt lgkmcnt(3)\n\t" \
  "v_mfma_f32_16x16x32_bf16 v[112:115], v[48:51], v[12:15], v[112:115]\n\t" \
  "s_waitcnt lgkmcnt(2)\n\t" \
  "v_mfma_f32_16x16x32_bf16 v[112:115], v[64:67], v[16:19], v[112:115]\n\t" \
  "ds_read_b128 v[28:31], v11 offset:2048\n\t" \
  "v_mfma_f32_16x16x32_bf16 v[116:119], v[52:55], v[12:15], v[116:119]\n\t" \
  "ds_read_b64_tr_b16 v[80:81], v7 offset:0\n\t" \
  "v_mfma_f32_16x16x32_bf16 v[116:119], v[68:71], v[16:19], v[116:119]\n\t" \
  "s_add_u32 %[st], %[st], 1\n\t" \
  "v_mfma_f32_16x16x32_bf16 v[120:123], v[56:59], v[12:15], v[120:123]\n\t" \
  "ds_read_b64_tr_b16 v[82:83], v7 offset:2048\n\t" \
  "v_mfma_f32_16x16x32_bf16 v[120:123], v[72:75], v[16:19], v[120:123]\n\t" \
  "s_cmp_eq_u32 %[st], 3\n\t" \
  "v_mfma_f32_16x16x32_bf16 v[124:127], v[60:63], v[12:15], v[124:127]\n\t" \
  "ds_read_b128 v[32:35], v11 offset:12288\n\t" \
  "v_mfma_f32_16x16x32_bf16 v[124:127], v[76:79], v[16:19], v[124:127]\n\t" \
  "s_waitcnt lgkmcnt(5)\n\t" \
  "v_mfma_f32_16x16x32_bf16 v[128:131], v[48:51], v[20:23], v[128:131]\n\t" \
  "s_waitcnt lgkmcnt(4)\n\t" \
  "v_mfma_f32_16x16x32_bf16 v[128:131], v[64:67], v[24:27], v[128:131]\n\t" \
  "ds_read_b128 v[12:15], v11 offset:3072\n\t" \
  "v_mfma_f32_16x16x32_bf16 v[132:135], v[52:55], v[20:23], v[132:135]\n\t" \
  "ds_read_b64_tr_b16 v[96:97], v7 offset:16384\n\t" \
  "v_mfma_f32_16x16x32_bf16 v[132:135], v[68:71], v[24:27], v[132:135]\n\t" \
  "s_cselect_b32 %[dl], %[dn], %[dp]\n\t" \
  "v_mfma_f32_16x16x32_bf16 v[136:139], v[56:59], v[20:23], v[136:139]\n\t" \
  "ds_read_b64_tr_b16 v[98:99], v7 offset:18432\n\t" \
  "v_mfma_f32_16x16x32_bf16 v[136:139], v[72:75], v[24:27], v[136:139]\n\t" \
  "s_cselect_b32 %[st], 0, %[st]\n\t" \
  "v_mfma_f32_16x16x32_bf16 v[140:143], v[60:63], v[20:23], v[140:143]\n\t" \
  "ds_read_b128 v[16:19], v11 offset:13312\n\t" \
  "v_mfma_f32_16x16x32_bf16 v[140:143], v[76:79], v[24:27], v[140:143]\n\t" \
  "s_waitcnt lgkmcnt(7)\n\t" \
  "v_mfma_f32_16x16x32_bf16 v[144:147], v[48:51], v[28:31], v[144:147]\n\t" \
  "s_waitcnt lgkmcnt(4)\n\t" \
  "v_mfma_f32_16x16x32_bf16 v[144:147], v[64:67], v[32:35], v[144:147]\n\t" \
  "ds_read_b128 v[20:23], v11 offset:4096\n\t" \
  "v_mfma_f32_16x16x32_bf16 v[148:151], v[52:55], v[28:31], v[148:151]\n\t" \
  "ds_read_b64_tr_b16 v[84:85], v8 offset:0\n\t" \
  "v_mfma_f32_16x16x32_bf16 v[148:151], v[68:71], v[32:35], v[148:151]\n\t" \
  "v_mfma_f32_16x16x32_bf16 v[152:155], v[56:59], v[28:31], v[152:155]\n\t" \
  "ds_read_b64_tr_b16 v[86:87], v8 offset:2048\n\t" \
  "v_mfma_f32_16x16x32_bf16 v[152:155], v[72:75], v[32:35], v[152:155]\n\t" \
  "ds_read_b64_tr_b16 v[100:101], v8 offset:16384\n\t" \
  "v_mfma_f32_16x16x32_bf16 v[156:159], v[60:63], v[28:31], v[156:159]\n\t" \
  "ds_read_b128 v[24:27], v11 offset:14336\n\t" \
  "v_mfma_f32_16x16x32_bf16 v[156:159], v[76:79], v[32:35], v[156:159]\n\t" \
  "s_waitcnt lgkmcnt(8)\n\t" \
  "v_mfma_f32_16x16x32_bf16 v[160:163], v[48:51], v[12:15], v[160:163]\n\t" \
  "s_waitcnt lgkmcnt(5)\n\t" \
  "v_mfma_f32_16x16x32_bf16 v[160:163], v[64:67], v[16:19], v[160:163]\n\t" \
  "ds_read_b128 v[28:31], v11 offset:5120\n\t" \
  "v_mfma_f32_16x16x32_bf16 v[164:167], v[52:55], v[12:15], v[164:167]\n\t" \
  "ds_read_b64_tr_b16 v[102:103], v8 offset:18432\n\t" \
  "v_mfma_f32_16x16x32_bf16 v[164:167], v[68:71], v[16:19], v[164:167]\n\t" \
  "v_mfma_f32_16x16x32_bf16 v[168:171], v[56:59], v[12:15], v[168:171]\n\t" \
  "ds_read_b64_tr_b16 v[88:89], v9 offset:0\n\t" \
  "v_mfma_f32_16x16x32_bf16 v[168:171], v[72:75], v[16:19], v[168:171]\n\t" \
  "ds_read_b64_tr_b16 v[90:91], v9 offset:2048\n\t" \
  "v_mfma_f32_16x16x32_bf16 v[172:175], v[60:63], v[12:15], v[172:175]\n\t" \
  "ds_read_b128 v[32:35], v11 offset:15360\n\t" \
  "v_mfma_f32_16x16x32_bf16 v[172:175], v[76:79], v[16:19], v[172:175]\n\t" \
  "s_waitcnt lgkmcnt(9)\n\t" \
  "v_mfma_f32_16x16x32_bf16 v[176:179], v[48:51], v[20:23], v[176:179]\n\t" \
  "s_waitcnt lgkmcnt(5)\n\t" \
  "v_mfma_f32_16x16x32_bf16 v[176:179], v[64:67], v[24:27], v[176:179]\n\t" \
  "ds_read_b128 v[12:15], v11 offset:6144\n\t" \
  "v_mfma_f32_16x16x32_bf16 v[180:183], v[52:55], v[20:23], v[180:183]\n\t" \
  "ds_read_b64_tr_b16 v[104:105], v9 offset:16384\n\t" \
  "v_mfma_f32_16x16x32_bf16 v[180:183], v[68:71], v[24:27], v[180:183]\n\t" \
  "v_mfma_f32_16x16x32_bf16 v[184:187], v[56:59], v[20:23], v[184:187]\n\t" \
  "ds_read_b64_tr_b16 v[106:107], v9 offset:18432\n\t" \
  "v_mfma_f32_16x16x32_bf16 v[184:187], v[72:75], v[24:27], v[184:187]\n\t" \
  "v_mfma_f32_16x16x32_bf16 v[188:191], v[60:63], v[20:23], v[188:191]\n\t" \
  "ds_read_b128 v[16:19], v11 offset:16384\n\t" \
  "v_mfma_f32_16x16x32_bf16 v[188:191], v[76:79], v[24:27], v[188:191]\n\t" \
  "s_waitcnt lgkmcnt(8)\n\t" \
  "v_mfma_f32_16x16x32_bf16 v[192:195], v[48:51], v[28:31], v[192:195]\n\t" \
  "s_waitcnt lgkmcnt(4)\n\t" \
  "v_mfma_f32_16x16x32_bf16 v[192:195], v[64:67], v[32:35], v[192:195]\n\t" \
  "ds_read_b128 v[20:23], v11 offset:7168\n\t" \
  "v_mfma_f32_16x16x32_bf16 v[196:199], v[52:55], v[28:31], v[196:199]\n\t" \
  "ds_read_b64_tr_b16 v[92:93], v10 offset:0\n\t" \
  "v_mfma_f32_16x16x32_bf16 v[196:199], v[68:71], v[32:35], v[196:199]\n\t" \
  "v_mfma_f32_16x16x32_bf16 v[200:203], v[56:59], v[28:31], v[200:203]\n\t" \
  "ds_read_b64_tr_b16 v[94:95], v10 offset:2048\n\t" \
  "v_mfma_f32_16x16x32_bf16 v[200:203], v[72:75], v[32:35], v[200:203]\n\t" \
  "v_mfma_f32_16x16x32_bf16 v[204:207], v[60:63], v[28:31], v[204:207]\n\t" \
  "ds_read_b128 v[24:27], v11 offset:17408\n\t" \
  "v_mfma_f32_16x16x32_bf16 v[204:207], v[76:79], v[32:35], v[204:207]\n\t" \
  "s_waitcnt lgkmcnt(7)\n\t" \
  "v_mfma_f32_16x16x32_bf16 v[208:211], v[48:51], v[12:15], v[208:211]\n\t" \
  "s_waitcnt lgkmcnt(4)\n\t" \
  "v_mfma_f32_16x16x32_bf16 v[208:211], v[64:67], v[16:19], v[208:211]\n\t" \
  "ds_read_b128 v[28:31], v11 offset:8192\n\t" \
  "v_mfma_f32_16x16x32_bf16 v[212:215], v[52:55], v[12:15], v[212:215]\n\t" \
  "ds_read_b64_tr_b16 v[108:109], v10 offset:16384\n\t" \
  "v_mfma_f32_16x16x32_bf16 v[212:215], v[68:71], v[16:19], v[212:215]\n\t" \
  "v_mfma_f32_16x16x32_bf16 v[216:219], v[56:59], v[12:15], v[216:219]\n\t" \
  "ds_read_b64_tr_b16 v[110:111], v10 offset:18432\n\t" \
  "v_mfma_f32_16x16x32_bf16 v[216:219], v[72:75], v[16:19], v[216:219]\n\t" \
  "v_mfma_f32_16x16x32_bf16 v[220:223], v[60:63], v[12:15], v[220:223]\n\t" \
  "ds_read_b128 v[32:35], v11 offset:18432\n\t" \
  "v_mfma_f32_16x16x32_bf16 v[220:223], v[76:79], v[16:19], v[220:223]\n\t" \
  "s_waitcnt lgkmcnt(7)\n\t" \
  "v_mfma_f32_16x16x32_bf16 v[224:227], v[48:51], v[20:23], v[224:227]\n\t" \
  "s_waitcnt lgkmcnt(4)\n\t" \
  "v_mfma_f32_16x16x32_bf16 v[224:227], v[64:67], v[24:27], v[224:227]\n\t" \
  "v_add_u32 v7, %[dl], v7\n\t" \
  "v_mfma_f32_16x16x32_bf16 v[228:231], v[52:55], v[20:23], v[228:231]\n\t" \
  "v_add_u32 v8, %[dl], v8\n\t" \
  "v_mfma_f32_16x16x32_bf16 v[228:231], v[68:71], v[24:27], v[228:231]\n\t" \
  "v_add_u32 v9, %[dl], v9\n\t" \
  "v_mfma_f32_16x16x32_bf16 v[232:235], v[56:59], v[20:23], v[232:235]\n\t" \
  "v_add_u32 v10, %[dl], v10\n\t" \
  "v_mfma_f32_16x16x32_bf16 v[232:235], v[72:75], v[24:27], v[232:235]\n\t" \
  "v_add_u32 v11, %[dl], v11\n\t" \
  "v_mfma_f32_16x16x32_bf16 v[236:239], v[60:63], v[20:23], v[236:239]\n\t" \
  "v_mfma_f32_16x16x32_bf16 v[236:239], v[76:79], v[24:27], v[236:239]\n\t" \
  "s_waitcnt lgkmcnt(0)\n\t" \
  "s_barrier\n\t" \
  "ds_read_b128 v[12:15], v11 offset:0\n\t" \
  "v_mfma_f32_16x16x32_bf16 v[240:243], v[48:51], v[28:31], v[240:243]\n\t" \
  "ds_read_b128 v[16:19], v11 offset:10240\n\t" \
  "v_mfma_f32_16x16x32_bf16 v[240:243], v[64:67], v[32:35], v[240:243]\n\t" \
  "ds_read_b128 v[20:23], v11 offset:1024\n\t" \
  "v_mfma_f32_16x16x32_bf16 v[244:247], v[52:55], v[28:31], v[244:247]\n\t" \
  "v_mfma_f32_16x16x32_bf16 v[244:247], v[68:71], v[32:35], v[244:247]\n\t" \
  "v_mfma_f32_16x16x32_bf16 v[248:251], v[56:59], v[28:31], v[248:251]\n\t" \
  "v_mfma_f32_16x16x32_bf16 v[248:251], v[72:75], v[32:35], v[248:251]\n\t" \
  "v_mfma_f32_16x16x32_bf16 v[252:255], v[60:63], v[28:31], v[252:255]\n\t" \
  "ds_read_b128 v[24:27], v11 offset:11264\n\t" \
  "v_mfma_f32_16x16x32_bf16 v[252:255], v[76:79], v[32:35], v[252:255]\n\t" \
  "s_waitcnt lgkmcnt(3)\n\t" \
  "v_mfma_f32_16x16x32_bf16 v[112:115], v[80:83], v[12:15], v[112:115]\n\t" \
  "s_waitcnt lgkmcnt(2)\n\t" \
  "v_mfma_f32_16x16x32_bf16 v[112:115], v[96:99], v[16:19], v[112:115]\n\t" \
  "ds_read_b128 v[28:31], v11 offset:2048\n\t" \
  "v_mfma_f32_16x16x32_bf16 v[116:119], v[84:87], v[12:15], v[116:119]\n\t" \
  "ds_read_b64_tr_b16 v[48:49], v7 offset:0\n\t" \
  "v_mfma_f32_16x16x32_bf16 v[116:119], v[100:103], v[16:19], v[116:119]\n\t" \
  "s_add_u32 %[st], %[st], 1\n\t" \
  "v_mfma_f32_16x16x32_bf16 v[120:123], v[88:91], v[12:15], v[120:123]\n\t" \
  "ds_read_b64_tr_b16 v[50:51], v7 offset:2048\n\t" \
  "v_mfma_f32_16x16x32_bf16 v[120:123], v[104:107], v[16:19], v[120:123]\n\t" \
  "s_cmp_eq_u32 %[st], 3\n\t" \
  "v_mfma_f32_16x16x32_bf16 v[124:127], v[92:95], v[12:15], v[124:127]\n\t" \
  "ds_read_b128 v[32:35], v11 offset:12288\n\t" \
  "v_mfma_f32_16x16x32_bf16 v[124:127], v[108:111], v[16:19], v[124:127]\n\t" \
  "s_waitcnt lgkmcnt(5)\n\t" \
  "v_mfma_f32_16x16x32_bf16 v[128:131], v[80:83], v[20:23], v[128:131]\n\t" \
  "s_waitcnt lgkmcnt(4)\n\t" \
  "v_mfma_f32_16x16x32_bf16 v[128:131], v[96:99], v[24:27], v[128:131]\n\t" \
  "ds_read_b128 v[12:15], v11 offset:3072\n\t" \
  "v_mfma_f32_16x16x32_bf16 v[132:135], v[84:87], v[20:23], v[132:135]\n\t" \
  "ds_read_b64_tr_b16 v[64:65], v7 offset:16384\n\t" \
  "v_mfma_f32_16x16x32_bf16 v[132:135], v[100:103], v[24:27], v[132:135]\n\t" \
  "s_cselect_b32 %[dl], %[dn], %[dp]\n\t" \
  "v_mfma_f32_16x16x32_bf16 v[136:139], v[88:91], v[20:23], v[136:139]\n\t" \
  "ds_read_b64_tr_b16 v[66:67], v7 offset:18432\n\t" \
  "v_mfma_f32_16x16x32_bf16 v[136:139], v[104:107], v[24:27], v[136:139]\n\t" \
  "s_cselect_b32 %[st], 0, %[st]\n\t" \
  "v_mfma_f32_16x16x32_bf16 v[140:143], v[92:95], v[20:23], v[140:143]\n\t" \
  "ds_read_b128 v[16:19], v11 offset:13312\n\t" \
  "v_mfma_f32_16x16x32_bf16 v[140:143], v[108:111], v[24:27], v[140:143]\n\t" \
  "s_waitcnt lgkmcnt(7)\n\t" \
  "v_mfma_f32_16x16x32_bf16 v[144:147], v[80:83], v[28:31], v[144:147]\n\t" \
  "s_waitcnt lgkmcnt(4)\n\t" \
  "v_mfma_f32_16x16x32_bf16 v[144:147], v[96:99], v[32:35], v[144:147]\n\t" \
  "ds_read_b128 v[20:23], v11 offset:4096\n\t" \
  "v_mfma_f32_16x16x32_bf16 v[148:151], v[84:87], v[28:31], v[148:151]\n\t" \
  "ds_read_b64_tr_b16 v[52:53], v8 offset:0\n\t" \
  "v_mfma_f32_16x16x32_bf16 v[148:151], v[100:103], v[32:35], v[148:151]\n\t" \
  "v_mfma_f32_16x16x32_bf16 v[152:155], v[88:91], v[28:31], v[152:155]\n\t" \
  "ds_read_b64_tr_b16 v[54:55], v8 offset:2048\n\t" \
  "v_mfma_f32_16x16x32_bf16 v[152:155], v[104:107], v[32:35], v[152:155]\n\t" \
  "ds_read_b64_tr_b16 v[68:69], v8 offset:16384\n\t" \
  "v_mfma_f32_16x16x32_bf16 v[156:159], v[92:95], v[28:31], v[156:159]\n\t" \
  "ds_read_b128 v[24:27], v11 offset:14336\n\t" \
  "v_mfma_f32_16x16x32_bf16 v[156:159], v[108:111], v[32:35], v[156:159]\n\t" \
  "s_waitcnt lgkmcnt(8)\n\t" \
  "v_mfma_f32_16x16x32_bf16 v[160:163], v[80:83], v[12:15], v[160:163]\n\t" \
  "s_waitcnt lgkmcnt(5)\n\t" \
  "v_mfma_f32_16x16x32_bf16 v[160:163], v[96:99], v[16:19], v[160:163]\n\t" \
  "ds_read_b128 v[28:31], v11 offset:5120\n\t" \
  "v_mfma_f32_16x16x32_bf16 v[164:167], v[84:87], v[12:15], v[164:167]\n\t" \
  "ds_read_b64_tr_b16 v[70:71], v8 offset:18432\n\t" \
  "v_mfma_f32_16x16x32_bf16 v[164:167], v[100:103], v[16:19], v[164:167]\n\t" \
  "v_mfma_f32_16x16x32_bf16 v[168:171], v[88:91], v[12:15], v[168:171]\n\t" \
  "ds_read_b64_tr_b16 v[56:57], v9 offset:0\n\t" \
  "v_mfma_f32_16x16x32_bf16 v[168:171], v[104:107], v[16:19], v[168:171]\n\t" \
  "ds_read_b64_tr_b16 v[58:59], v9 offset:2048\n\t" \
  "v_mfma_f32_16x16x32_bf16 v[172:175], v[92:95], v[12:15], v[172:175]\n\t" \
  "ds_read_b128 v[32:35], v11 offset:15360\n\t" \
  "v_mfma_f32_16x16x32_bf16 v[172:175], v[108:111], v[16:19], v[172:175]\n\t" \
  "s_waitcnt lgkmcnt(9)\n\t" \
  "v_mfma_f32_16x16x32_bf16 v[176:179], v[80:83], v[20:23], v[176:179]\n\t" \
  "s_waitcnt lgkmcnt(5)\n\t" \
  "v_mfma_f32_16x16x32_bf16 v[176:179], v[96:99], v[24:27], v[176:179]\n\t" \
  "ds_read_b128 v[12:15], v11 offset:6144\n\t" \
  "v_mfma_f32_16x16x32_bf16 v[180:183], v[84:87], v[20:23], v[180:183]\n\t" \
  "ds_read_b64_tr_b16 v[72:73], v9 offset:16384\n\t" \
  "v_mfma_f32_16x16x32_bf16 v[180:183], v[100:103], v[24:27], v[180:183]\n\t" \
  "v_mfma_f32_16x16x32_bf16 v[184:187], v[88:91], v[20:23], v[184:187]\n\t" \
  "ds_read_b64_tr_b16 v[74:75], v9 offset:18432\n\t" \
  "v_mfma_f32_16x16x32_bf16 v[184:187], v[104:107], v[24:27], v[184:187]\n\t" \
  "v_mfma_f32_16x16x32_bf16 v[188:191], v[92:95], v[20:23], v[188:191]\n\t" \
  "ds_read_b128 v[16:19], v11 offset:16384\n\t" \
  "v_mfma_f32_16x16x32_bf16 v[188:191], v[108:111], v[24:27], v[188:191]\n\t" \
  "s_waitcnt lgkmcnt(8)\n\t" \
  "v_mfma_f32_16x16x32_bf16 v[192:195], v[80:83], v[28:31], v[192:195]\n\t" \
  "s_waitcnt lgkmcnt(4)\n\t" \
  "v_mfma_f32_16x16x32_bf16 v[192:195], v[96:99], v[32:35], v[192:195]\n\t" \
  "ds_read_b128 v[20:23], v11 offset:7168\n\t" \
  "v_mfma_f32_16x16x32_bf16 v[196:199], v[84:87], v[28:31], v[196:199]\n\t" \
  "ds_read_b64_tr_b16 v[60:61], v10 offset:0\n\t" \
  "v_mfma_f32_16x16x32_bf16 v[196:199], v[100:103], v[32:35], v[196:199]\n\t" \
  "v_mfma_f32_16x16x32_bf16 v[200:203], v[88:91], v[28:31], v[200:203]\n\t" \
  "ds_read_b64_tr_b16 v[62:63], v10 offset:2048\n\t" \
  "v_mfma_f32_16x16x32_bf16 v[200:203], v[104:107], v[32:35], v[200:203]\n\t" \
  "v_mfma_f32_16x16x32_bf16 v[204:207], v[92:95], v[28:31], v[204:207]\n\t" \
  "ds_read_b128 v[24:27], v11 offset:17408\n\t" \
  "v_mfma_f32_16x16x32_bf16 v[204:207], v[108:111], v[32:35], v[204:207]\n\t" \
  "s_waitcnt lgkmcnt(7)\n\t" \
  "v_mfma_f32_16x16x32_bf16 v[208:211], v[80:83], v[12:15], v[208:211]\n\t" \
  "s_waitcnt lgkmcnt(4)\n\t" \
  "v_mfma_f32_16x16x32_bf16 v[208:211], v[96:99], v[16:19], v[208:211]\n\t" \
  "ds_read_b128 v[28:31], v11 offset:8192\n\t" \
  "v_mfma_f32_16x16x32_bf16 v[212:215], v[84:87], v[12:15], v[212:215]\n\t" \
  "ds_read_b64_tr_b16 v[76:77], v10 offset:16384\n\t" \
  "v_mfma_f32_16x16x32_bf16 v[212:215], v[100:103], v[16:19], v[212:215]\n\t" \
  "v_mfma_f32_16x16x32_bf16 v[216:219], v[88:91], v[12:15], v[216:219]\n\t" \
  "ds_read_b64_tr_b16 v[78:79], v10 offset:18432\n\t" \
  "v_mfma_f32_16x16x32_bf16 v[216:219], v[104:107], v[16:19], v[216:219]\n\t" \
  "v_mfma_f32_16x16x32_bf16 v[220:223], v[92:95], v[12:15], v[220:223]\n\t" \
  "ds_read_b128 v[32:35], v11 offset:18432\n\t" \
  "v_mfma_f32_16x16x32_bf16 v[220:223], v[108:111], v[16:19], v[220:223]\n\t" \
  "s_waitcnt lgkmcnt(7)\n\t" \
  "v_mfma_f32_16x16x32_bf16 v[224:227], v[80:83], v[20:23], v[224:227]\n\t" \
  "s_waitcnt lgkmcnt(4)\n\t" \
  "v_mfma_f32_16x16x32_bf16 v[224:227], v[96:99], v[24:27], v[224:227]\n\t" \
  "v_add_u32 v7, %[dl], v7\n\t" \
  "v_mfma_f32_16x16x32_bf16 v[228:231], v[84:87], v[20:23], v[228:231]\n\t" \
  "v_add_u32 v8, %[dl], v8\n\t" \
  "v_mfma_f32_16x16x32_bf16 v[228:231], v[100:103], v[24:27], v[228:231]\n\t" \
  "v_add_u32 v9, %[dl], v9\n\t" \
  "v_mfma_f32_16x16x32_bf16 v[232:235], v[88:91], v[20:23], v[232:235]\n\t" \
  "v_add_u32 v10, %[dl], v10\n\t" \
  "v_mfma_f32_16x16x32_bf16 v[232:235], v[104:107], v[24:27], v[232:235]\n\t" \
  "v_add_u32 v11, %[dl], v11\n\t" \
  "v_mfma_f32_16x16x32_bf16 v[236:239], v[92:95], v[20:23], v[236:239]\n\t" \
  "v_mfma_f32_16x16x32_bf16 v[236:239], v[108:111], v[24:27], v[236:239]\n\t" \
  "s_waitcnt lgkmcnt(0)\n\t" \
  "s_barrier\n\t" \
  "ds_read_b128 v[12:15], v11 offset:0\n\t" \
  "v_mfma_f32_16x16x32_bf16 v[240:243], v[80:83], v[28:31], v[240:243]\n\t" \
  "ds_read_b128 v[16:19], v11 offset:10240\n\t" \
  "ds_read_b128 v[20:23], v5 offset:0\n\t" \
  "v_mfma_f32_16x16x32_bf16 v[240:243], v[96:99], v[32:35], v[240:243]\n\t" \
  "ds_read_b64_tr_b16 v[80:81], v7 offset:0\n\t" \
  "ds_read_b64_tr_b16 v[82:83], v7 offset:2048\n\t" \
  "v_mfma_f32_16x16x32_bf16 v[244:247], v[84:87], v[28:31], v[244:247]\n\t" \
  "ds_read_b64_tr_b16 v[96:97], v7 offset:16384\n\t" \
  "ds_read_b64_tr_b16 v[98:99], v7 offset:18432\n\t" \
  "v_mfma_f32_16x16x32_bf16 v[244:247], v[100:103], v[32:35], v[244:247]\n\t" \
  "ds_read_b64_tr_b16 v[84:85], v8 offset:0\n\t" \
  "ds_read_b64_tr_b16 v[86:87], v8 offset:2048\n\t" \
  "v_mfma_f32_16x16x32_bf16 v[248:251], v[88:91], v[28:31], v[248:251]\n\t" \
  "ds_read_b64_tr_b16 v[100:101], v8 offset:16384\n\t" \
  "ds_read_b64_tr_b16 v[102:103], v8 offset:18432\n\t" \
  "v_mfma_f32_16x16x32_bf16 v[248:251], v[104:107], v[32:35], v[248:251]\n\t" \
  "ds_read_b64_tr_b16 v[88:89], v9 offset:0\n\t" \
  "ds_read_b64_tr_b16 v[90:91], v9 offset:2048\n\t" \
  "v_mfma_f32_16x16x32_bf16 v[252:255], v[92:95], v[28:31], v[252:255]\n\t" \
  "ds_read_b128 v[24:27], v5 offset:10240\n\t" \
  "ds_read_b64_tr_b16 v[104:105], v9 offset:16384\n\t" \
  "ds_read_b64_tr_b16 v[106:107], v9 offset:18432\n\t" \
  "v_mfma_f32_16x16x32_bf16 v[252:255], v[108:111], v[32:35], v[252:255]\n\t" \
  "ds_read_b64_tr_b16 v[92:93], v10 offset:0\n\t" \
  "ds_read_b64_tr_b16 v[94:95], v10 offset:2048\n\t" \
  "s_waitcnt lgkmcnt(15)\n\t" \
  "v_mfma_f32_16x16x32_bf16 v[112:115], v[48:51], v[12:15], v[112:115]\n\t" \
  "ds_read_b64_tr_b16 v[108:109], v10 offset:16384\n\t" \
  "ds_read_b64_tr_b16 v[110:111], v10 offset:18432\n\t" \
  "s_waitcnt lgkmcnt(15)\n\t" \
  "v_mfma_f32_16x16x32_bf16 v[112:115], v[64:67], v[16:19], v[112:115]\n\t" \
  "ds_read_b128 v[28:31], v11 offset:1024\n\t" \
  "v_mfma_f32_16x16x32_bf16 v[116:119], v[52:55], v[12:15], v[116:119]\n\t" \
  "v_mov_b32 v6, %[bad]\n\t" \
  "ds_read_b32 v6, v6\n\t" \
  "v_mfma_f32_16x16x32_bf16 v[116:119], v[68:71], v[16:19], v[116:119]\n\t" \
  "v_mfma_f32_16x16x32_bf16 v[120:123], v[56:59], v[12:15], v[120:123]\n\t" \
  "v_mfma_f32_16x16x32_bf16 v[120:123], v[72:75], v[16:19], v[120:123]\n\t" \
  "v_mfma_f32_16x16x32_bf16 v[124:127], v[60:63], v[12:15], v[124:127]\n\t" \
  "ds_read_b128 v[32:35], v11 offset:11264\n\t" \
  "v_mfma_f32_16x16x32_bf16 v[124:127], v[76:79], v[16:19], v[124:127]\n\t" \
  "s_waitcnt lgkmcnt(15)\n\t" \
  "v_mfma_f32_16x16x32_bf16 v[112:115], v[80:83], v[20:23], v[112:115]\n\t" \
  "s_waitcnt lgkmcnt(9)\n\t" \
  "v_mfma_f32_16x16x32_bf16 v[112:115], v[96:99], v[24:27], v[112:115]\n\t" \
  "v_xor_b32 v7, 0, %[ew]\n\t" \
  "ds_read_b128 v[12:15], v5 offset:1024\n\t" \
  "v_mfma_f32_16x16x32_bf16 v[116:119], v[84:87], v[20:23], v[116:119]\n\t" \
  "v_xor_b32 v8, 32, %[ew]\n\t" \
  "v_mfma_f32_16x16x32_bf16 v[116:119], v[100:103], v[24:27], v[116:119]\n\t" \
  "v_xor_b32 v9, 64, %[ew]\n\t" \
  "v_mfma_f32_16x16x32_bf16 v[120:123], v[88:91], v[20:23], v[120:123]\n\t" \
  "v_xor_b32 v10, 96, %[ew]\n\t" \
  "s_waitcnt lgkmcnt(8)\n\t" \
  "v_mfma_f32_16x16x32_bf16 v[120:123], v[104:107], v[24:27], v[120:123]\n\t" \
  "s_waitcnt lgkmcnt(6)\n\t" \
  "v_mfma_f32_16x16x32_bf16 v[124:127], v[92:95], v[20:23], v[124:127]\n\t" \
  "ds_read_b128 v[16:19], v5 offset:11264\n\t" \
  "s_waitcnt lgkmcnt(5)\n\t" \
  "v_mfma_f32_16x16x32_bf16 v[124:127], v[108:111], v[24:27], v[124:127]\n\t" \
  "s_waitcnt lgkmcnt(3)\n\t" \
  "v_mfma_f32_16x16x32_bf16 v[128:131], v[48:51], v[28:31], v[128:131]\n\t" \
  "v_cvt_pk_bf16_f32 v112, v112, v113\n\t" \
  "v_cvt_pk_bf16_f32 v113, v114, v115\n\t" \
  "s_waitcnt lgkmcnt(2)\n\t" \
  "v_mfma_f32_16x16x32_bf16 v[128:131], v[64:67], v[32:35], v[128:131]\n\t" \
  "ds_read_b128 v[20:23], v11 offset:2048\n\t" \
  "v_pk_max_i16 v112, v112, v6\n\t" \
  "v_pk_max_i16 v113, v113, v6\n\t" \
  "v_mfma_f32_16x16x32_bf16 v[132:135], v[52:55], v[28:31], v[132:135]\n\t" \
  "ds_write_b64 v7, v[112:113] offset:0\n\t" \
  "v_mfma_f32_16x16x32_bf16 v[132:135], v[68:71], v[32:35], v[132:135]\n\t" \
  "v_cvt_pk_bf16_f32 v116, v116, v117\n\t" \
  "v_cvt_pk_bf16_f32 v117, v118, v119\n\t" \
  "v_mfma_f32_16x16x32_bf16 v[136:139], v[56:59], v[28:31], v[136:139]\n\t" \
  "v_pk_max_i16 v116, v116, v6\n\t" \
  "v_pk_max_i16 v117, v117, v6\n\t" \
  "v_mfma_f32_16x16x32_bf16 v[136:139], v[72:75], v[32:35], v[136:139]\n\t" \
  "ds_write_b64 v8, v[116:117] offset:0\n\t" \
  "v_mfma_f32_16x16x32_bf16 v[140:143], v[60:63], v[28:31], v[140:143]\n\t" \
  "ds_read_b128 v[24:27], v11 offset:12288\n\t" \
  "v_cvt_pk_bf16_f32 v120, v120, v121\n\t" \
  "v_cvt_pk_bf16_f32 v121, v122, v123\n\t" \
  "v_mfma_f32_16x16x32_bf16 v[140:143], v[76:79], v[32:35], v[140:143]\n\t" \
  "v_pk_max_i16 v120, v120, v6\n\t" \
  "v_pk_max_i16 v121, v121, v6\n\t" \
  "s_waitcnt lgkmcnt(5)\n\t" \
  "v_mfma_f32_16x16x32_bf16 v[128:131], v[80:83], v[12:15], v[128:131]\n\t" \
  "ds_write_b64 v9, v[120:121] offset:0\n\t" \
  "s_waitcnt lgkmcnt(5)\n\t" \
  "v_mfma_f32_16x16x32_bf16 v[128:131], v[96:99], v[16:19], v[128:131]\n\t" \
  "ds_read_b128 v[28:31], v5 offset:2048\n\t" \
  "v_cvt_pk_bf16_f32 v124, v124, v125\n\t" \
  "v_cvt_pk_bf16_f32 v125, v126, v127\n\t" \
  "v_mfma_f32_16x16x32_bf16 v[132:135], v[84:87], v[12:15], v[132:135]\n\t" \
  "v_pk_max_i16 v124, v124, v6\n\t" \
  "v_pk_max_i16 v125, v125, v6\n\t" \
  "v_mfma_f32_16x16x32_bf16 v[132:135], v[100:103], v[16:19], v[132:135]\n\t" \
  "ds_write_b64 v10, v[124:125] offset:0\n\t" \
  "v_mfma_f32_16x16x32_bf16 v[136:139], v[88:91], v[12:15], v[136:139]\n\t" \
  "ds_read_b128 v[112:115], %[er] offset:0\n\t" \
  "ds_read_b128 v[116:119], %[er] offset:1024\n\t" \
  "v_mfma_f32_16x16x32_bf16 v[136:139], v[104:107], v[16:19], v[136:139]\n\t" \
  "v_mfma_f32_16x16x32_bf16 v[140:143], v[92:95], v[12:15], v[140:143]\n\t" \
  "ds_read_b128 v[32:35], v5 offset:12288\n\t" \
  "v_mfma_f32_16x16x32_bf16 v[140:143], v[108:111], v[16:19], v[140:143]\n\t" \
  "s_waitcnt lgkmcnt(9)\n\t" \
  "v_mfma_f32_16x16x32_bf16 v[144:147], v[48:51], v[20:23], v[144:147]\n\t" \
  "v_cvt_pk_bf16_f32 v128, v128, v129\n\t" \
  "v_cvt_pk_bf16_f32 v129, v130, v131\n\t" \
  "s_waitcnt lgkmcnt(6)\n\t" \
  "v_mfma_f32_16x16x32_bf16 v[144:147], v[64:67], v[24:27], v[144:147]\n\t" \
  "ds_read_b128 v[12:15], v11 offset:3072\n\t" \
  "v_pk_max_i16 v128, v128, v6\n\t" \
  "v_pk_max_i16 v129, v129, v6\n\t" \
  "v_mfma_f32_16x16x32_bf16 v[148:151], v[52:55], v[20:23], v[148:151]\n\t" \
  "ds_write_b64 v7, v[128:129] offset:2048\n\t" \
  "v_mfma_f32_16x16x32_bf16 v[148:151], v[68:71], v[24:27], v[148:151]\n\t" \
  "v_cvt_pk_bf16_f32 v132, v132, v133\n\t" \
  "v_cvt_pk_bf16_f32 v133, v134, v135\n\t" \
  "s_waitcnt lgkmcnt(4)\n\t" \
  "buffer_store_dwordx4 v[112:115], %[vo], %[out], %[so] offen" OUT_POL "\n\t" \
  "s_add_u32 %[so], %[so], 32768\n\t" \
  "s_and_b32 %[so], %[so], 0xffffff\n\t" \
  "v_mfma_f32_16x16x32_bf16 v[152:155], v[56:59], v[20:23], v[152:155]\n\t" \
  "v_pk_max_i16 v132, v132, v6\n\t" \
  "v_pk_max_i16 v133, v133, v6\n\t" \
  "v_mfma_f32_16x16x32_bf16 v[152:155], v[72:75], v[24:27], v[152:155]\n\t" \
  "ds_write_b64 v8, v[132:133] offset:2048\n\t" \
  "s_waitcnt lgkmcnt(4)\n\t" \
  "buffer_store_dwordx4 v[116:119], %[vo], %[out], %[so] offen" OUT_POL "\n\t" \
  "s_add_u32 %[so], %[so], 32768\n\t" \
  "s_and_b32 %[so], %[so], 0xffffff\n\t" \
  "v_mfma_f32_16x16x32_bf16 v[156:159], v[60:63], v[20:23], v[156:159]\n\t" \
  "ds_read_b128 v[16:19], v11 offset:13312\n\t" \
  "v_cvt_pk_bf16_f32 v136, v136, v137\n\t" \
  "v_cvt_pk_bf16_f32 v137, v138, v139\n\t" \
  "v_mfma_f32_16x16x32_bf16 v[156:159], v[76:79], v[24:27], v[156:159]\n\t" \
  "v_pk_max_i16 v136, v136, v6\n\t" \
  "v_pk_max_i16 v137, v137, v6\n\t" \
  "v_mfma_f32_16x16x32_bf16 v[144:147], v[80:83], v[28:31], v[144:147]\n\t" \
  "ds_write_b64 v9, v[136:137] offset:2048\n\t" \
  "s_waitcnt lgkmcnt(5)\n\t" \
  "v_mfma_f32_16x16x32_bf16 v[144:147], v[96:99], v[32:35], v[144:147]\n\t" \
  "ds_read_b128 v[20:23], v5 offset:3072\n\t" \
  "v_cvt_pk_bf16_f32 v140, v140, v141\n\t" \
  "v_cvt_pk_bf16_f32 v141, v142, v143\n\t" \
  "v_mfma_f32_16x16x32_bf16 v[148:151], v[84:87], v[28:31], v[148:151]\n\t" \
  "v_pk_max_i16 v140, v140, v6\n\t" \
  "v_pk_max_i16 v141, v141, v6\n\t" \
  "v_mfma_f32_16x16x32_bf16 v[148:151], v[100:103], v[32:35], v[148:151]\n\t" \
  "ds_write_b64 v10, v[140:141] offset:2048\n\t" \
  "v_mfma_f32_16x16x32_bf16 v[152:155], v[88:91], v[28:31], v[152:155]\n\t" \
  "ds_read_b128 v[128:131], %[er] offset:2048\n\t" \
  "ds_read_b128 v[132:135], %[er] offset:3072\n\t" \
  "v_mfma_f32_16x16x32_bf16 v[152:155], v[104:107], v[32:35], v[152:155]\n\t" \
  "v_mfma_f32_16x16x32_bf16 v[156:159], v[92:95], v[28:31], v[156:159]\n\t" \
  "ds_read_b128 v[24:27], v5 offset:13312\n\t" \
  "v_mfma_f32_16x16x32_bf16 v[156:159], v[108:111], v[32:35], v[156:159]\n\t" \
  "s_waitcnt lgkmcnt(9)\n\t" \
  "v_mfma_f32_16x16x32_bf16 v[160:163], v[48:51], v[12:15], v[160:163]\n\t" \
  "v_cvt_pk_bf16_f32 v144, v144, v145\n\t" \
  "v_cvt_pk_bf16_f32 v145, v146, v147\n\t" \
  "s_waitcnt lgkmcnt(6)\n\t" \
  "v_mfma_f32_16x16x32_bf16 v[160:163], v[64:67], v[16:19], v[160:163]\n\t" \
  "ds_read_b128 v[28:31], v11 offset:4096\n\t" \
  "v_pk_max_i16 v144, v144, v6\n\t" \
  "v_pk_max_i16 v145, v145, v6\n\t" \
  "v_mfma_f32_16x16x32_bf16 v[164:167], v[52:55], v[12:15], v[164:167]\n\t" \
  "ds_write_b64 v7, v[144:145] offset:4096\n\t" \
  "v_mfma_f32_16x16x32_bf16 v[164:167], v[68:71], v[16:19], v[164:167]\n\t" \
  "v_cvt_pk_bf16_f32 v148, v148, v149\n\t" \
  "v_cvt_pk_bf16_f32 v149, v150, v151\n\t" \
  "s_waitcnt lgkmcnt(4)\n\t" \
  "buffer_store_dwordx4 v[128:131], %[vo], %[out], %[so] offen" OUT_POL "\n\t" \
  "s_add_u32 %[so], %[so], 32768\n\t" \
  "s_and_b32 %[so], %[so], 0xffffff\n\t" \
  "v_mfma_f32_16x16x32_bf16 v[168:171], v[56:59], v[12:15], v[168:171]\n\t" \
  "v_pk_max_i16 v148, v148, v6\n\t" \
  "v_pk_max_i16 v149, v149, v6\n\t" \
  "v_mfma_f32_16x16x32_bf16 v[168:171], v[72:75], v[16:19], v[168:171]\n\t" \
  "ds_write_b64 v8, v[148:149] offset:4096\n\t" \
  "s_waitcnt lgkmcnt(4)\n\t" \
  "buffer_store_dwordx4 v[132:135], %[vo], %[out], %[so] offen" OUT_POL "\n\t" \
  "s_add_u32 %[so], %[so], 32768\n\t" \
  "s_and_b32 %[so], %[so], 0xffffff\n\t" \
  "v_mfma_f32_16x16x32_bf16 v[172:175], v[60:63], v[12:15], v[172:175]\n\t" \
  "ds_read_b128 v[32:35], v11 offset:14336\n\t" \
  "v_cvt_pk_bf16_f32 v152, v152, v153\n\t" \
  "v_cvt_pk_bf16_f32 v153, v154, v155\n\t" \
  "v_mfma_f32_16x16x32_bf16 v[172:175], v[76:79], v[16:19], v[172:175]\n\t" \
  "v_pk_max_i16 v152, v152, v6\n\t" \
  "v_pk_max_i16 v153, v153, v6\n\t" \
  "v_mfma_f32_16x16x32_bf16 v[160:163], v[80:83], v[20:23], v[160:163]\n\t" \
  "ds_write_b64 v9, v[152:153] offset:4096\n\t" \
  "s_waitcnt lgkmcnt(5)\n\t" \
  "v_mfma_f32_16x16x32_bf16 v[160:163], v[96:99], v[24:27], v[160:163]\n\t" \
  "ds_read_b128 v[12:15], v5 offset:4096\n\t" \
  "v_cvt_pk_bf16_f32 v156, v156, v157\n\t" \
  "v_cvt_pk_bf16_f32 v157, v158, v159\n\t" \
  "v_mfma_f32_16x16x32_bf16 v[164:167], v[84:87], v[20:23], v[164:167]\n\t" \
  "v_pk_max_i16 v156, v156, v6\n\t" \
  "v_pk_max_i16 v157, v157, v6\n\t" \
  "v_mfma_f32_16x16x32_bf16 v[164:167], v[100:103], v[24:27], v[164:167]\n\t" \
  "ds_write_b64 v10, v[156:157] offset:4096\n\t" \
  "v_mfma_f32_16x16x32_bf16 v[168:171], v[88:91], v[20:23], v[168:171]\n\t" \
  "ds_read_b128 v[144:147], %[er] offset:4096\n\t" \
  "ds_read_b128 v[148:151], %[er] offset:5120\n\t" \
  "v_mfma_f32_16x16x32_bf16 v[168:171], v[104:107], v[24:27], v[168:171]\n\t" \
  "v_mfma_f32_16x16x32_bf16 v[172:175], v[92:95], v[20:23], v[172:175]\n\t" \
  "ds_read_b128 v[16:19], v5 offset:14336\n\t" \
  "v_mfma_f32_16x16x32_bf16 v[172:175], v[108:111], v[24:27], v[172:175]\n\t" \
  "s_waitcnt lgkmcnt(9)\n\t" \
  "v_mfma_f32_16x16x32_bf16 v[176:179], v[48:51], v[28:31], v[176:179]\n\t" \
  "v_cvt_pk_bf16_f32 v160, v160, v161\n\t" \
  "v_cvt_pk_bf16_f32 v161, v162, v163\n\t" \
  "s_waitcnt lgkmcnt(6)\n\t" \
  "v_mfma_f32_16x16x32_bf16 v[176:179], v[64:67], v[32:35], v[176:179]\n\t" \
  "ds_read_b128 v[20:23], v11 offset:5120\n\t" \
  "v_pk_max_i16 v160, v160, v6\n\t" \
  "v_pk_max_i16 v161, v161, v6\n\t" \
  "v_mfma_f32_16x16x32_bf16 v[180:183], v[52:55], v[28:31], v[180:183]\n\t" \
  "ds_write_b64 v7, v[160:161] offset:6144\n\t" \
  "v_mfma_f32_16x16x32_bf16 v[180:183], v[68:71], v[32:35], v[180:183]\n\t" \
  "v_cvt_pk_bf16_f32 v164, v164, v165\n\t" \
  "v_cvt_pk_bf16_f32 v165, v166, v167\n\t" \
  "s_waitcnt lgkmcnt(4)\n\t" \
  "buffer_store_dwordx4 v[144:147], %[vo], %[out], %[so] offen" OUT_POL "\n\t" \
  "s_add_u32 %[so], %[so], 32768\n\t" \
  "s_and_b32 %[so], %[so], 0xffffff\n\t" \
  "v_mfma_f32_16x16x32_bf16 v[184:187], v[56:59], v[28:31], v[184:187]\n\t" \
  "v_pk_max_i16 v164, v164, v6\n\t" \
  "v_pk_max_i16 v165, v165, v6\n\t" \
  "v_mfma_f32_16x16x32_bf16 v[184:187], v[72:75], v[32:35], v[184:187]\n\t" \
  "ds_write_b64 v8, v[164:165] offset:6144\n\t" \
  "s_waitcnt lgkmcnt(4)\n\t" \
  "buffer_store_dwordx4 v[148:151], %[vo], %[out], %[so] offen" OUT_POL "\n\t" \
  "s_add_u32 %[so], %[so], 32768\n\t" \
  "s_and_b32 %[so], %[so], 0xffffff\n\t" \
  "v_mfma_f32_16x16x32_bf16 v[188:191], v[60:63], v[28:31], v[188:191]\n\t" \
  "ds_read_b128 v[24:27], v11 offset:15360\n\t" \
  "v_cvt_pk_bf16_f32 v168, v168, v169\n\t" \
  "v_cvt_pk_bf16_f32 v169, v170, v171\n\t" \
  "v_mfma_f32_16x16x32_bf16 v[188:191], v[76:79], v[32:35], v[188:191]\n\t" \
  "v_pk_max_i16 v168, v168, v6\n\t" \
  "v_pk_max_i16 v169, v169, v6\n\t" \
  "v_mfma_f32_16x16x32_bf16 v[176:179], v[80:83], v[12:15], v[176:179]\n\t" \
  "ds_write_b64 v9, v[168:169] offset:6144\n\t" \
  "s_waitcnt lgkmcnt(5)\n\t" \
  "v_mfma_f32_16x16x32_bf16 v[176:179], v[96:99], v[16:19], v[176:179]\n\t" \
  "ds_read_b128 v[28:31], v5 offset:5120\n\t" \
  "v_cvt_pk_bf16_f32 v172, v172, v173\n\t" \
  "v_cvt_pk_bf16_f32 v173, v174, v175\n\t" \
  "v_mfma_f32_16x16x32_bf16 v[180:183], v[84:87], v[12:15], v[180:183]\n\t" \
  "v_pk_max_i16 v172, v172, v6\n\t" \
  "v_pk_max_i16 v173, v173, v6\n\t" \
  "v_mfma_f32_16x16x32_bf16 v[180:183], v[100:103], v[16:19], v[180:183]\n\t" \
  "ds_write_b64 v10, v[172:173] offset:6144\n\t" \
  "v_mfma_f32_16x16x32_bf16 v[184:187], v[88:91], v[12:15], v[184:187]\n\t" \
  "ds_read_b128 v[160:163], %[er] offset:6144\n\t" \
  "ds_read_b128 v[164:167], %[er] offset:7168\n\t" \
  "v_mfma_f32_16x16x32_bf16 v[184:187], v[104:107], v[16:19], v[184:187]\n\t" \
  "v_mfma_f32_16x16x32_bf16 v[188:191], v[92:95], v[12:15], v[188:191]\n\t" \
  "ds_read_b128 v[32:35], v5 offset:15360\n\t" \
  "v_mfma_f32_16x16x32_bf16 v[188:191], v[108:111], v[16:19], v[188:191]\n\t" \
  "s_waitcnt lgkmcnt(9)\n\t" \
  "v_mfma_f32_16x16x32_bf16 v[192:195], v[48:51], v[20:23], v[192:195]\n\t" \
  "v_cvt_pk_bf16_f32 v176, v176, v177\n\t" \
  "v_cvt_pk_bf16_f32 v177, v178, v179\n\t" \
  "s_waitcnt lgkmcnt(6)\n\t" \
  "v_mfma_f32_16x16x32_bf16 v[192:195], v[64:67], v[24:27], v[192:195]\n\t" \
  "ds_read_b128 v[12:15], v11 offset:6144\n\t" \
  "v_pk_max_i16 v176, v176, v6\n\t" \
  "v_pk_max_i16 v177, v177, v6\n\t" \
  "v_mfma_f32_16x16x32_bf16 v[196:199], v[52:55], v[20:23], v[196:199]\n\t" \
  "ds_write_b64 v7, v[176:177] offset:8192\n\t" \
  "v_mfma_f32_16x16x32_bf16 v[196:199], v[68:71], v[24:27], v[196:199]\n\t" \
  "v_cvt_pk_bf16_f32 v180, v180, v181\n\t" \
  "v_cvt_pk_bf16_f32 v181, v182, v183\n\t" \
  "s_waitcnt lgkmcnt(4)\n\t" \
  "buffer_store_dwordx4 v[160:163], %[vo], %[out], %[so] offen" OUT_POL "\n\t" \
  "s_add_u32 %[so], %[so], 32768\n\t" \
  "s_and_b32 %[so], %[so], 0xffffff\n\t" \
  "v_mfma_f32_16x16x32_bf16 v[200:203], v[56:59], v[20:23], v[200:203]\n\t" \
  "v_pk_max_i16 v180, v180, v6\n\t" \
  "v_pk_max_i16 v181, v181, v6\n\t" \
  "v_mfma_f32_16x16x32_bf16 v[200:203], v[72:75], v[24:27], v[200:203]\n\t" \
  "ds_write_b64 v8, v[180:181] offset:8192\n\t" \
  "s_waitcnt lgkmcnt(4)\n\t" \
  "buffer_store_dwordx4 v[164:167], %[vo], %[out], %[so] offen" OUT_POL "\n\t" \
  "s_add_u32 %[so], %[so], 32768\n\t" \
  "s_and_b32 %[so], %[so], 0xffffff\n\t" \
  "v_mfma_f32_16x16x32_bf16 v[204:207], v[60:63], v[20:23], v[204:207]\n\t" \
  "ds_read_b128 v[16:19], v11 offset:16384\n\t" \
  "v_cvt_pk_bf16_f32 v184, v184, v185\n\t" \
  "v_cvt_pk_bf16_f32 v185, v186, v187\n\t" \
  "v_mfma_f32_16x16x32_bf16 v[204:207], v[76:79], v[24:27], v[204:207]\n\t" \
  "v_pk_max_i16 v184, v184, v6\n\t" \
  "v_pk_max_i16 v185, v185, v6\n\t" \
  "v_mfma_f32_16x16x32_bf16 v[192:195], v[80:83], v[28:31], v[192:195]\n\t" \
  "ds_write_b64 v9, v[184:185] offset:8192\n\t" \
  "s_waitcnt lgkmcnt(5)\n\t" \
  "v_mfma_f32_16x16x32_bf16 v[192:195], v[96:99], v[32:35], v[192:195]\n\t" \
  "ds_read_b128 v[20:23], v5 offset:6144\n\t" \
  "v_cvt_pk_bf16_f32 v188, v188, v189\n\t" \
  "v_cvt_pk_bf16_f32 v189, v190, v191\n\t" \
  "v_mfma_f32_16x16x32_bf16 v[196:199], v[84:87], v[28:31], v[196:199]\n\t" \
  "v_pk_max_i16 v188, v188, v6\n\t" \
  "v_pk_max_i16 v189, v189, v6\n\t" \
  "v_mfma_f32_16x16x32_bf16 v[196:199], v[100:103], v[32:35], v[196:199]\n\t" \
  "ds_write_b64 v10, v[188:189] offset:8192\n\t" \
  "v_mfma_f32_16x16x32_bf16 v[200:203], v[88:91], v[28:31], v[200:203]\n\t" \
  "ds_read_b128 v[176:179], %[er] offset:8192\n\t" \
  "ds_read_b128 v[180:183], %[er] offset:9216\n\t" \
  "v_mfma_f32_16x16x32_bf16 v[200:203], v[104:107], v[32:35], v[200:203]\n\t" \
  "v_mfma_f32_16x16x32_bf16 v[204:207], v[92:95], v[28:31], v[204:207]\n\t" \
  "ds_read_b128 v[24:27], v5 offset:16384\n\t" \
  "v_mfma_f32_16x16x32_bf16 v[204:207], v[108:111], v[32:35], v[204:207]\n\t" \
  "s_waitcnt lgkmcnt(9)\n\t" \
  "v_mfma_f32_16x16x32_bf16 v[208:211], v[48:51], v[12:15], v[208:211]\n\t" \
  "v_cvt_pk_bf16_f32 v192, v192, v193\n\t" \
  "v_cvt_pk_bf16_f32 v193, v194, v195\n\t" \
  "s_waitcnt lgkmcnt(6)\n\t" \
  "v_mfma_f32_16x16x32_bf16 v[208:211], v[64:67], v[16:19], v[208:211]\n\t" \
  "ds_read_b128 v[28:31], v11 offset:7168\n\t" \
  "v_pk_max_i16 v192, v192, v6\n\t" \
  "v_pk_max_i16 v193, v193, v6\n\t" \
  "v_mfma_f32_16x16x32_bf16 v[212:215], v[52:55], v[12:15], v[212:215]\n\t" \
  "ds_write_b64 v7, v[192:193] offset:10240\n\t" \
  "v_mfma_f32_16x16x32_bf16 v[212:215], v[68:71], v[16:19], v[212:215]\n\t" \
  "v_cvt_pk_bf16_f32 v196, v196, v197\n\t" \
  "v_cvt_pk_bf16_f32 v197, v198, v199\n\t" \
  "s_waitcnt lgkmcnt(4)\n\t" \
  "buffer_store_dwordx4 v[176:179], %[vo], %[out], %[so] offen" OUT_POL "\n\t" \
  "s_add_u32 %[so], %[so], 32768\n\t" \
  "s_and_b32 %[so], %[so], 0xffffff\n\t" \
  "v_mfma_f32_16x16x32_bf16 v[216:219], v[56:59], v[12:15], v[216:219]\n\t" \
  "v_pk_max_i16 v196, v196, v6\n\t" \
  "v_pk_max_i16 v197, v197, v6\n\t" \
  "v_mfma_f32_16x16x32_bf16 v[216:219], v[72:75], v[16:19], v[216:219]\n\t" \
  "ds_write_b64 v8, v[196:197] offset:10240\n\t" \
  "s_waitcnt lgkmcnt(4)\n\t" \
  "buffer_store_dwordx4 v[180:183], %[vo], %[out], %[so] offen" OUT_POL "\n\t" \
  "s_add_u32 %[so], %[so], 32768\n\t" \
  "s_and_b32 %[so], %[so], 0xffffff\n\t" \
  "v_mfma_f32_16x16x32_bf16 v[220:223], v[60:63], v[12:15], v[220:223]\n\t" \
  "ds_read_b128 v[32:35], v11 offset:17408\n\t" \
  "v_cvt_pk_bf16_f32 v200, v200, v201\n\t" \
  "v_cvt_pk_bf16_f32 v201, v202, v203\n\t" \
  "v_mfma_f32_16x16x32_bf16 v[220:223], v[76:79], v[16:19], v[220:223]\n\t" \
  "v_pk_max_i16 v200, v200, v6\n\t" \
  "v_pk_max_i16 v201, v201, v6\n\t" \
  "v_mfma_f32_16x16x32_bf16 v[208:211], v[80:83], v[20:23], v[208:211]\n\t" \
  "ds_write_b64 v9, v[200:201] offset:10240\n\t" \
  "s_waitcnt lgkmcnt(5)\n\t" \
  "v_mfma_f32_16x16x32_bf16 v[208:211], v[96:99], v[24:27], v[208:211]\n\t" \
  "ds_read_b128 v[12:15], v5 offset:7168\n\t" \
  "v_cvt_pk_bf16_f32 v204, v204, v205\n\t" \
  "v_cvt_pk_bf16_f32 v205, v206, v207\n\t" \
  "v_mfma_f32_16x16x32_bf16 v[212:215], v[84:87], v[20:23], v[212:215]\n\t" \
  "v_pk_max_i16 v204, v204, v6\n\t" \
  "v_pk_max_i16 v205, v205, v6\n\t" \
  "v_mfma_f32_16x16x32_bf16 v[212:215], v[100:103], v[24:27], v[212:215]\n\t" \
  "ds_write_b64 v10, v[204:205] offset:10240\n\t" \
  "v_mfma_f32_16x16x32_bf16 v[216:219], v[88:91], v[20:23], v[216:219]\n\t" \
  "ds_read_b128 v[192:195], %[er] offset:10240\n\t" \
  "ds_read_b128 v[196:199], %[er] offset:11264\n\t" \
  "v_mfma_f32_16x16x32_bf16 v[216:219], v[104:107], v[24:27], v[216:219]\n\t" \
  "v_mfma_f32_16x16x32_bf16 v[220:223], v[92:95], v[20:23], v[220:223]\n\t" \
  "ds_read_b128 v[16:19], v5 offset:17408\n\t" \
  "v_mfma_f32_16x16x32_bf16 v[220:223], v[108:111], v[24:27], v[220:223]\n\t" \
  "s_waitcnt lgkmcnt(9)\n\t" \
  "v_mfma_f32_16x16x32_bf16 v[224:227], v[48:51], v[28:31], v[224:227]\n\t" \
  "v_cvt_pk_bf16_f32 v208, v208, v209\n\t" \
  "v_cvt_pk_bf16_f32 v209, v210, v211\n\t" \
  "s_waitcnt lgkmcnt(6)\n\t" \
  "v_mfma_f32_16x16x32_bf16 v[224:227], v[64:67], v[32:35], v[224:227]\n\t" \
  "ds_read_b128 v[20:23], v11 offset:8192\n\t" \
  "v_pk_max_i16 v208, v208, v6\n\t" \
  "v_pk_max_i16 v209, v209, v6\n\t" \
  "v_mfma_f32_16x16x32_bf16 v[228:231], v[52:55], v[28:31], v[228:231]\n\t" \
  "ds_write_b64 v7, v[208:209] offset:12288\n\t" \
  "v_mfma_f32_16x16x32_bf16 v[228:231], v[68:71], v[32:35], v[228:231]\n\t" \
  "v_cvt_pk_bf16_f32 v212, v212, v213\n\t" \
  "v_cvt_pk_bf16_f32 v213, v214, v215\n\t" \
  "s_waitcnt lgkmcnt(4)\n\t" \
  "buffer_store_dwordx4 v[192:195], %[vo], %[out], %[so] offen" OUT_POL "\n\t" \
  "s_add_u32 %[so], %[so], 32768\n\t" \
  "s_and_b32 %[so], %[so], 0xffffff\n\t" \
  "v_mfma_f32_16x16x32_bf16 v[232:235], v[56:59], v[28:31], v[232:235]\n\t" \
  "v_pk_max_i16 v212, v212, v6\n\t" \
  "v_pk_max_i16 v213, v213, v6\n\t" \
  "v_mfma_f32_16x16x32_bf16 v[232:235], v[72:75], v[32:35], v[232:235]\n\t" \
  "ds_write_b64 v8, v[212:213] offset:12288\n\t" \
  "s_waitcnt lgkmcnt(4)\n\t" \
  "buffer_store_dwordx4 v[196:199], %[vo], %[out], %[so] offen" OUT_POL "\n\t" \
  "s_add_u32 %[so], %[so], 32768\n\t" \
  "s_and_b32 %[so], %[so], 0xffffff\n\t" \
  "v_mfma_f32_16x16x32_bf16 v[236:239], v[60:63], v[28:31], v[236:239]\n\t" \
  "ds_read_b128 v[24:27], v11 offset:18432\n\t" \
  "v_cvt_pk_bf16_f32 v216, v216, v217\n\t" \
  "v_cvt_pk_bf16_f32 v217, v218, v219\n\t" \
  "v_mfma_f32_16x16x32_bf16 v[236:239], v[76:79], v[32:35], v[236:239]\n\t" \
  "v_pk_max_i16 v216, v216, v6\n\t" \
  "v_pk_max_i16 v217, v217, v6\n\t" \
  "v_mfma_f32_16x16x32_bf16 v[224:227], v[80:83], v[12:15], v[224:227]\n\t" \
  "ds_write_b64 v9, v[216:217] offset:12288\n\t" \
  "s_waitcnt lgkmcnt(5)\n\t" \
  "v_mfma_f32_16x16x32_bf16 v[224:227], v[96:99], v[16:19], v[224:227]\n\t" \
  "ds_read_b128 v[28:31], v5 offset:8192\n\t" \
  "v_cvt_pk_bf16_f32 v220, v220, v221\n\t" \
  "v_cvt_pk_bf16_f32 v221, v222, v223\n\t" \
  "v_mfma_f32_16x16x32_bf16 v[228:231], v[84:87], v[12:15], v[228:231]\n\t" \
  "v_pk_max_i16 v220, v220, v6\n\t" \
  "v_pk_max_i16 v221, v221, v6\n\t" \
  "v_mfma_f32_16x16x32_bf16 v[228:231], v[100:103], v[16:19], v[228:231]\n\t" \
  "ds_write_b64 v10, v[220:221] offset:12288\n\t" \
  "v_mfma_f32_16x16x32_bf16 v[232:235], v[88:91], v[12:15], v[232:235]\n\t" \
  "ds_read_b128 v[208:211], %[er] offset:12288\n\t" \
  "ds_read_b128 v[212:215], %[er] offset:13312\n\t" \
  "v_mfma_f32_16x16x32_bf16 v[232:235], v[104:107], v[16:19], v[232:235]\n\t" \
  "v_mfma_f32_16x16x32_bf16 v[236:239], v[92:95], v[12:15], v[236:239]\n\t" \
  "ds_read_b128 v[32:35], v5 offset:18432\n\t" \
  "v_mfma_f32_16x16x32_bf16 v[236:239], v[108:111], v[16:19], v[236:239]\n\t" \
  "s_waitcnt lgkmcnt(9)\n\t" \
  "v_mfma_f32_16x16x32_bf16 v[240:243], v[48:51], v[20:23], v[240:243]\n\t" \
  "v_cvt_pk_bf16_f32 v224, v224, v225\n\t" \
  "v_cvt_pk_bf16_f32 v225, v226, v227\n\t" \
  "s_waitcnt lgkmcnt(6)\n\t" \
  "v_mfma_f32_16x16x32_bf16 v[240:243], v[64:67], v[24:27], v[240:243]\n\t" \
  "v_pk_max_i16 v224, v224, v6\n\t" \
  "v_pk_max_i16 v225, v225, v6\n\t" \
  "v_mfma_f32_16x16x32_bf16 v[244:247], v[52:55], v[20:23], v[244:247]\n\t" \
  "ds_write_b64 v7, v[224:225] offset:14336\n\t" \
  "v_mfma_f32_16x16x32_bf16 v[244:247], v[68:71], v[24:27], v[244:247]\n\t" \
  "v_cvt_pk_bf16_f32 v228, v228, v229\n\t" \
  "v_cvt_pk_bf16_f32 v229, v230, v231\n\t" \
  "s_waitcnt lgkmcnt(3)\n\t" \
  "buffer_store_dwordx4 v[208:211], %[vo], %[out], %[so] offen" OUT_POL "\n\t" \
  "s_add_u32 %[so], %[so], 32768\n\t" \
  "s_and_b32 %[so], %[so], 0xffffff\n\t" \
  "v_mfma_f32_16x16x32_bf16 v[248:251], v[56:59], v[20:23], v[248:251]\n\t" \
  "v_pk_max_i16 v228, v228, v6\n\t" \
  "v_pk_max_i16 v229, v229, v6\n\t" \
  "v_mfma_f32_16x16x32_bf16 v[248:251], v[72:75], v[24:27], v[248:251]\n\t" \
  "ds_write_b64 v8, v[228:229] offset:14336\n\t" \
  "s_waitcnt lgkmcnt(3)\n\t" \
  "buffer_store_dwordx4 v[212:215], %[vo], %[out], %[so] offen" OUT_POL "\n\t" \
  "s_add_u32 %[so], %[so], 32768\n\t" \
  "s_and_b32 %[so], %[so], 0xffffff\n\t" \
  "v_mfma_f32_16x16x32_bf16 v[252:255], v[60:63], v[20:23], v[252:255]\n\t" \
  "v_cvt_pk_bf16_f32 v232, v232, v233\n\t" \
  "v_cvt_pk_bf16_f32 v233, v234, v235\n\t" \
  "v_mfma_f32_16x16x32_bf16 v[252:255], v[76:79], v[24:27], v[252:255]\n\t" \
  "v_pk_max_i16 v232, v232, v6\n\t" \
  "v_pk_max_i16 v233, v233, v6\n\t" \
  "v_mfma_f32_16x16x32_bf16 v[240:243], v[80:83], v[28:31], v[240:243]\n\t" \
  "ds_write_b64 v9, v[232:233] offset:14336\n\t" \
  "s_waitcnt lgkmcnt(3)\n\t" \
  "v_mfma_f32_16x16x32_bf16 v[240:243], v[96:99], v[32:35], v[240:243]\n\t" \
  "v_cvt_pk_bf16_f32 v236, v236, v237\n\t" \
  "v_cvt_pk_bf16_f32 v237, v238, v239\n\t" \
  "v_mfma_f32_16x16x32_bf16 v[244:247], v[84:87], v[28:31], v[244:247]\n\t" \
  "v_pk_max_i16 v236, v236, v6\n\t" \
  "v_pk_max_i16 v237, v237, v6\n\t" \
  "v_mfma_f32_16x16x32_bf16 v[244:247], v[100:103], v[32:35], v[244:247]\n\t" \
  "ds_write_b64 v10, v[236:237] offset:14336\n\t" \
  "v_mfma_f32_16x16x32_bf16 v[248:251], v[88:91], v[28:31], v[248:251]\n\t" \
  "ds_read_b128 v[224:227], %[er] offset:14336\n\t" \
  "ds_read_b128 v[228:231], %[er] offset:15360\n\t" \
  "v_mfma_f32_16x16x32_bf16 v[248:251], v[104:107], v[32:35], v[248:251]\n\t" \
  "v_mfma_f32_16x16x32_bf16 v[252:255], v[92:95], v[28:31], v[252:255]\n\t" \
  "v_mfma_f32_16x16x32_bf16 v[252:255], v[108:111], v[32:35], v[252:255]\n\t" \
  "s_nop 3\n\t" \
  "v_cvt_pk_bf16_f32 v240, v240, v241\n\t" \
  "v_cvt_pk_bf16_f32 v241, v242, v243\n\t" \
  "v_pk_max_i16 v240, v240, v6\n\t" \
  "v_pk_max_i16 v241, v241, v6\n\t" \
  "ds_write_b64 v7, v[240:241] offset:16384\n\t" \
  "v_cvt_pk_bf16_f32 v244, v244, v245\n\t" \
  "v_cvt_pk_bf16_f32 v245, v246, v247\n\t" \
  "v_pk_max_i16 v244, v244, v6\n\t" \
  "v_pk_max_i16 v245, v245, v6\n\t" \
  "ds_write_b64 v8, v[244:245] offset:16384\n\t" \
  "s_waitcnt lgkmcnt(3)\n\t" \
  "buffer_store_dwordx4 v[224:227], %[vo], %[out], %[so] offen" OUT_POL "\n\t" \
  "s_add_u32 %[so], %[so], 32768\n\t" \
  "s_and_b32 %[so], %[so], 0xffffff\n\t" \
  "s_waitcnt lgkmcnt(2)\n\t" \
  "buffer_store_dwordx4 v[228:231], %[vo], %[out], %[so] offen" OUT_POL "\n\t" \
  "s_add_u32 %[so], %[so], 32768\n\t" \
  "s_and_b32 %[so], %[so], 0xffffff\n\t" \
  "v_cvt_pk_bf16_f32 v248, v248, v249\n\t" \
  "v_cvt_pk_bf16_f32 v249, v250, v251\n\t" \
  "v_pk_max_i16 v248, v248, v6\n\t" \
  "v_pk_max_i16 v249, v249, v6\n\t" \
  "ds_write_b64 v9, v[248:249] offset:16384\n\t" \
  "s_nop 3\n\t" \
  "v_cvt_pk_bf16_f32 v252, v252, v253\n\t" \
  "v_cvt_pk_bf16_f32 v253, v254, v255\n\t" \
  "v_pk_max_i16 v252, v252, v6\n\t" \
  "v_pk_max_i16 v253, v253, v6\n\t" \
  "ds_write_b64 v10, v[252:253] offset:16384\n\t" \
  "ds_read_b128 v[240:243], %[er] offset:16384\n\t" \
  "ds_read_b128 v[244:247], %[er] offset:17408\n\t" \
  "s_waitcnt lgkmcnt(1)\n\t" \
  "buffer_store_dwordx4 v[240:243], %[vo], %[out], %[so] offen" OUT_POL "\n\t" \
  "s_add_u32 %[so], %[so], 32768\n\t" \
  "s_and_b32 %[so], %[so], 0xffffff\n\t" \
  "s_waitcnt lgkmcnt(0)\n\t" \
  "buffer_store_dwordx4 v[244:247], %[vo], %[out], %[so] offen" OUT_POL "\n\t" \
  "s_add_u32 %[so], %[so], 32768\n\t" \
  "s_and_b32 %[so], %[so], 0xffffff\n\t" \
  "s_setprio 0\n\t" \
  ""
#define CONS_ASM_10 \
  "s_setprio 3\n\t" \
  "v_mov_b32 v96, 0\n\t" \
  "v_mov_b32 v97, 0\n\t" \
  "v_mov_b32 v98, 0\n\t" \
  "v_mov_b32 v99, 0\n\t" \
  "v_mov_b32 v100, 0\n\t" \
  "v_mov_b32 v101, 0\n\t" \
  "v_mov_b32 v102, 0\n\t" \
  "v_mov_b32 v103, 0\n\t" \
  "v_mov_b32 v104, 0\n\t" \
  "v_mov_b32 v105, 0\n\t" \
  "v_mov_b32 v106, 0\n\t" \
  "v_mov_b32 v107, 0\n\t" \
  "v_mov_b32 v108, 0\n\t" \
  "v_mov_b32 v109, 0\n\t" \
  "v_mov_b32 v110, 0\n\t" \
  "v_mov_b32 v111, 0\n\t" \
  "v_mov_b32 v112, 0\n\t" \
  "v_mov_b32 v113, 0\n\t" \
  "v_mov_b32 v114, 0\n\t" \
  "v_mov_b32 v115, 0\n\t" \
  "v_mov_b32 v116, 0\n\t" \
  "v_mov_b32 v117, 0\n\t" \
  "v_mov_b32 v118, 0\n\t" \
  "v_mov_b32 v119, 0\n\t" \
  "v_mov_b32 v120, 0\n\t" \
  "v_mov_b32 v121, 0\n\t" \
  "v_mov_b32 v122, 0\n\t" \
  "v_mov_b32 v123, 0\n\t" \
  "v_mov_b32 v124, 0\n\t" \
  "v_mov_b32 v125, 0\n\t" \
  "v_mov_b32 v126, 0\n\t" \
  "v_mov_b32 v127, 0\n\t" \
  "v_mov_b32 v128, 0\n\t" \
  "v_mov_b32 v129, 0\n\t" \
  "v_mov_b32 v130, 0\n\t" \
  "v_mov_b32 v131, 0\n\t" \
  "v_mov_b32 v132, 0\n\t" \
  "v_mov_b32 v133, 0\n\t" \
  "v_mov_b32 v134, 0\n\t" \
  "v_mov_b32 v135, 0\n\t" \
  "v_mov_b32 v136, 0\n\t" \
  "v_mov_b32 v137, 0\n\t" \
  "v_mov_b32 v138, 0\n\t" \
  "v_mov_b32 v139, 0\n\t" \
  "v_mov_b32 v140, 0\n\t" \
  "v_mov_b32 v141, 0\n\t" \
  "v_mov_b32 v142, 0\n\t" \
  "v_mov_b32 v143, 0\n\t" \
  "v_mov_b32 v144, 0\n\t" \
  "v_mov_b32 v145, 0\n\t" \
  "v_mov_b32 v146, 0\n\t" \
  "v_mov_b32 v147, 0\n\t" \
  "v_mov_b32 v148, 0\n\t" \
  "v_mov_b32 v149, 0\n\t" \
  "v_mov_b32 v150, 0\n\t" \
  "v_mov_b32 v151, 0\n\t" \
  "v_mov_b32 v152, 0\n\t" \
  "v_mov_b32 v153, 0\n\t" \
  "v_mov_b32 v154, 0\n\t" \
  "v_mov_b32 v155, 0\n\t" \
  "v_mov_b32 v156, 0\n\t" \
  "v_mov_b32 v157, 0\n\t" \
  "v_mov_b32 v158, 0\n\t" \
  "v_mov_b32 v159, 0\n\t" \
  "v_mov_b32 v160, 0\n\t" \
  "v_mov_b32 v161, 0\n\t" \
  "v_mov_b32 v162, 0\n\t" \
  "v_mov_b32 v163, 0\n\t" \
  "v_mov_b32 v164, 0\n\t" \
  "v_mov_b32 v165, 0\n\t" \
  "v_mov_b32 v166, 0\n\t" \
  "v_mov_b32 v167, 0\n\t" \
  "v_mov_b32 v168, 0\n\t" \
  "v_mov_b32 v169, 0\n\t" \
  "v_mov_b32 v170, 0\n\t" \
  "v_mov_b32 v171, 0\n\t" \
  "v_mov_b32 v172, 0\n\t" \
  "v_mov_b32 v173, 0\n\t" \
  "v_mov_b32 v174, 0\n\t" \
  "v_mov_b32 v175, 0\n\t" \
  "v_mov_b32 v176, 0\n\t" \
  "v_mov_b32 v177, 0\n\t" \
  "v_mov_b32 v178, 0\n\t" \
  "v_mov_b32 v179, 0\n\t" \
  "v_mov_b32 v180, 0\n\t" \
  "v_mov_b32 v181, 0\n\t" \
  "v_mov_b32 v182, 0\n\t" \
  "v_mov_b32 v183, 0\n\t" \
  "v_mov_b32 v184, 0\n\t" \
  "v_mov_b32 v185, 0\n\t" \
  "v_mov_b32 v186, 0\n\t" \
  "v_mov_b32 v187, 0\n\t" \
  "v_mov_b32 v188, 0\n\t" \
  "v_mov_b32 v189, 0\n\t" \
  "v_mov_b32 v190, 0\n\t" \
  "v_mov_b32 v191, 0\n\t" \
  "v_mov_b32 v192, 0\n\t" \
  "v_mov_b32 v193, 0\n\t" \
  "v_mov_b32 v194, 0\n\t" \
  "v_mov_b32 v195, 0\n\t" \
  "v_mov_b32 v196, 0\n\t" \
  "v_mov_b32 v197, 0\n\t" \
  "v_mov_b32 v198, 0\n\t" \
  "v_mov_b32 v199, 0\n\t" \
  "v_mov_b32 v200, 0\n\t" \
  "v_mov_b32 v201, 0\n\t" \
  "v_mov_b32 v202, 0\n\t" \
  "v_mov_b32 v203, 0\n\t" \
  "v_mov_b32 v204, 0\n\t" \
  "v_mov_b32 v205, 0\n\t" \
  "v_mov_b32 v206, 0\n\t" \
  "v_mov_b32 v207, 0\n\t" \
  "v_mov_b32 v208, 0\n\t" \
  "v_mov_b32 v209, 0\n\t" \
  "v_mov_b32 v210, 0\n\t" \
  "v_mov_b32 v211, 0\n\t" \
  "v_mov_b32 v212, 0\n\t" \
  "v_mov_b32 v213, 0\n\t" \
  "v_mov_b32 v214, 0\n\t" \
  "v_mov_b32 v215, 0\n\t" \
  "v_mov_b32 v216, 0\n\t" \
  "v_mov_b32 v217, 0\n\t" \
  "v_mov_b32 v218, 0\n\t" \
  "v_mov_b32 v219, 0\n\t" \
  "v_mov_b32 v220, 0\n\t" \
  "v_mov_b32 v221, 0\n\t" \
  "v_mov_b32 v222, 0\n\t" \
  "v_mov_b32 v223, 0\n\t" \
  "v_mov_b32 v224, 0\n\t" \
  "v_mov_b32 v225, 0\n\t" \
  "v_mov_b32 v226, 0\n\t" \
  "v_mov_b32 v227, 0\n\t" \
  "v_mov_b32 v228, 0\n\t" \
  "v_mov_b32 v229, 0\n\t" \
  "v_mov_b32 v230, 0\n\t" \
  "v_mov_b32 v231, 0\n\t" \
  "v_mov_b32 v232, 0\n\t" \
  "v_mov_b32 v233, 0\n\t" \
  "v_mov_b32 v234, 0\n\t" \
  "v_mov_b32 v235, 0\n\t" \
  "v_mov_b32 v236, 0\n\t" \
  "v_mov_b32 v237, 0\n\t" \
  "v_mov_b32 v238, 0\n\t" \
  "v_mov_b32 v239, 0\n\t" \
  "v_mov_b32 v240, 0\n\t" \
  "v_mov_b32 v241, 0\n\t" \
  "v_mov_b32 v242, 0\n\t" \
  "v_mov_b32 v243, 0\n\t" \
  "v_mov_b32 v244, 0\n\t" \
  "v_mov_b32 v245, 0\n\t" \
  "v_mov_b32 v246, 0\n\t" \
  "v_mov_b32 v247, 0\n\t" \
  "v_mov_b32 v248, 0\n\t" \
  "v_mov_b32 v249, 0\n\t" \
  "v_mov_b32 v250, 0\n\t" \
  "v_mov_b32 v251, 0\n\t" \
  "v_mov_b32 v252, 0\n\t" \
  "v_mov_b32 v253, 0\n\t" \
  "v_mov_b32 v254, 0\n\t" \
  "v_mov_b32 v255, 0\n\t" \
  "v_xor_b32 v8, 32, v7\n\t" \
  "v_xor_b32 v9, 64, v7\n\t" \
  "v_xor_b32 v10, 96, v7\n\t" \
  "s_mov_b32 %[st], 0\n\t" \
  "s_mov_b32 %[dp], 53248\n\t" \
  "s_mov_b32 %[dn], 4294860800\n\t" \
  "s_mov_b32 %[cnt], 30\n\t" \
  "v_mov_b32 v5, v11\n\t" \
  "s_barrier\n\t" \
  "ds_read_b64_tr_b16 v[32:33], v7 offset:0\n\t" \
  "ds_read_b64_tr_b16 v[34:35], v7 offset:2048\n\t" \
  "ds_read_b64_tr_b16 v[48:49], v7 offset:16384\n\t" \
  "ds_read_b64_tr_b16 v[50:51], v7 offset:18432\n\t" \
  "ds_read_b64_tr_b16 v[36:37], v8 offset:0\n\t" \
  "ds_read_b64_tr_b16 v[38:39], v8 offset:2048\n\t" \
  "ds_read_b64_tr_b16 v[52:53], v8 offset:16384\n\t" \
  "ds_read_b64_tr_b16 v[54:55], v8 offset:18432\n\t" \
  "ds_read_b64_tr_b16 v[40:41], v9 offset:0\n\t" \
  "ds_read_b64_tr_b16 v[42:43], v9 offset:2048\n\t" \
  "ds_read_b64_tr_b16 v[56:57], v9 offset:16384\n\t" \
  "ds_read_b64_tr_b16 v[58:59], v9 offset:18432\n\t" \
  "ds_read_b64_tr_b16 v[44:45], v10 offset:0\n\t" \
  "ds_read_b64_tr_b16 v[46:47], v10 offset:2048\n\t" \
  "ds_read_b64_tr_b16 v[60:61], v10 offset:16384\n\t" \
  "ds_read_b64_tr_b16 v[62:63], v10 offset:18432\n\t" \
  "v_add_u32 v7, %[dn], v7\n\t" \
  "v_add_u32 v8, %[dn], v8\n\t" \
  "v_add_u32 v9, %[dn], v9\n\t" \
  "v_add_u32 v10, %[dn], v10\n\t" \
  "s_waitcnt lgkmcnt(0)\n\t" \
  "s_barrier\n\t" \
  "ds_read_b128 v[12:15], v11 offset:0\n\t" \
  "ds_read_b128 v[16:19], v11 offset:10240\n\t" \
  "ds_read_b128 v[20:23], v11 offset:1024\n\t" \
  "ds_read_b128 v[24:27], v11 offset:11264\n\t" \
  ".p2align 6\n\t" \
  "L_cons_loop10_%=:\n\t" \
  "s_waitcnt lgkmcnt(3)\n\t" \
  "v_mfma_f32_16x16x32_bf16 v[96:99], v[32:35], v[12:15], v[96:99]\n\t" \
  "s_waitcnt lgkmcnt(2)\n\t" \
  "v_mfma_f32_16x16x32_bf16 v[96:99], v[48:51], v[16:19], v[96:99]\n\t" \
  "ds_read_b128 v[28:31], v11 offset:2048\n\t" \
  "v_mfma_f32_16x16x32_bf16 v[100:103], v[36:39], v[12:15], v[100:103]\n\t" \
  "ds_read_b64_tr_b16 v[64:65], v7 offset:0\n\t" \
  "v_mfma_f32_16x16x32_bf16 v[100:103], v[52:55], v[16:19], v[100:103]\n\t" \
  "s_add_u32 %[st], %[st], 1\n\t" \
  "v_mfma_f32_16x16x32_bf16 v[104:107], v[40:43], v[12:15], v[104:107]\n\t" \
  "ds_read_b64_tr_b16 v[66:67], v7 offset:2048\n\t" \
  "v_mfma_f32_16x16x32_bf16 v[104:107], v[56:59], v[16:19], v[104:107]\n\t" \
  "s_cmp_eq_u32 %[st], 3\n\t" \
  "v_mfma_f32_16x16x32_bf16 v[108:111], v[44:47], v[12:15], v[108:111]\n\t" \
  "ds_read_b128 v[12:15], v11 offset:12288\n\t" \
  "v_mfma_f32_16x16x32_bf16 v[108:111], v[60:63], v[16:19], v[108:111]\n\t" \
  "s_waitcnt lgkmcnt(5)\n\t" \
  "v_mfma_f32_16x16x32_bf16 v[112:115], v[32:35], v[20:23], v[112:115]\n\t" \
  "s_waitcnt lgkmcnt(4)\n\t" \
  "v_mfma_f32_16x16x32_bf16 v[112:115], v[48:51], v[24:27], v[112:115]\n\t" \
  "ds_read_b128 v[16:19], v11 offset:3072\n\t" \
  "v_mfma_f32_16x16x32_bf16 v[116:119], v[36:39], v[20:23], v[116:119]\n\t" \
  "ds_read_b64_tr_b16 v[80:81], v7 offset:16384\n\t" \
  "v_mfma_f32_16x16x32_bf16 v[116:119], v[52:55], v[24:27], v[116:119]\n\t" \
  "s_cselect_b32 %[dl], %[dn], %[dp]\n\t" \
  "v_mfma_f32_16x16x32_bf16 v[120:123], v[40:43], v[20:23], v[120:123]\n\t" \
  "ds_read_b64_tr_b16 v[82:83], v7 offset:18432\n\t" \
  "v_mfma_f32_16x16x32_bf16 v[120:123], v[56:59], v[24:27], v[120:123]\n\t" \
  "s_cselect_b32 %[st], 0, %[st]\n\t" \
  "v_mfma_f32_16x16x32_bf16 v[124:127], v[44:47], v[20:23], v[124:127]\n\t" \
  "ds_read_b128 v[20:23], v11 offset:13312\n\t" \
  "v_mfma_f32_16x16x32_bf16 v[124:127], v[60:63], v[24:27], v[124:127]\n\t" \
  "s_waitcnt lgkmcnt(7)\n\t" \
  "v_mfma_f32_16x16x32_bf16 v[128:131], v[32:35], v[28:31], v[128:131]\n\t" \
  "s_waitcnt lgkmcnt(4)\n\t" \
  "v_mfma_f32_16x16x32_bf16 v[128:131], v[48:51], v[12:15], v[128:131]\n\t" \
  "ds_read_b128 v[24:27], v11 offset:4096\n\t" \
  "v_mfma_f32_16x16x32_bf16 v[132:135], v[36:39], v[28:31], v[132:135]\n\t" \
  "ds_read_b64_tr_b16 v[68:69], v8 offset:0\n\t" \
  "v_mfma_f32_16x16x32_bf16 v[132:135], v[52:55], v[12:15], v[132:135]\n\t" \
  "v_mfma_f32_16x16x32_bf16 v[136:139], v[40:43], v[28:31], v[136:139]\n\t" \
  "ds_read_b64_tr_b16 v[70:71], v8 offset:2048\n\t" \
  "v_mfma_f32_16x16x32_bf16 v[136:139], v[56:59], v[12:15], v[136:139]\n\t" \
  "v_mfma_f32_16x16x32_bf16 v[140:143], v[44:47], v[28:31], v[140:143]\n\t" \
  "ds_read_b128 v[28:31], v11 offset:14336\n\t" \
  "v_mfma_f32_16x16x32_bf16 v[140:143], v[60:63], v[12:15], v[140:143]\n\t" \
  "s_waitcnt lgkmcnt(7)\n\t" \
  "v_mfma_f32_16x16x32_bf16 v[144:147], v[32:35], v[16:19], v[144:147]\n\t" \
  "s_waitcnt lgkmcnt(4)\n\t" \
  "v_mfma_f32_16x16x32_bf16 v[144:147], v[48:51], v[20:23], v[144:147]\n\t" \
  "ds_read_b128 v[12:15], v11 offset:5120\n\t" \
  "v_mfma_f32_16x16x32_bf16 v[148:151], v[36:39], v[16:19], v[148:151]\n\t" \
  "ds_read_b64_tr_b16 v[84:85], v8 offset:16384\n\t" \
  "v_mfma_f32_16x16x32_bf16 v[148:151], v[52:55], v[20:23], v[148:151]\n\t" \
  "v_mfma_f32_16x16x32_bf16 v[152:155], v[40:43], v[16:19], v[152:155]\n\t" \
  "ds_read_b64_tr_b16 v[86:87], v8 offset:18432\n\t" \
  "v_mfma_f32_16x16x32_bf16 v[152:155], v[56:59], v[20:23], v[152:155]\n\t" \
  "v_mfma_f32_16x16x32_bf16 v[156:159], v[44:47], v[16:19], v[156:159]\n\t" \
  "ds_read_b128 v[16:19], v11 offset:15360\n\t" \
  "v_mfma_f32_16x16x32_bf16 v[156:159], v[60:63], v[20:23], v[156:159]\n\t" \
  "s_waitcnt lgkmcnt(7)\n\t" \
  "v_mfma_f32_16x16x32_bf16 v[160:163], v[32:35], v[24:27], v[160:163]\n\t" \
  "s_waitcnt lgkmcnt(4)\n\t" \
  "v_mfma_f32_16x16x32_bf16 v[160:163], v[48:51], v[28:31], v[160:163]\n\t" \
  "ds_read_b128 v[20:23], v11 offset:6144\n\t" \
  "v_mfma_f32_16x16x32_bf16 v[164:167], v[36:39], v[24:27], v[164:167]\n\t" \
  "ds_read_b64_tr_b16 v[72:73], v9 offset:0\n\t" \
  "v_mfma_f32_16x16x32_bf16 v[164:167], v[52:55], v[28:31], v[164:167]\n\t" \
  "v_mfma_f32_16x16x32_bf16 v[168:171], v[40:43], v[24:27], v[168:171]\n\t" \
  "ds_read_b64_tr_b16 v[74:75], v9 offset:2048\n\t" \
  "v_mfma_f32_16x16x32_bf16 v[168:171], v[56:59], v[28:31], v[168:171]\n\t" \
  "v_mfma_f32_16x16x32_bf16 v[172:175], v[44:47], v[24:27], v[172:175]\n\t" \
  "ds_read_b128 v[24:27], v11 offset:16384\n\t" \
  "v_mfma_f32_16x16x32_bf16 v[172:175], v[60:63], v[28:31], v[172:175]\n\t" \
  "s_waitcnt lgkmcnt(7)\n\t" \
  "v_mfma_f32_16x16x32_bf16 v[176:179], v[32:35], v[12:15], v[176:179]\n\t" \
  "s_waitcnt lgkmcnt(4)\n\t" \
  "v_mfma_f32_16x16x32_bf16 v[176:179], v[48:51], v[16:19], v[176:179]\n\t" \
  "ds_read_b128 v[28:31], v11 offset:7168\n\t" \
  "v_mfma_f32_16x16x32_bf16 v[180:183], v[36:39], v[12:15], v[180:183]\n\t" \
  "ds_read_b64_tr_b16 v[88:89], v9 offset:16384\n\t" \
  "v_mfma_f32_16x16x32_bf16 v[180:183], v[52:55], v[16:19], v[180:183]\n\t" \
  "v_mfma_f32_16x16x32_bf16 v[184:187], v[40:43], v[12:15], v[184:187]\n\t" \
  "ds_read_b64_tr_b16 v[90:91], v9 offset:18432\n\t" \
  "v_mfma_f32_16x16x32_bf16 v[184:187], v[56:59], v[16:19], v[184:187]\n\t" \
  "v_mfma_f32_16x16x32_bf16 v[188:191], v[44:47], v[12:15], v[188:191]\n\t" \
  "ds_read_b128 v[12:15], v11 offset:17408\n\t" \
  "v_mfma_f32_16x16x32_bf16 v[188:191], v[60:63], v[16:19], v[188:191]\n\t" \
  "s_waitcnt lgkmcnt(7)\n\t" \
  "v_mfma_f32_16x16x32_bf16 v[192:195], v[32:35], v[20:23], v[192:195]\n\t" \
  "s_waitcnt lgkmcnt(4)\n\t" \
  "v_mfma_f32_16x16x32_bf16 v[192:195], v[48:51], v[24:27], v[192:195]\n\t" \
  "ds_read_b128 v[16:19], v11 offset:8192\n\t" \
  "v_mfma_f32_16x16x32_bf16 v[196:199], v[36:39], v[20:23], v[196:199]\n\t" \
  "ds_read_b64_tr_b16 v[76:77], v10 offset:0\n\t" \
  "v_mfma_f32_16x16x32_bf16 v[196:199], v[52:55], v[24:27], v[196:199]\n\t" \
  "v_mfma_f32_16x16x32_bf16 v[200:203], v[40:43], v[20:23], v[200:203]\n\t" \
  "ds_read_b64_tr_b16 v[78:79], v10 offset:2048\n\t" \
  "v_mfma_f32_16x16x32_bf16 v[200:203], v[56:59], v[24:27], v[200:203]\n\t" \
  "v_mfma_f32_16x16x32_bf16 v[204:207], v[44:47], v[20:23], v[204:207]\n\t" \
  "ds_read_b128 v[20:23], v11 offset:18432\n\t" \
  "v_mfma_f32_16x16x32_bf16 v[204:207], v[60:63], v[24:27], v[204:207]\n\t" \
  "s_waitcnt lgkmcnt(7)\n\t" \
  "v_mfma_f32_16x16x32_bf16 v[208:211], v[32:35], v[28:31], v[208:211]\n\t" \
  "s_waitcnt lgkmcnt(4)\n\t" \
  "v_mfma_f32_16x16x32_bf16 v[208:211], v[48:51], v[12:15], v[208:211]\n\t" \
  "ds_read_b128 v[24:27], v11 offset:9216\n\t" \
  "v_mfma_f32_16x16x32_bf16 v[212:215], v[36:39], v[28:31], v[212:215]\n\t" \
  "ds_read_b64_tr_b16 v[92:93], v10 offset:16384\n\t" \
  "v_mfma_f32_16x16x32_bf16 v[212:215], v[52:55], v[12:15], v[212:215]\n\t" \
  "v_mfma_f32_16x16x32_bf16 v[216:219], v[40:43], v[28:31], v[216:219]\n\t" \
  "ds_read_b64_tr_b16 v[94:95], v10 offset:18432\n\t" \
  "v_mfma_f32_16x16x32_bf16 v[216:219], v[56:59], v[12:15], v[216:219]\n\t" \
  "v_mfma_f32_16x16x32_bf16 v[220:223], v[44:47], v[28:31], v[220:223]\n\t" \
  "ds_read_b128 v[28:31], v11 offset:19456\n\t" \
  "v_mfma_f32_16x16x32_bf16 v[220:223], v[60:63], v[12:15], v[220:223]\n\t" \
  "s_waitcnt lgkmcnt(7)\n\t" \
  "v_mfma_f32_16x16x32_bf16 v[224:227], v[32:35], v[16:19], v[224:227]\n\t" \
  "s_waitcnt lgkmcnt(4)\n\t" \
  "v_mfma_f32_16x16x32_bf16 v[224:227], v[48:51], v[20:23], v[224:227]\n\t" \
  "v_add_u32 v7, %[dl], v7\n\t" \
  "v_mfma_f32_16x16x32_bf16 v[228:231], v[36:39], v[16:19], v[228:231]\n\t" \
  "v_add_u32 v8, %[dl], v8\n\t" \
  "v_mfma_f32_16x16x32_bf16 v[228:231], v[52:55], v[20:23], v[228:231]\n\t" \
  "v_add_u32 v9, %[dl], v9\n\t" \
  "v_mfma_f32_16x16x32_bf16 v[232:235], v[40:43], v[16:19], v[232:235]\n\t" \
  "v_add_u32 v10, %[dl], v10\n\t" \
  "v_mfma_f32_16x16x32_bf16 v[232:235], v[56:59], v[20:23], v[232:235]\n\t" \
  "v_add_u32 v11, %[dl], v11\n\t" \
  "v_mfma_f32_16x16x32_bf16 v[236:239], v[44:47], v[16:19], v[236:239]\n\t" \
  "v_mfma_f32_16x16x32_bf16 v[236:239], v[60:63], v[20:23], v[236:239]\n\t" \
  "s_waitcnt lgkmcnt(0)\n\t" \
  "s_barrier\n\t" \
  "ds_read_b128 v[12:15], v11 offset:0\n\t" \
  "v_mfma_f32_16x16x32_bf16 v[240:243], v[32:35], v[24:27], v[240:243]\n\t" \
  "ds_read_b128 v[16:19], v11 offset:10240\n\t" \
  "v_mfma_f32_16x16x32_bf16 v[240:243], v[48:51], v[28:31], v[240:243]\n\t" \
  "ds_read_b128 v[20:23], v11 offset:1024\n\t" \
  "v_mfma_f32_16x16x32_bf16 v[244:247], v[36:39], v[24:27], v[244:247]\n\t" \
  "v_mfma_f32_16x16x32_bf16 v[244:247], v[52:55], v[28:31], v[244:247]\n\t" \
  "v_mfma_f32_16x16x32_bf16 v[248:251], v[40:43], v[24:27], v[248:251]\n\t" \
  "v_mfma_f32_16x16x32_bf16 v[248:251], v[56:59], v[28:31], v[248:251]\n\t" \
  "v_mfma_f32_16x16x32_bf16 v[252:255], v[44:47], v[24:27], v[252:255]\n\t" \
  "ds_read_b128 v[24:27], v11 offset:11264\n\t" \
  "v_mfma_f32_16x16x32_bf16 v[252:255], v[60:63], v[28:31], v[252:255]\n\t" \
  "s_waitcnt lgkmcnt(3)\n\t" \
  "v_mfma_f32_16x16x32_bf16 v[96:99], v[64:67], v[12:15], v[96:99]\n\t" \
  "s_waitcnt lgkmcnt(2)\n\t" \
  "v_mfma_f32_16x16x32_bf16 v[96:99], v[80:83], v[16:19], v[96:99]\n\t" \
  "ds_read_b128 v[28:31], v11 offset:2048\n\t" \
  "v_mfma_f32_16x16x32_bf16 v[100:103], v[68:71], v[12:15], v[100:103]\n\t" \
  "ds_read_b64_tr_b16 v[32:33], v7 offset:0\n\t" \
  "v_mfma_f32_16x16x32_bf16 v[100:103], v[84:87], v[16:19], v[100:103]\n\t" \
  "s_add_u32 %[st], %[st], 1\n\t" \
  "v_mfma_f32_16x16x32_bf16 v[104:107], v[72:75], v[12:15], v[104:107]\n\t" \
  "ds_read_b64_tr_b16 v[34:35], v7 offset:2048\n\t" \
  "v_mfma_f32_16x16x32_bf16 v[104:107], v[88:91], v[16:19], v[104:107]\n\t" \
  "s_cmp_eq_u32 %[st], 3\n\t" \
  "v_mfma_f32_16x16x32_bf16 v[108:111], v[76:79], v[12:15], v[108:111]\n\t" \
  "ds_read_b128 v[12:15], v11 offset:12288\n\t" \
  "v_mfma_f32_16x16x32_bf16 v[108:111], v[92:95], v[16:19], v[108:111]\n\t" \
  "s_waitcnt lgkmcnt(5)\n\t" \
  "v_mfma_f32_16x16x32_bf16 v[112:115], v[64:67], v[20:23], v[112:115]\n\t" \
  "s_waitcnt lgkmcnt(4)\n\t" \
  "v_mfma_f32_16x16x32_bf16 v[112:115], v[80:83], v[24:27], v[112:115]\n\t" \
  "ds_read_b128 v[16:19], v11 offset:3072\n\t" \
  "v_mfma_f32_16x16x32_bf16 v[116:119], v[68:71], v[20:23], v[116:119]\n\t" \
  "ds_read_b64_tr_b16 v[48:49], v7 offset:16384\n\t" \
  "v_mfma_f32_16x16x32_bf16 v[116:119], v[84:87], v[24:27], v[116:119]\n\t" \
  "s_cselect_b32 %[dl], %[dn], %[dp]\n\t" \
  "v_mfma_f32_16x16x32_bf16 v[120:123], v[72:75], v[20:23], v[120:123]\n\t" \
  "ds_read_b64_tr_b16 v[50:51], v7 offset:18432\n\t" \
  "v_mfma_f32_16x16x32_bf16 v[120:123], v[88:91], v[24:27], v[120:123]\n\t" \
  "s_cselect_b32 %[st], 0, %[st]\n\t" \
  "v_mfma_f32_16x16x32_bf16 v[124:127], v[76:79], v[20:23], v[124:127]\n\t" \
  "ds_read_b128 v[20:23], v11 offset:13312\n\t" \
  "v_mfma_f32_16x16x32_bf16 v[124:127], v[92:95], v[24:27], v[124:127]\n\t" \
  "s_waitcnt lgkmcnt(7)\n\t" \
  "v_mfma_f32_16x16x32_bf16 v[128:131], v[64:67], v[28:31], v[128:131]\n\t" \
  "s_waitcnt lgkmcnt(4)\n\t" \
  "v_mfma_f32_16x16x32_bf16 v[128:131], v[80:83], v[12:15], v[128:131]\n\t" \
  "ds_read_b128 v[24:27], v11 offset:4096\n\t" \
  "v_mfma_f32_16x16x32_bf16 v[132:135], v[68:71], v[28:31], v[132:135]\n\t" \
  "ds_read_b64_tr_b16 v[36:37], v8 offset:0\n\t" \
  "v_mfma_f32_16x16x32_bf16 v[132:135], v[84:87], v[12:15], v[132:135]\n\t" \
  "v_mfma_f32_16x16x32_bf16 v[136:139], v[72:75], v[28:31], v[136:139]\n\t" \
  "ds_read_b64_tr_b16 v[38:39], v8 offset:2048\n\t" \
  "v_mfma_f32_16x16x32_bf16 v[136:139], v[88:91], v[12:15], v[136:139]\n\t" \
  "v_mfma_f32_16x16x32_bf16 v[140:143], v[76:79], v[28:31], v[140:143]\n\t" \
  "ds_read_b128 v[28:31], v11 offset:14336\n\t" \
  "v_mfma_f32_16x16x32_bf16 v[140:143], v[92:95], v[12:15], v[140:143]\n\t" \
  "s_waitcnt lgkmcnt(7)\n\t" \
  "v_mfma_f32_16x16x32_bf16 v[144:147], v[64:67], v[16:19], v[144:147]\n\t" \
  "s_waitcnt lgkmcnt(4)\n\t" \
  "v_mfma_f32_16x16x32_bf16 v[144:147], v[80:83], v[20:23], v[144:147]\n\t" \
  "ds_read_b128 v[12:15], v11 offset:5120\n\t" \
  "v_mfma_f32_16x16x32_bf16 v[148:151], v[68:71], v[16:19], v[148:151]\n\t" \
  "ds_read_b64_tr_b16 v[52:53], v8 offset:16384\n\t" \
  "v_mfma_f32_16x16x32_bf16 v[148:151], v[84:87], v[20:23], v[148:151]\n\t" \
  "v_mfma_f32_16x16x32_bf16 v[152:155], v[72:75], v[16:19], v[152:155]\n\t" \
  "ds_read_b64_tr_b16 v[54:55], v8 offset:18432\n\t" \
  "v_mfma_f32_16x16x32_bf16 v[152:155], v[88:91], v[20:23], v[152:155]\n\t" \
  "v_mfma_f32_16x16x32_bf16 v[156:159], v[76:79], v[16:19], v[156:159]\n\t" \
  "ds_read_b128 v[16:19], v11 offset:15360\n\t" \
  "v_mfma_f32_16x16x32_bf16 v[156:159], v[92:95], v[20:23], v[156:159]\n\t" \
  "s_waitcnt lgkmcnt(7)\n\t" \
  "v_mfma_f32_16x16x32_bf16 v[160:163], v[64:67], v[24:27], v[160:163]\n\t" \
  "s_waitcnt lgkmcnt(4)\n\t" \
  "v_mfma_f32_16x16x32_bf16 v[160:163], v[80:83], v[28:31], v[160:163]\n\t" \
  "ds_read_b128 v[20:23], v11 offset:6144\n\t" \
  "v_mfma_f32_16x16x32_bf16 v[164:167], v[68:71], v[24:27], v[164:167]\n\t" \
  "ds_read_b64_tr_b16 v[40:41], v9 offset:0\n\t" \
  "v_mfma_f32_16x16x32_bf16 v[164:167], v[84:87], v[28:31], v[164:167]\n\t" \
  "v_mfma_f32_16x16x32_bf16 v[168:171], v[72:75], v[24:27], v[168:171]\n\t" \
  "ds_read_b64_tr_b16 v[42:43], v9 offset:2048\n\t" \
  "v_mfma_f32_16x16x32_bf16 v[168:171], v[88:91], v[28:31], v[168:171]\n\t" \
  "v_mfma_f32_16x16x32_bf16 v[172:175], v[76:79], v[24:27], v[172:175]\n\t" \
  "ds_read_b128 v[24:27], v11 offset:16384\n\t" \
  "v_mfma_f32_16x16x32_bf16 v[172:175], v[92:95], v[28:31], v[172:175]\n\t" \
  "s_waitcnt lgkmcnt(7)\n\t" \
  "v_mfma_f32_16x16x32_bf16 v[176:179], v[64:67], v[12:15], v[176:179]\n\t" \
  "s_waitcnt lgkmcnt(4)\n\t" \
  "v_mfma_f32_16x16x32_bf16 v[176:179], v[80:83], v[16:19], v[176:179]\n\t" \
  "ds_read_b128 v[28:31], v11 offset:7168\n\t" \
  "v_mfma_f32_16x16x32_bf16 v[180:183], v[68:71], v[12:15], v[180:183]\n\t" \
  "ds_read_b64_tr_b16 v[56:57], v9 offset:16384\n\t" \
  "v_mfma_f32_16x16x32_bf16 v[180:183], v[84:87], v[16:19], v[180:183]\n\t" \
  "v_mfma_f32_16x16x32_bf16 v[184:187], v[72:75], v[12:15], v[184:187]\n\t" \
  "ds_read_b64_tr_b16 v[58:59], v9 offset:18432\n\t" \
  "v_mfma_f32_16x16x32_bf16 v[184:187], v[88:91], v[16:19], v[184:187]\n\t" \
  "v_mfma_f32_16x16x32_bf16 v[188:191], v[76:79], v[12:15], v[188:191]\n\t" \
  "ds_read_b128 v[12:15], v11 offset:17408\n\t" \
  "v_mfma_f32_16x16x32_bf16 v[188:191], v[92:95], v[16:19], v[188:191]\n\t" \
  "s_waitcnt lgkmcnt(7)\n\t" \
  "v_mfma_f32_16x16x32_bf16 v[192:195], v[64:67], v[20:23], v[192:195]\n\t" \
  "s_waitcnt lgkmcnt(4)\n\t" \
  "v_mfma_f32_16x16x32_bf16 v[192:195], v[80:83], v[24:27], v[192:195]\n\t" \
  "ds_read_b128 v[16:19], v11 offset:8192\n\t" \
  "v_mfma_f32_16x16x32_bf16 v[196:199], v[68:71], v[20:23], v[196:199]\n\t" \
  "ds_read_b64_tr_b16 v[44:45], v10 offset:0\n\t" \
  "v_mfma_f32_16x16x32_bf16 v[196:199], v[84:87], v[24:27], v[196:199]\n\t" \
  "v_mfma_f32_16x16x32_bf16 v[200:203], v[72:75], v[20:23], v[200:203]\n\t" \
  "ds_read_b64_tr_b16 v[46:47], v10 offset:2048\n\t" \
  "v_mfma_f32_16x16x32_bf16 v[200:203], v[88:91], v[24:27], v[200:203]\n\t" \
  "v_mfma_f32_16x16x32_bf16 v[204:207], v[76:79], v[20:23], v[204:207]\n\t" \
  "ds_read_b128 v[20:23], v11 offset:18432\n\t" \
  "v_mfma_f32_16x16x32_bf16 v[204:207], v[92:95], v[24:27], v[204:207]\n\t" \
  "s_waitcnt lgkmcnt(7)\n\t" \
  "v_mfma_f32_16x16x32_bf16 v[208:211], v[64:67], v[28:31], v[208:211]\n\t" \
  "s_waitcnt lgkmcnt(4)\n\t" \
  "v_mfma_f32_16x16x32_bf16 v[208:211], v[80:83], v[12:15], v[208:211]\n\t" \
  "ds_read_b128 v[24:27], v11 offset:9216\n\t" \
  "v_mfma_f32_16x16x32_bf16 v[212:215], v[68:71], v[28:31], v[212:215]\n\t" \
  "ds_read_b64_tr_b16 v[60:61], v10 offset:16384\n\t" \
  "v_mfma_f32_16x16x32_bf16 v[212:215], v[84:87], v[12:15], v[212:215]\n\t" \
  "v_mfma_f32_16x16x32_bf16 v[216:219], v[72:75], v[28:31], v[216:219]\n\t" \
  "ds_read_b64_tr_b16 v[62:63], v10 offset:18432\n\t" \
  "v_mfma_f32_16x16x32_bf16 v[216:219], v[88:91], v[12:15], v[216:219]\n\t" \
  "v_mfma_f32_16x16x32_bf16 v[220:223], v[76:79], v[28:31], v[220:223]\n\t" \
  "ds_read_b128 v[28:31], v11 offset:19456\n\t" \
  "v_mfma_f32_16x16x32_bf16 v[220:223], v[92:95], v[12:15], v[220:223]\n\t" \
  "s_waitcnt lgkmcnt(7)\n\t" \
  "v_mfma_f32_16x16x32_bf16 v[224:227], v[64:67], v[16:19], v[224:227]\n\t" \
  "s_waitcnt lgkmcnt(4)\n\t" \
  "v_mfma_f32_16x16x32_bf16 v[224:227], v[80:83], v[20:23], v[224:227]\n\t" \
  "v_add_u32 v7, %[dl], v7\n\t" \
  "v_mfma_f32_16x16x32_bf16 v[228:231], v[68:71], v[16:19], v[228:231]\n\t" \
  "v_add_u32 v8, %[dl], v8\n\t" \
  "v_mfma_f32_16x16x32_bf16 v[228:231], v[84:87], v[20:23], v[228:231]\n\t" \
  "v_add_u32 v9, %[dl], v9\n\t" \
  "v_mfma_f32_16x16x32_bf16 v[232:235], v[72:75], v[16:19], v[232:235]\n\t" \
  "v_add_u32 v10, %[dl], v10\n\t" \
  "v_mfma_f32_16x16x32_bf16 v[232:235], v[88:91], v[20:23], v[232:235]\n\t" \
  "v_add_u32 v11, %[dl], v11\n\t" \
  "v_mfma_f32_16x16x32_bf16 v[236:239], v[76:79], v[16:19], v[236:239]\n\t" \
  "v_mfma_f32_16x16x32_bf16 v[236:239], v[92:95], v[20:23], v[236:239]\n\t" \
  "s_waitcnt lgkmcnt(0)\n\t" \
  "s_barrier\n\t" \
  "ds_read_b128 v[12:15], v11 offset:0\n\t" \
  "v_mfma_f32_16x16x32_bf16 v[240:243], v[64:67], v[24:27], v[240:243]\n\t" \
  "ds_read_b128 v[16:19], v11 offset:10240\n\t" \
  "v_mfma_f32_16x16x32_bf16 v[240:243], v[80:83], v[28:31], v[240:243]\n\t" \
  "ds_read_b128 v[20:23], v11 offset:1024\n\t" \
  "v_mfma_f32_16x16x32_bf16 v[244:247], v[68:71], v[24:27], v[244:247]\n\t" \
  "v_mfma_f32_16x16x32_bf16 v[244:247], v[84:87], v[28:31], v[244:247]\n\t" \
  "v_mfma_f32_16x16x32_bf16 v[248:251], v[72:75], v[24:27], v[248:251]\n\t" \
  "s_sub_u32 %[cnt], %[cnt], 1\n\t" \
  "v_mfma_f32_16x16x32_bf16 v[248:251], v[88:91], v[28:31], v[248:251]\n\t" \
  "s_cmp_lg_u32 %[cnt], 0\n\t" \
  "v_mfma_f32_16x16x32_bf16 v[252:255], v[76:79], v[24:27], v[252:255]\n\t" \
  "ds_read_b128 v[24:27], v11 offset:11264\n\t" \
  "v_mfma_f32_16x16x32_bf16 v[252:255], v[92:95], v[28:31], v[252:255]\n\t" \
  "s_cbranch_scc1 L_cons_loop10_%=\n\t" \
  "s_waitcnt lgkmcnt(3)\n\t" \
  "v_mfma_f32_16x16x32_bf16 v[96:99], v[32:35], v[12:15], v[96:99]\n\t" \
  "s_waitcnt lgkmcnt(2)\n\t" \
  "v_mfma_f32_16x16x32_bf16 v[96:99], v[48:51], v[16:19], v[96:99]\n\t" \
  "ds_read_b128 v[28:31], v11 offset:2048\n\t" \
  "v_mfma_f32_16x16x32_bf16 v[100:103], v[36:39], v[12:15], v[100:103]\n\t" \
  "ds_read_b64_tr_b16 v[64:65], v7 offset:0\n\t" \
  "v_mfma_f32_16x16x32_bf16 v[100:103], v[52:55], v[16:19], v[100:103]\n\t" \
  "s_add_u32 %[st], %[st], 1\n\t" \
  "v_mfma_f32_16x16x32_bf16 v[104:107], v[40:43], v[12:15], v[104:107]\n\t" \
  "ds_read_b64_tr_b16 v[66:67], v7 offset:2048\n\t" \
  "v_mfma_f32_16x16x32_bf16 v[104:107], v[56:59], v[16:19], v[104:107]\n\t" \
  "s_cmp_eq_u32 %[st], 3\n\t" \
  "v_mfma_f32_16x16x32_bf16 v[108:111], v[44:47], v[12:15], v[108:111]\n\t" \
  "ds_read_b128 v[12:15], v11 offset:12288\n\t" \
  "v_mfma_f32_16x16x32_bf16 v[108:111], v[60:63], v[16:19], v[108:111]\n\t" \
  "s_waitcnt lgkmcnt(5)\n\t" \
  "v_mfma_f32_16x16x32_bf16 v[112:115], v[32:35], v[20:23], v[112:115]\n\t" \
  "s_waitcnt lgkmcnt(4)\n\t" \
  "v_mfma_f32_16x16x32_bf16 v[112:115], v[48:51], v[24:27], v[112:115]\n\t" \
  "ds_read_b128 v[16:19], v11 offset:3072\n\t" \
  "v_mfma_f32_16x16x32_bf16 v[116:119], v[36:39], v[20:23], v[116:119]\n\t" \
  "ds_read_b64_tr_b16 v[80:81], v7 offset:16384\n\t" \
  "v_mfma_f32_16x16x32_bf16 v[116:119], v[52:55], v[24:27], v[116:119]\n\t" \
  "s_cselect_b32 %[dl], %[dn], %[dp]\n\t" \
  "v_mfma_f32_16x16x32_bf16 v[120:123], v[40:43], v[20:23], v[120:123]\n\t" \
  "ds_read_b64_tr_b16 v[82:83], v7 offset:18432\n\t" \
  "v_mfma_f32_16x16x32_bf16 v[120:123], v[56:59], v[24:27], v[120:123]\n\t" \
  "s_cselect_b32 %[st], 0, %[st]\n\t" \
  "v_mfma_f32_16x16x32_bf16 v[124:127], v[44:47], v[20:23], v[124:127]\n\t" \
  "ds_read_b128 v[20:23], v11 offset:13312\n\t" \
  "v_mfma_f32_16x16x32_bf16 v[124:127], v[60:63], v[24:27], v[124:127]\n\t" \
  "s_waitcnt lgkmcnt(7)\n\t" \
  "v_mfma_f32_16x16x32_bf16 v[128:131], v[32:35], v[28:31], v[128:131]\n\t" \
  "s_waitcnt lgkmcnt(4)\n\t" \
  "v_mfma_f32_16x16x32_bf16 v[128:131], v[48:51], v[12:15], v[128:131]\n\t" \
  "ds_read_b128 v[24:27], v11 offset:4096\n\t" \
  "v_mfma_f32_16x16x32_bf16 v[132:135], v[36:39], v[28:31], v[132:135]\n\t" \
  "ds_read_b64_tr_b16 v[68:69], v8 offset:0\n\t" \
  "v_mfma_f32_16x16x32_bf16 v[132:135], v[52:55], v[12:15], v[132:135]\n\t" \
  "v_mfma_f32_16x16x32_bf16 v[136:139], v[40:43], v[28:31], v[136:139]\n\t" \
  "ds_read_b64_tr_b16 v[70:71], v8 offset:2048\n\t" \
  "v_mfma_f32_16x16x32_bf16 v[136:139], v[56:59], v[12:15], v[136:139]\n\t" \
  "v_mfma_f32_16x16x32_bf16 v[140:143], v[44:47], v[28:31], v[140:143]\n\t" \
  "ds_read_b128 v[28:31], v11 offset:14336\n\t" \
  "v_mfma_f32_16x16x32_bf16 v[140:143], v[60:63], v[12:15], v[140:143]\n\t" \
  "s_waitcnt lgkmcnt(7)\n\t" \
  "v_mfma_f32_16x16x32_bf16 v[144:147], v[32:35], v[16:19], v[144:147]\n\t" \
  "s_waitcnt lgkmcnt(4)\n\t" \
  "v_mfma_f32_16x16x32_bf16 v[144:147], v[48:51], v[20:23], v[144:147]\n\t" \
  "ds_read_b128 v[12:15], v11 offset:5120\n\t" \
  "v_mfma_f32_16x16x32_bf16 v[148:151], v[36:39], v[16:19], v[148:151]\n\t" \
  "ds_read_b64_tr_b16 v[84:85], v8 offset:16384\n\t" \
  "v_mfma_f32_16x16x32_bf16 v[148:151], v[52:55], v[20:23], v[148:151]\n\t" \
  "v_mfma_f32_16x16x32_bf16 v[152:155], v[40:43], v[16:19], v[152:155]\n\t" \
  "ds_read_b64_tr_b16 v[86:87], v8 offset:18432\n\t" \
  "v_mfma_f32_16x16x32_bf16 v[152:155], v[56:59], v[20:23], v[152:155]\n\t" \
  "v_mfma_f32_16x16x32_bf16 v[156:159], v[44:47], v[16:19], v[156:159]\n\t" \
  "ds_read_b128 v[16:19], v11 offset:15360\n\t" \
  "v_mfma_f32_16x16x32_bf16 v[156:159], v[60:63], v[20:23], v[156:159]\n\t" \
  "s_waitcnt lgkmcnt(7)\n\t" \
  "v_mfma_f32_16x16x32_bf16 v[160:163], v[32:35], v[24:27], v[160:163]\n\t" \
  "s_waitcnt lgkmcnt(4)\n\t" \
  "v_mfma_f32_16x16x32_bf16 v[160:163], v[48:51], v[28:31], v[160:163]\n\t" \
  "ds_read_b128 v[20:23], v11 offset:6144\n\t" \
  "v_mfma_f32_16x16x32_bf16 v[164:167], v[36:39], v[24:27], v[164:167]\n\t" \
  "ds_read_b64_tr_b16 v[72:73], v9 offset:0\n\t" \
  "v_mfma_f32_16x16x32_bf16 v[164:167], v[52:55], v[28:31], v[164:167]\n\t" \
  "v_mfma_f32_16x16x32_bf16 v[168:171], v[40:43], v[24:27], v[168:171]\n\t" \
  "ds_read_b64_tr_b16 v[74:75], v9 offset:2048\n\t" \
  "v_mfma_f32_16x16x32_bf16 v[168:171], v[56:59], v[28:31], v[168:171]\n\t" \
  "v_mfma_f32_16x16x32_bf16 v[172:175], v[44:47], v[24:27], v[172:175]\n\t" \
  "ds_read_b128 v[24:27], v11 offset:16384\n\t" \
  "v_mfma_f32_16x16x32_bf16 v[172:175], v[60:63], v[28:31], v[172:175]\n\t" \
  "s_waitcnt lgkmcnt(7)\n\t" \
  "v_mfma_f32_16x16x32_bf16 v[176:179], v[32:35], v[12:15], v[176:179]\n\t" \
  "s_waitcnt lgkmcnt(4)\n\t" \
  "v_mfma_f32_16x16x32_bf16 v[176:179], v[48:51], v[16:19], v[176:179]\n\t" \
  "ds_read_b128 v[28:31], v11 offset:7168\n\t" \
  "v_mfma_f32_16x16x32_bf16 v[180:183], v[36:39], v[12:15], v[180:183]\n\t" \
  "ds_read_b64_tr_b16 v[88:89], v9 offset:16384\n\t" \
  "v_mfma_f32_16x16x32_bf16 v[180:183], v[52:55], v[16:19], v[180:183]\n\t" \
  "v_mfma_f32_16x16x32_bf16 v[184:187], v[40:43], v[12:15], v[184:187]\n\t" \
  "ds_read_b64_tr_b16 v[90:91], v9 offset:18432\n\t" \
  "v_mfma_f32_16x16x32_bf16 v[184:187], v[56:59], v[16:19], v[184:187]\n\t" \
  "v_mfma_f32_16x16x32_bf16 v[188:191], v[44:47], v[12:15], v[188:191]\n\t" \
  "ds_read_b128 v[12:15], v11 offset:17408\n\t" \
  "v_mfma_f32_16x16x32_bf16 v[188:191], v[60:63], v[16:19], v[188:191]\n\t" \
  "s_waitcnt lgkmcnt(7)\n\t" \
  "v_mfma_f32_16x16x32_bf16 v[192:195], v[32:35], v[20:23], v[192:195]\n\t" \
  "s_waitcnt lgkmcnt(4)\n\t" \
  "v_mfma_f32_16x16x32_bf16 v[192:195], v[48:51], v[24:27], v[192:195]\n\t" \
  "ds_read_b128 v[16:19], v11 offset:8192\n\t" \
  "v_mfma_f32_16x16x32_bf16 v[196:199], v[36:39], v[20:23], v[196:199]\n\t" \
  "ds_read_b64_tr_b16 v[76:77], v10 offset:0\n\t" \
  "v_mfma_f32_16x16x32_bf16 v[196:199], v[52:55], v[24:27], v[196:199]\n\t" \
  "v_mfma_f32_16x16x32_bf16 v[200:203], v[40:43], v[20:23], v[200:203]\n\t" \
  "ds_read_b64_tr_b16 v[78:79], v10 offset:2048\n\t" \
  "v_mfma_f32_16x16x32_bf16 v[200:203], v[56:59], v[24:27], v[200:203]\n\t" \
  "v_mfma_f32_16x16x32_bf16 v[204:207], v[44:47], v[20:23], v[204:207]\n\t" \
  "ds_read_b128 v[20:23], v11 offset:18432\n\t" \
  "v_mfma_f32_16x16x32_bf16 v[204:207], v[60:63], v[24:27], v[204:207]\n\t" \
  "s_waitcnt lgkmcnt(7)\n\t" \
  "v_mfma_f32_16x16x32_bf16 v[208:211], v[32:35], v[28:31], v[208:211]\n\t" \
  "s_waitcnt lgkmcnt(4)\n\t" \
  "v_mfma_f32_16x16x32_bf16 v[208:211], v[48:51], v[12:15], v[208:211]\n\t" \
  "ds_read_b128 v[24:27], v11 offset:9216\n\t" \
  "v_mfma_f32_16x16x32_bf16 v[212:215], v[36:39], v[28:31], v[212:215]\n\t" \
  "ds_read_b64_tr_b16 v[92:93], v10 offset:16384\n\t" \
  "v_mfma_f32_16x16x32_bf16 v[212:215], v[52:55], v[12:15], v[212:215]\n\t" \
  "v_mfma_f32_16x16x32_bf16 v[216:219], v[40:43], v[28:31], v[216:219]\n\t" \
  "ds_read_b64_tr_b16 v[94:95], v10 offset:18432\n\t" \
  "v_mfma_f32_16x16x32_bf16 v[216:219], v[56:59], v[12:15], v[216:219]\n\t" \
  "v_mfma_f32_16x16x32_bf16 v[220:223], v[44:47], v[28:31], v[220:223]\n\t" \
  "ds_read_b128 v[28:31], v11 offset:19456\n\t" \
  "v_mfma_f32_16x16x32_bf16 v[220:223], v[60:63], v[12:15], v[220:223]\n\t" \
  "s_waitcnt lgkmcnt(7)\n\t" \
  "v_mfma_f32_16x16x32_bf16 v[224:227], v[32:35], v[16:19], v[224:227]\n\t" \
  "s_waitcnt lgkmcnt(4)\n\t" \
  "v_mfma_f32_16x16x32_bf16 v[224:227], v[48:51], v[20:23], v[224:227]\n\t" \
  "v_add_u32 v7, %[dl], v7\n\t" \
  "v_mfma_f32_16x16x32_bf16 v[228:231], v[36:39], v[16:19], v[228:231]\n\t" \
  "v_add_u32 v8, %[dl], v8\n\t" \
  "v_mfma_f32_16x16x32_bf16 v[228:231], v[52:55], v[20:23], v[228:231]\n\t" \
  "v_add_u32 v9, %[dl], v9\n\t" \
  "v_mfma_f32_16x16x32_bf16 v[232:235], v[40:43], v[16:19], v[232:235]\n\t" \
  "v_add_u32 v10, %[dl], v10\n\t" \
  "v_mfma_f32_16x16x32_bf16 v[232:235], v[56:59], v[20:23], v[232:235]\n\t" \
  "v_add_u32 v11, %[dl], v11\n\t" \
  "v_mfma_f32_16x16x32_bf16 v[236:239], v[44:47], v[16:19], v[236:239]\n\t" \
  "v_mfma_f32_16x16x32_bf16 v[236:239], v[60:63], v[20:23], v[236:239]\n\t" \
  "s_waitcnt lgkmcnt(0)\n\t" \
  "s_barrier\n\t" \
  "ds_read_b128 v[12:15], v11 offset:0\n\t" \
  "v_mfma_f32_16x16x32_bf16 v[240:243], v[32:35], v[24:27], v[240:243]\n\t" \
  "ds_read_b128 v[16:19], v11 offset:10240\n\t" \
  "v_mfma_f32_16x16x32_bf16 v[240:243], v[48:51], v[28:31], v[240:243]\n\t" \
  "ds_read_b128 v[20:23], v11 offset:1024\n\t" \
  "v_mfma_f32_16x16x32_bf16 v[244:247], v[36:39], v[24:27], v[244:247]\n\t" \
  "v_mfma_f32_16x16x32_bf16 v[244:247], v[52:55], v[28:31], v[244:247]\n\t" \
  "v_mfma_f32_16x16x32_bf16 v[248:251], v[40:43], v[24:27], v[248:251]\n\t" \
  "v_mfma_f32_16x16x32_bf16 v[248:251], v[56:59], v[28:31], v[248:251]\n\t" \
  "v_mfma_f32_16x16x32_bf16 v[252:255], v[44:47], v[24:27], v[252:255]\n\t" \
  "ds_read_b128 v[24:27], v11 offset:11264\n\t" \
  "v_mfma_f32_16x16x32_bf16 v[252:255], v[60:63], v[28:31], v[252:255]\n\t" \
  "s_waitcnt lgkmcnt(3)\n\t" \
  "v_mfma_f32_16x16x32_bf16 v[96:99], v[64:67], v[12:15], v[96:99]\n\t" \
  "s_waitcnt lgkmcnt(2)\n\t" \
  "v_mfma_f32_16x16x32_bf16 v[96:99], v[80:83], v[16:19], v[96:99]\n\t" \
  "ds_read_b128 v[28:31], v11 offset:2048\n\t" \
  "v_mfma_f32_16x16x32_bf16 v[100:103], v[68:71], v[12:15], v[100:103]\n\t" \
  "ds_read_b64_tr_b16 v[32:33], v7 offset:0\n\t" \
  "v_mfma_f32_16x16x32_bf16 v[100:103], v[84:87], v[16:19], v[100:103]\n\t" \
  "s_add_u32 %[st], %[st], 1\n\t" \
  "v_mfma_f32_16x16x32_bf16 v[104:107], v[72:75], v[12:15], v[104:107]\n\t" \
  "ds_read_b64_tr_b16 v[34:35], v7 offset:2048\n\t" \
  "v_mfma_f32_16x16x32_bf16 v[104:107], v[88:91], v[16:19], v[104:107]\n\t" \
  "s_cmp_eq_u32 %[st], 3\n\t" \
  "v_mfma_f32_16x16x32_bf16 v[108:111], v[76:79], v[12:15], v[108:111]\n\t" \
  "ds_read_b128 v[12:15], v11 offset:12288\n\t" \
  "v_mfma_f32_16x16x32_bf16 v[108:111], v[92:95], v[16:19], v[108:111]\n\t" \
  "s_waitcnt lgkmcnt(5)\n\t" \
  "v_mfma_f32_16x16x32_bf16 v[112:115], v[64:67], v[20:23], v[112:115]\n\t" \
  "s_waitcnt lgkmcnt(4)\n\t" \
  "v_mfma_f32_16x16x32_bf16 v[112:115], v[80:83], v[24:27], v[112:115]\n\t" \
  "ds_read_b128 v[16:19], v11 offset:3072\n\t" \
  "v_mfma_f32_16x16x32_bf16 v[116:119], v[68:71], v[20:23], v[116:119]\n\t" \
  "ds_read_b64_tr_b16 v[48:49], v7 offset:16384\n\t" \
  "v_mfma_f32_16x16x32_bf16 v[116:119], v[84:87], v[24:27], v[116:119]\n\t" \
  "s_cselect_b32 %[dl], %[dn], %[dp]\n\t" \
  "v_mfma_f32_16x16x32_bf16 v[120:123], v[72:75], v[20:23], v[120:123]\n\t" \
  "ds_read_b64_tr_b16 v[50:51], v7 offset:18432\n\t" \
  "v_mfma_f32_16x16x32_bf16 v[120:123], v[88:91], v[24:27], v[120:123]\n\t" \
  "s_cselect_b32 %[st], 0, %[st]\n\t" \
  "v_mfma_f32_16x16x32_bf16 v[124:127], v[76:79], v[20:23], v[124:127]\n\t" \
  "ds_read_b128 v[20:23], v11 offset:13312\n\t" \
  "v_mfma_f32_16x16x32_bf16 v[124:127], v[92:95], v[24:27], v[124:127]\n\t" \
  "s_waitcnt lgkmcnt(7)\n\t" \
  "v_mfma_f32_16x16x32_bf16 v[128:131], v[64:67], v[28:31], v[128:131]\n\t" \
  "s_waitcnt lgkmcnt(4)\n\t" \
  "v_mfma_f32_16x16x32_bf16 v[128:131], v[80:83], v[12:15], v[128:131]\n\t" \
  "ds_read_b128 v[24:27], v11 offset:4096\n\t" \
  "v_mfma_f32_16x16x32_bf16 v[132:135], v[68:71], v[28:31], v[132:135]\n\t" \
  "ds_read_b64_tr_b16 v[36:37], v8 offset:0\n\t" \
  "v_mfma_f32_16x16x32_bf16 v[132:135], v[84:87], v[12:15], v[132:135]\n\t" \
  "v_mfma_f32_16x16x32_bf16 v[136:139], v[72:75], v[28:31], v[136:139]\n\t" \
  "ds_read_b64_tr_b16 v[38:39], v8 offset:2048\n\t" \
  "v_mfma_f32_16x16x32_bf16 v[136:139], v[88:91], v[12:15], v[136:139]\n\t" \
  "v_mfma_f32_16x16x32_bf16 v[140:143], v[76:79], v[28:31], v[140:143]\n\t" \
  "ds_read_b128 v[28:31], v11 offset:14336\n\t" \
  "v_mfma_f32_16x16x32_bf16 v[140:143], v[92:95], v[12:15], v[140:143]\n\t" \
  "s_waitcnt lgkmcnt(7)\n\t" \
  "v_mfma_f32_16x16x32_bf16 v[144:147], v[64:67], v[16:19], v[144:147]\n\t" \
  "s_waitcnt lgkmcnt(4)\n\t" \
  "v_mfma_f32_16x16x32_bf16 v[144:147], v[80:83], v[20:23], v[144:147]\n\t" \
  "ds_read_b128 v[12:15], v11 offset:5120\n\t" \
  "v_mfma_f32_16x16x32_bf16 v[148:151], v[68:71], v[16:19], v[148:151]\n\t" \
  "ds_read_b64_tr_b16 v[52:53], v8 offset:16384\n\t" \
  "v_mfma_f32_16x16x32_bf16 v[148:151], v[84:87], v[20:23], v[148:151]\n\t" \
  "v_mfma_f32_16x16x32_bf16 v[152:155], v[72:75], v[16:19], v[152:155]\n\t" \
  "ds_read_b64_tr_b16 v[54:55], v8 offset:18432\n\t" \
  "v_mfma_f32_16x16x32_bf16 v[152:155], v[88:91], v[20:23], v[152:155]\n\t" \
  "v_mfma_f32_16x16x32_bf16 v[156:159], v[76:79], v[16:19], v[156:159]\n\t" \
  "ds_read_b128 v[16:19], v11 offset:15360\n\t" \
  "v_mfma_f32_16x16x32_bf16 v[156:159], v[92:95], v[20:23], v[156:159]\n\t" \
  "s_waitcnt lgkmcnt(7)\n\t" \
  "v_mfma_f32_16x16x32_bf16 v[160:163], v[64:67], v[24:27], v[160:163]\n\t" \
  "s_waitcnt lgkmcnt(4)\n\t" \
  "v_mfma_f32_16x16x32_bf16 v[160:163], v[80:83], v[28:31], v[160:163]\n\t" \
  "ds_read_b128 v[20:23], v11 offset:6144\n\t" \
  "v_mfma_f32_16x16x32_bf16 v[164:167], v[68:71], v[24:27], v[164:167]\n\t" \
  "ds_read_b64_tr_b16 v[40:41], v9 offset:0\n\t" \
  "v_mfma_f32_16x16x32_bf16 v[164:167], v[84:87], v[28:31], v[164:167]\n\t" \
  "v_mfma_f32_16x16x32_bf16 v[168:171], v[72:75], v[24:27], v[168:171]\n\t" \
  "ds_read_b64_tr_b16 v[42:43], v9 offset:2048\n\t" \
  "v_mfma_f32_16x16x32_bf16 v[168:171], v[88:91], v[28:31], v[168:171]\n\t" \
  "v_mfma_f32_16x16x32_bf16 v[172:175], v[76:79], v[24:27], v[172:175]\n\t" \
  "ds_read_b128 v[24:27], v11 offset:16384\n\t" \
  "v_mfma_f32_16x16x32_bf16 v[172:175], v[92:95], v[28:31], v[172:175]\n\t" \
  "s_waitcnt lgkmcnt(7)\n\t" \
  "v_mfma_f32_16x16x32_bf16 v[176:179], v[64:67], v[12:15], v[176:179]\n\t" \
  "s_waitcnt lgkmcnt(4)\n\t" \
  "v_mfma_f32_16x16x32_bf16 v[176:179], v[80:83], v[16:19], v[176:179]\n\t" \
  "ds_read_b128 v[28:31], v11 offset:7168\n\t" \
  "v_mfma_f32_16x16x32_bf16 v[180:183], v[68:71], v[12:15], v[180:183]\n\t" \
  "ds_read_b64_tr_b16 v[56:57], v9 offset:16384\n\t" \
  "v_mfma_f32_16x16x32_bf16 v[180:183], v[84:87], v[16:19], v[180:183]\n\t" \
  "v_mfma_f32_16x16x32_bf16 v[184:187], v[72:75], v[12:15], v[184:187]\n\t" \
  "ds_read_b64_tr_b16 v[58:59], v9 offset:18432\n\t" \
  "v_mfma_f32_16x16x32_bf16 v[184:187], v[88:91], v[16:19], v[184:187]\n\t" \
  "v_mfma_f32_16x16x32_bf16 v[188:191], v[76:79], v[12:15], v[188:191]\n\t" \
  "ds_read_b128 v[12:15], v11 offset:17408\n\t" \
  "v_mfma_f32_16x16x32_bf16 v[188:191], v[92:95], v[16:19], v[188:191]\n\t" \
  "s_waitcnt lgkmcnt(7)\n\t" \
  "v_mfma_f32_16x16x32_bf16 v[192:195], v[64:67], v[20:23], v[192:195]\n\t" \
  "s_waitcnt lgkmcnt(4)\n\t" \
  "v_mfma_f32_16x16x32_bf16 v[192:195], v[80:83], v[24:27], v[192:195]\n\t" \
  "ds_read_b128 v[16:19], v11 offset:8192\n\t" \
  "v_mfma_f32_16x16x32_bf16 v[196:199], v[68:71], v[20:23], v[196:199]\n\t" \
  "ds_read_b64_tr_b16 v[44:45], v10 offset:0\n\t" \
  "v_mfma_f32_16x16x32_bf16 v[196:199], v[84:87], v[24:27], v[196:199]\n\t" \
  "v_mfma_f32_16x16x32_bf16 v[200:203], v[72:75], v[20:23], v[200:203]\n\t" \
  "ds_read_b64_tr_b16 v[46:47], v10 offset:2048\n\t" \
  "v_mfma_f32_16x16x32_bf16 v[200:203], v[88:91], v[24:27], v[200:203]\n\t" \
  "v_mfma_f32_16x16x32_bf16 v[204:207], v[76:79], v[20:23], v[204:207]\n\t" \
  "ds_read_b128 v[20:23], v11 offset:18432\n\t" \
  "v_mfma_f32_16x16x32_bf16 v[204:207], v[92:95], v[24:27], v[204:207]\n\t" \
  "s_waitcnt lgkmcnt(7)\n\t" \
  "v_mfma_f32_16x16x32_bf16 v[208:211], v[64:67], v[28:31], v[208:211]\n\t" \
  "s_waitcnt lgkmcnt(4)\n\t" \
  "v_mfma_f32_16x16x32_bf16 v[208:211], v[80:83], v[12:15], v[208:211]\n\t" \
  "ds_read_b128 v[24:27], v11 offset:9216\n\t" \
  "v_mfma_f32_16x16x32_bf16 v[212:215], v[68:71], v[28:31], v[212:215]\n\t" \
  "ds_read_b64_tr_b16 v[60:61], v10 offset:16384\n\t" \
  "v_mfma_f32_16x16x32_bf16 v[212:215], v[84:87], v[12:15], v[212:215]\n\t" \
  "v_mfma_f32_16x16x32_bf16 v[216:219], v[72:75], v[28:31], v[216:219]\n\t" \
  "ds_read_b64_tr_b16 v[62:63], v10 offset:18432\n\t" \
  "v_mfma_f32_16x16x32_bf16 v[216:219], v[88:91], v[12:15], v[216:219]\n\t" \
  "v_mfma_f32_16x16x32_bf16 v[220:223], v[76:79], v[28:31], v[220:223]\n\t" \
  "ds_read_b128 v[28:31], v11 offset:19456\n\t" \
  "v_mfma_f32_16x16x32_bf16 v[220:223], v[92:95], v[12:15], v[220:223]\n\t" \
  "s_waitcnt lgkmcnt(7)\n\t" \
  "v_mfma_f32_16x16x32_bf16 v[224:227], v[64:67], v[16:19], v[224:227]\n\t" \
  "s_waitcnt lgkmcnt(4)\n\t" \
  "v_mfma_f32_16x16x32_bf16 v[224:227], v[80:83], v[20:23], v[224:227]\n\t" \
  "v_add_u32 v7, %[dl], v7\n\t" \
  "v_mfma_f32_16x16x32_bf16 v[228:231], v[68:71], v[16:19], v[228:231]\n\t" \
  "v_add_u32 v8, %[dl], v8\n\t" \
  "v_mfma_f32_16x16x32_bf16 v[228:231], v[84:87], v[20:23], v[228:231]\n\t" \
  "v_add_u32 v9, %[dl], v9\n\t" \
  "v_mfma_f32_16x16x32_bf16 v[232:235], v[72:75], v[16:19], v[232:235]\n\t" \
  "v_add_u32 v10, %[dl], v10\n\t" \
  "v_mfma_f32_16x16x32_bf16 v[232:235], v[88:91], v[20:23], v[232:235]\n\t" \
  "v_add_u32 v11, %[dl], v11\n\t" \
  "v_mfma_f32_16x16x32_bf16 v[236:239], v[76:79], v[16:19], v[236:239]\n\t" \
  "v_mfma_f32_16x16x32_bf16 v[236:239], v[92:95], v[20:23], v[236:239]\n\t" \
  "s_waitcnt lgkmcnt(0)\n\t" \
  "s_barrier\n\t" \
  "ds_read_b128 v[12:15], v11 offset:0\n\t" \
  "v_mfma_f32_16x16x32_bf16 v[240:243], v[64:67], v[24:27], v[240:243]\n\t" \
  "ds_read_b128 v[16:19], v11 offset:10240\n\t" \
  "ds_read_b128 v[20:23], v5 offset:0\n\t" \
  "v_mfma_f32_16x16x32_bf16 v[240:243], v[80:83], v[28:31], v[240:243]\n\t" \
  "ds_read_b64_tr_b16 v[64:65], v7 offset:0\n\t" \
  "ds_read_b64_tr_b16 v[66:67], v7 offset:2048\n\t" \
  "v_mfma_f32_16x16x32_bf16 v[244:247], v[68:71], v[24:27], v[244:247]\n\t" \
  "ds_read_b64_tr_b16 v[80:81], v7 offset:16384\n\t" \
  "ds_read_b64_tr_b16 v[82:83], v7 offset:18432\n\t" \
  "v_mfma_f32_16x16x32_bf16 v[244:247], v[84:87], v[28:31], v[244:247]\n\t" \
  "ds_read_b64_tr_b16 v[68:69], v8 offset:0\n\t" \
  "ds_read_b64_tr_b16 v[70:71], v8 offset:2048\n\t" \
  "v_mfma_f32_16x16x32_bf16 v[248:251], v[72:75], v[24:27], v[248:251]\n\t" \
  "ds_read_b64_tr_b16 v[84:85], v8 offset:16384\n\t" \
  "ds_read_b64_tr_b16 v[86:87], v8 offset:18432\n\t" \
  "v_mfma_f32_16x16x32_bf16 v[248:251], v[88:91], v[28:31], v[248:251]\n\t" \
  "ds_read_b64_tr_b16 v[72:73], v9 offset:0\n\t" \
  "ds_read_b64_tr_b16 v[74:75], v9 offset:2048\n\t" \
  "v_mfma_f32_16x16x32_bf16 v[252:255], v[76:79], v[24:27], v[252:255]\n\t" \
  "ds_read_b128 v[24:27], v5 offset:10240\n\t" \
  "ds_read_b64_tr_b16 v[88:89], v9 offset:16384\n\t" \
  "ds_read_b64_tr_b16 v[90:91], v9 offset:18432\n\t" \
  "v_mfma_f32_16x16x32_bf16 v[252:255], v[92:95], v[28:31], v[252:255]\n\t" \
  "ds_read_b64_tr_b16 v[76:77], v10 offset:0\n\t" \
  "ds_read_b64_tr_b16 v[78:79], v10 offset:2048\n\t" \
  "s_waitcnt lgkmcnt(15)\n\t" \
  "v_mfma_f32_16x16x32_bf16 v[96:99], v[32:35], v[12:15], v[96:99]\n\t" \
  "ds_read_b64_tr_b16 v[92:93], v10 offset:16384\n\t" \
  "ds_read_b64_tr_b16 v[94:95], v10 offset:18432\n\t" \
  "s_waitcnt lgkmcnt(15)\n\t" \
  "v_mfma_f32_16x16x32_bf16 v[96:99], v[48:51], v[16:19], v[96:99]\n\t" \
  "ds_read_b128 v[28:31], v11 offset:1024\n\t" \
  "v_mfma_f32_16x16x32_bf16 v[100:103], v[36:39], v[12:15], v[100:103]\n\t" \
  "v_mov_b32 v6, %[bad]\n\t" \
  "ds_read_b32 v6, v6\n\t" \
  "v_mfma_f32_16x16x32_bf16 v[100:103], v[52:55], v[16:19], v[100:103]\n\t" \
  "v_mfma_f32_16x16x32_bf16 v[104:107], v[40:43], v[12:15], v[104:107]\n\t" \
  "v_mfma_f32_16x16x32_bf16 v[104:107], v[56:59], v[16:19], v[104:107]\n\t" \
  "v_mfma_f32_16x16x32_bf16 v[108:111], v[44:47], v[12:15], v[108:111]\n\t" \
  "ds_read_b128 v[12:15], v11 offset:11264\n\t" \
  "v_mfma_f32_16x16x32_bf16 v[108:111], v[60:63], v[16:19], v[108:111]\n\t" \
  "s_waitcnt lgkmcnt(15)\n\t" \
  "v_mfma_f32_16x16x32_bf16 v[96:99], v[64:67], v[20:23], v[96:99]\n\t" \
  "s_waitcnt lgkmcnt(9)\n\t" \
  "v_mfma_f32_16x16x32_bf16 v[96:99], v[80:83], v[24:27], v[96:99]\n\t" \
  "v_xor_b32 v7, 0, %[ew]\n\t" \
  "ds_read_b128 v[16:19], v5 offset:1024\n\t" \
  "v_mfma_f32_16x16x32_bf16 v[100:103], v[68:71], v[20:23], v[100:103]\n\t" \
  "v_xor_b32 v8, 32, %[ew]\n\t" \
  "v_mfma_f32_16x16x32_bf16 v[100:103], v[84:87], v[24:27], v[100:103]\n\t" \
  "v_xor_b32 v9, 64, %[ew]\n\t" \
  "v_mfma_f32_16x16x32_bf16 v[104:107], v[72:75], v[20:23], v[104:107]\n\t" \
  "v_xor_b32 v10, 96, %[ew]\n\t" \
  "s_waitcnt lgkmcnt(8)\n\t" \
  "v_mfma_f32_16x16x32_bf16 v[104:107], v[88:91], v[24:27], v[104:107]\n\t" \
  "s_waitcnt lgkmcnt(6)\n\t" \
  "v_mfma_f32_16x16x32_bf16 v[108:111], v[76:79], v[20:23], v[108:111]\n\t" \
  "ds_read_b128 v[20:23], v5 offset:11264\n\t" \
  "s_waitcnt lgkmcnt(5)\n\t" \
  "v_mfma_f32_16x16x32_bf16 v[108:111], v[92:95], v[24:27], v[108:111]\n\t" \
  "s_waitcnt lgkmcnt(3)\n\t" \
  "v_mfma_f32_16x16x32_bf16 v[112:115], v[32:35], v[28:31], v[112:115]\n\t" \
  "v_cvt_pk_bf16_f32 v96, v96, v97\n\t" \
  "v_cvt_pk_bf16_f32 v97, v98, v99\n\t" \
  "s_waitcnt lgkmcnt(2)\n\t" \
  "v_mfma_f32_16x16x32_bf16 v[112:115], v[48:51], v[12:15], v[112:115]\n\t" \
  "ds_read_b128 v[24:27], v11 offset:2048\n\t" \
  "v_pk_max_i16 v96, v96, v6\n\t" \
  "v_pk_max_i16 v97, v97, v6\n\t" \
  "v_mfma_f32_16x16x32_bf16 v[116:119], v[36:39], v[28:31], v[116:119]\n\t" \
  "ds_write_b64 v7, v[96:97] offset:0\n\t" \
  "v_mfma_f32_16x16x32_bf16 v[116:119], v[52:55], v[12:15], v[116:119]\n\t" \
  "v_cvt_pk_bf16_f32 v100, v100, v101\n\t" \
  "v_cvt_pk_bf16_f32 v101, v102, v103\n\t" \
  "v_mfma_f32_16x16x32_bf16 v[120:123], v[40:43], v[28:31], v[120:123]\n\t" \
  "v_pk_max_i16 v100, v100, v6\n\t" \
  "v_pk_max_i16 v101, v101, v6\n\t" \
  "v_mfma_f32_16x16x32_bf16 v[120:123], v[56:59], v[12:15], v[120:123]\n\t" \
  "ds_write_b64 v8, v[100:101] offset:0\n\t" \
  "v_mfma_f32_16x16x32_bf16 v[124:127], v[44:47], v[28:31], v[124:127]\n\t" \
  "ds_read_b128 v[28:31], v11 offset:12288\n\t" \
  "v_cvt_pk_bf16_f32 v104, v104, v105\n\t" \
  "v_cvt_pk_bf16_f32 v105, v106, v107\n\t" \
  "v_mfma_f32_16x16x32_bf16 v[124:127], v[60:63], v[12:15], v[124:127]\n\t" \
  "v_pk_max_i16 v104, v104, v6\n\t" \
  "v_pk_max_i16 v105, v105, v6\n\t" \
  "s_waitcnt lgkmcnt(5)\n\t" \
  "v_mfma_f32_16x16x32_bf16 v[112:115], v[64:67], v[16:19], v[112:115]\n\t" \
  "ds_write_b64 v9, v[104:105] offset:0\n\t" \
  "s_waitcnt lgkmcnt(5)\n\t" \
  "v_mfma_f32_16x16x32_bf16 v[112:115], v[80:83], v[20:23], v[112:115]\n\t" \
  "ds_read_b128 v[12:15], v5 offset:2048\n\t" \
  "v_cvt_pk_bf16_f32 v108, v108, v109\n\t" \
  "v_cvt_pk_bf16_f32 v109, v110, v111\n\t" \
  "v_mfma_f32_16x16x32_bf16 v[116:119], v[68:71], v[16:19], v[116:119]\n\t" \
  "v_pk_max_i16 v108, v108, v6\n\t" \
  "v_pk_max_i16 v109, v109, v6\n\t" \
  "v_mfma_f32_16x16x32_bf16 v[116:119], v[84:87], v[20:23], v[116:119]\n\t" \
  "ds_write_b64 v10, v[108:109] offset:0\n\t" \
  "v_mfma_f32_16x16x32_bf16 v[120:123], v[72:75], v[16:19], v[120:123]\n\t" \
  "ds_read_b128 v[96:99], %[er] offset:0\n\t" \
  "ds_read_b128 v[100:103], %[er] offset:1024\n\t" \
  "v_mfma_f32_16x16x32_bf16 v[120:123], v[88:91], v[20:23], v[120:123]\n\t" \
  "v_mfma_f32_16x16x32_bf16 v[124:127], v[76:79], v[16:19], v[124:127]\n\t" \
  "ds_read_b128 v[16:19], v5 offset:12288\n\t" \
  "v_mfma_f32_16x16x32_bf16 v[124:127], v[92:95], v[20:23], v[124:127]\n\t" \
  "s_waitcnt lgkmcnt(9)\n\t" \
  "v_mfma_f32_16x16x32_bf16 v[128:131], v[32:35], v[24:27], v[128:131]\n\t" \
  "v_cvt_pk_bf16_f32 v112, v112, v113\n\t" \
  "v_cvt_pk_bf16_f32 v113, v114, v115\n\t" \
  "s_waitcnt lgkmcnt(6)\n\t" \
  "v_mfma_f32_16x16x32_bf16 v[128:131], v[48:51], v[28:31], v[128:131]\n\t" \
  "ds_read_b128 v[20:23], v11 offset:3072\n\t" \
  "v_pk_max_i16 v112, v112, v6\n\t" \
  "v_pk_max_i16 v113, v113, v6\n\t" \
  "v_mfma_f32_16x16x32_bf16 v[132:135], v[36:39], v[24:27], v[132:135]\n\t" \
  "ds_write_b64 v7, v[112:113] offset:2048\n\t" \
  "v_mfma_f32_16x16x32_bf16 v[132:135], v[52:55], v[28:31], v[132:135]\n\t" \
  "v_cvt_pk_bf16_f32 v116, v116, v117\n\t" \
  "v_cvt_pk_bf16_f32 v117, v118, v119\n\t" \
  "s_waitcnt lgkmcnt(4)\n\t" \
  "buffer_store_dwordx4 v[96:99], %[vo], %[out], %[so] offen" OUT_POL "\n\t" \
  "s_add_u32 %[so], %[so], 32768\n\t" \
  "s_and_b32 %[so], %[so], 0xffffff\n\t" \
  "v_mfma_f32_16x16x32_bf16 v[136:139], v[40:43], v[24:27], v[136:139]\n\t" \
  "v_pk_max_i16 v116, v116, v6\n\t" \
  "v_pk_max_i16 v117, v117, v6\n\t" \
  "v_mfma_f32_16x16x32_bf16 v[136:139], v[56:59], v[28:31], v[136:139]\n\t" \
  "ds_write_b64 v8, v[116:117] offset:2048\n\t" \
  "s_waitcnt lgkmcnt(4)\n\t" \
  "buffer_store_dwordx4 v[100:103], %[vo], %[out], %[so] offen" OUT_POL "\n\t" \
  "s_add_u32 %[so], %[so], 32768\n\t" \
  "s_and_b32 %[so], %[so], 0xffffff\n\t" \
  "v_mfma_f32_16x16x32_bf16 v[140:143], v[44:47], v[24:27], v[140:143]\n\t" \
  "ds_read_b128 v[24:27], v11 offset:13312\n\t" \
  "v_cvt_pk_bf16_f32 v120, v120, v121\n\t" \
  "v_cvt_pk_bf16_f32 v121, v122, v123\n\t" \
  "v_mfma_f32_16x16x32_bf16 v[140:143], v[60:63], v[28:31], v[140:143]\n\t" \
  "v_pk_max_i16 v120, v120, v6\n\t" \
  "v_pk_max_i16 v121, v121, v6\n\t" \
  "v_mfma_f32_16x16x32_bf16 v[128:131], v[64:67], v[12:15], v[128:131]\n\t" \
  "ds_write_b64 v9, v[120:121] offset:2048\n\t" \
  "s_waitcnt lgkmcnt(5)\n\t" \
  "v_mfma_f32_16x16x32_bf16 v[128:131], v[80:83], v[16:19], v[128:131]\n\t" \
  "ds_read_b128 v[28:31], v5 offset:3072\n\t" \
  "v_cvt_pk_bf16_f32 v124, v124, v125\n\t" \
  "v_cvt_pk_bf16_f32 v125, v126, v127\n\t" \
  "v_mfma_f32_16x16x32_bf16 v[132:135], v[68:71], v[12:15], v[132:135]\n\t" \
  "v_pk_max_i16 v124, v124, v6\n\t" \
  "v_pk_max_i16 v125, v125, v6\n\t" \
  "v_mfma_f32_16x16x32_bf16 v[132:135], v[84:87], v[16:19], v[132:135]\n\t" \
  "ds_write_b64 v10, v[124:125] offset:2048\n\t" \
  "v_mfma_f32_16x16x32_bf16 v[136:139], v[72:75], v[12:15], v[136:139]\n\t" \
  "ds_read_b128 v[112:115], %[er] offset:2048\n\t" \
  "ds_read_b128 v[116:119], %[er] offset:3072\n\t" \
  "v_mfma_f32_16x16x32_bf16 v[136:139], v[88:91], v[16:19], v[136:139]\n\t" \
  "v_mfma_f32_16x16x32_bf16 v[140:143], v[76:79], v[12:15], v[140:143]\n\t" \
  "ds_read_b128 v[12:15], v5 offset:13312\n\t" \
  "v_mfma_f32_16x16x32_bf16 v[140:143], v[92:95], v[16:19], v[140:143]\n\t" \
  "s_waitcnt lgkmcnt(9)\n\t" \
  "v_mfma_f32_16x16x32_bf16 v[144:147], v[32:35], v[20:23], v[144:147]\n\t" \
  "v_cvt_pk_bf16_f32 v128, v128, v129\n\t" \
  "v_cvt_pk_bf16_f32 v129, v130, v131\n\t" \
  "s_waitcnt lgkmcnt(6)\n\t" \
  "v_mfma_f32_16x16x32_bf16 v[144:147], v[48:51], v[24:27], v[144:147]\n\t" \
  "ds_read_b128 v[16:19], v11 offset:4096\n\t" \
  "v_pk_max_i16 v128, v128, v6\n\t" \
  "v_pk_max_i16 v129, v129, v6\n\t" \
  "v_mfma_f32_16x16x32_bf16 v[148:151], v[36:39], v[20:23], v[148:151]\n\t" \
  "ds_write_b64 v7, v[128:129] offset:4096\n\t" \
  "v_mfma_f32_16x16x32_bf16 v[148:151], v[52:55], v[24:27], v[148:151]\n\t" \
  "v_cvt_pk_bf16_f32 v132, v132, v133\n\t" \
  "v_cvt_pk_bf16_f32 v133, v134, v135\n\t" \
  "s_waitcnt lgkmcnt(4)\n\t" \
  "buffer_store_dwordx4 v[112:115], %[vo], %[out], %[so] offen" OUT_POL "\n\t" \
  "s_add_u32 %[so], %[so], 32768\n\t" \
  "s_and_b32 %[so], %[so], 0xffffff\n\t" \
  "v_mfma_f32_16x16x32_bf16 v[152:155], v[40:43], v[20:23], v[152:155]\n\t" \
  "v_pk_max_i16 v132, v132, v6\n\t" \
  "v_pk_max_i16 v133, v133, v6\n\t" \
  "v_mfma_f32_16x16x32_bf16 v[152:155], v[56:59], v[24:27], v[152:155]\n\t" \
  "ds_write_b64 v8, v[132:133] offset:4096\n\t" \
  "s_waitcnt lgkmcnt(4)\n\t" \
  "buffer_store_dwordx4 v[116:119], %[vo], %[out], %[so] offen" OUT_POL "\n\t" \
  "s_add_u32 %[so], %[so], 32768\n\t" \
  "s_and_b32 %[so], %[so], 0xffffff\n\t" \
  "v_mfma_f32_16x16x32_bf16 v[156:159], v[44:47], v[20:23], v[156:159]\n\t" \
  "ds_read_b128 v[20:23], v11 offset:14336\n\t" \
  "v_cvt_pk_bf16_f32 v136, v136, v137\n\t" \
  "v_cvt_pk_bf16_f32 v137, v138, v139\n\t" \
  "v_mfma_f32_16x16x32_bf16 v[156:159], v[60:63], v[24:27], v[156:159]\n\t" \
  "v_pk_max_i16 v136, v136, v6\n\t" \
  "v_pk_max_i16 v137, v137, v6\n\t" \
  "v_mfma_f32_16x16x32_bf16 v[144:147], v[64:67], v[28:31], v[144:147]\n\t" \
  "ds_write_b64 v9, v[136:137] offset:4096\n\t" \
  "s_waitcnt lgkmcnt(5)\n\t" \
  "v_mfma_f32_16x16x32_bf16 v[144:147], v[80:83], v[12:15], v[144:147]\n\t" \
  "ds_read_b128 v[24:27], v5 offset:4096\n\t" \
  "v_cvt_pk_bf16_f32 v140, v140, v141\n\t" \
  "v_cvt_pk_bf16_f32 v141, v142, v143\n\t" \
  "v_mfma_f32_16x16x32_bf16 v[148:151], v[68:71], v[28:31], v[148:151]\n\t" \
  "v_pk_max_i16 v140, v140, v6\n\t" \
  "v_pk_max_i16 v141, v141, v6\n\t" \
  "v_mfma_f32_16x16x32_bf16 v[148:151], v[84:87], v[12:15], v[148:151]\n\t" \
  "ds_write_b64 v10, v[140:141] offset:4096\n\t" \
  "v_mfma_f32_16x16x32_bf16 v[152:155], v[72:75], v[28:31], v[152:155]\n\t" \
  "ds_read_b128 v[128:131], %[er] offset:4096\n\t" \
  "ds_read_b128 v[132:135], %[er] offset:5120\n\t" \
  "v_mfma_f32_16x16x32_bf16 v[152:155], v[88:91], v[12:15], v[152:155]\n\t" \
  "v_mfma_f32_16x16x32_bf16 v[156:159], v[76:79], v[28:31], v[156:159]\n\t" \
  "ds_read_b128 v[28:31], v5 offset:14336\n\t" \
  "v_mfma_f32_16x16x32_bf16 v[156:159], v[92:95], v[12:15], v[156:159]\n\t" \
  "s_waitcnt lgkmcnt(9)\n\t" \
  "v_mfma_f32_16x16x32_bf16 v[160:163], v[32:35], v[16:19], v[160:163]\n\t" \
  "v_cvt_pk_bf16_f32 v144, v144, v145\n\t" \
  "v_cvt_pk_bf16_f32 v145, v146, v147\n\t" \
  "s_waitcnt lgkmcnt(6)\n\t" \
  "v_mfma_f32_16x16x32_bf16 v[160:163], v[48:51], v[20:23], v[160:163]\n\t" \
  "ds_read_b128 v[12:15], v11 offset:5120\n\t" \
  "v_pk_max_i16 v144, v144, v6\n\t" \
  "v_pk_max_i16 v145, v145, v6\n\t" \
  "v_mfma_f32_16x16x32_bf16 v[164:167], v[36:39], v[16:19], v[164:167]\n\t" \
  "ds_write_b64 v7, v[144:145] offset:6144\n\t" \
  "v_mfma_f32_16x16x32_bf16 v[164:167], v[52:55], v[20:23], v[164:167]\n\t" \
  "v_cvt_pk_bf16_f32 v148, v148, v149\n\t" \
  "v_cvt_pk_bf16_f32 v149, v150, v151\n\t" \
  "s_waitcnt lgkmcnt(4)\n\t" \
  "buffer_store_dwordx4 v[128:131], %[vo], %[out], %[so] offen" OUT_POL "\n\t" \
  "s_add_u32 %[so], %[so], 32768\n\t" \
  "s_and_b32 %[so], %[so], 0xffffff\n\t" \
  "v_mfma_f32_16x16x32_bf16 v[168:171], v[40:43], v[16:19], v[168:171]\n\t" \
  "v_pk_max_i16 v148, v148, v6\n\t" \
  "v_pk_max_i16 v149, v149, v6\n\t" \
  "v_mfma_f32_16x16x32_bf16 v[168:171], v[56:59], v[20:23], v[168:171]\n\t" \
  "ds_write_b64 v8, v[148:149] offset:6144\n\t" \
  "s_waitcnt lgkmcnt(4)\n\t" \
  "buffer_store_dwordx4 v[132:135], %[vo], %[out], %[so] offen" OUT_POL "\n\t" \
  "s_add_u32 %[so], %[so], 32768\n\t" \
  "s_and_b32 %[so], %[so], 0xffffff\n\t" \
  "v_mfma_f32_16x16x32_bf16 v[172:175], v[44:47], v[16:19], v[172:175]\n\t" \
  "ds_read_b128 v[16:19], v11 offset:15360\n\t" \
  "v_cvt_pk_bf16_f32 v152, v152, v153\n\t" \
  "v_cvt_pk_bf16_f32 v153, v154, v155\n\t" \
  "v_mfma_f32_16x16x32_bf16 v[172:175], v[60:63], v[20:23], v[172:175]\n\t" \
  "v_pk_max_i16 v152, v152, v6\n\t" \
  "v_pk_max_i16 v153, v153, v6\n\t" \
  "v_mfma_f32_16x16x32_bf16 v[160:163], v[64:67], v[24:27], v[160:163]\n\t" \
  "ds_write_b64 v9, v[152:153] offset:6144\n\t" \
  "s_waitcnt lgkmcnt(5)\n\t" \
  "v_mfma_f32_16x16x32_bf16 v[160:163], v[80:83], v[28:31], v[160:163]\n\t" \
  "ds_read_b128 v[20:23], v5 offset:5120\n\t" \
  "v_cvt_pk_bf16_f32 v156, v156, v157\n\t" \
  "v_cvt_pk_bf16_f32 v157, v158, v159\n\t" \
  "v_mfma_f32_16x16x32_bf16 v[164:167], v[68:71], v[24:27], v[164:167]\n\t" \
  "v_pk_max_i16 v156, v156, v6\n\t" \
  "v_pk_max_i16 v157, v157, v6\n\t" \
  "v_mfma_f32_16x16x32_bf16 v[164:167], v[84:87], v[28:31], v[164:167]\n\t" \
  "ds_write_b64 v10, v[156:157] offset:6144\n\t" \
  "v_mfma_f32_16x16x32_bf16 v[168:171], v[72:75], v[24:27], v[168:171]\n\t" \
  "ds_read_b128 v[144:147], %[er] offset:6144\n\t" \
  "ds_read_b128 v[148:151], %[er] offset:7168\n\t" \
  "v_mfma_f32_16x16x32_bf16 v[168:171], v[88:91], v[28:31], v[168:171]\n\t" \
  "v_mfma_f32_16x16x32_bf16 v[172:175], v[76:79], v[24:27], v[172:175]\n\t" \
  "ds_read_b128 v[24:27], v5 offset:15360\n\t" \
  "v_mfma_f32_16x16x32_bf16 v[172:175], v[92:95], v[28:31], v[172:175]\n\t" \
  "s_waitcnt lgkmcnt(9)\n\t" \
  "v_mfma_f32_16x16x32_bf16 v[176:179], v[32:35], v[12:15], v[176:179]\n\t" \
  "v_cvt_pk_bf16_f32 v160, v160, v161\n\t" \
  "v_cvt_pk_bf16_f32 v161, v162, v163\n\t" \
  "s_waitcnt lgkmcnt(6)\n\t" \
  "v_mfma_f32_16x16x32_bf16 v[176:179], v[48:51], v[16:19], v[176:179]\n\t" \
  "ds_read_b128 v[28:31], v11 offset:6144\n\t" \
  "v_pk_max_i16 v160, v160, v6\n\t" \
  "v_pk_max_i16 v161, v161, v6\n\t" \
  "v_mfma_f32_16x16x32_bf16 v[180:183], v[36:39], v[12:15], v[180:183]\n\t" \
  "ds_write_b64 v7, v[160:161] offset:8192\n\t" \
  "v_mfma_f32_16x16x32_bf16 v[180:183], v[52:55], v[16:19], v[180:183]\n\t" \
  "v_cvt_pk_bf16_f32 v164, v164, v165\n\t" \
  "v_cvt_pk_bf16_f32 v165, v166, v167\n\t" \
  "s_waitcnt lgkmcnt(4)\n\t" \
  "buffer_store_dwordx4 v[144:147], %[vo], %[out], %[so] offen" OUT_POL "\n\t" \
  "s_add_u32 %[so], %[so], 32768\n\t" \
  "s_and_b32 %[so], %[so], 0xffffff\n\t" \
  "v_mfma_f32_16x16x32_bf16 v[184:187], v[40:43], v[12:15], v[184:187]\n\t" \
  "v_pk_max_i16 v164, v164, v6\n\t" \
  "v_pk_max_i16 v165, v165, v6\n\t" \
  "v_mfma_f32_16x16x32_bf16 v[184:187], v[56:59], v[16:19], v[184:187]\n\t" \
  "ds_write_b64 v8, v[164:165] offset:8192\n\t" \
  "s_waitcnt lgkmcnt(4)\n\t" \
  "buffer_store_dwordx4 v[148:151], %[vo], %[out], %[so] offen" OUT_POL "\n\t" \
  "s_add_u32 %[so], %[so], 32768\n\t" \
  "s_and_b32 %[so], %[so], 0xffffff\n\t" \
  "v_mfma_f32_16x16x32_bf16 v[188:191], v[44:47], v[12:15], v[188:191]\n\t" \
  "ds_read_b128 v[12:15], v11 offset:16384\n\t" \
  "v_cvt_pk_bf16_f32 v168, v168, v169\n\t" \
  "v_cvt_pk_bf16_f32 v169, v170, v171\n\t" \
  "v_mfma_f32_16x16x32_bf16 v[188:191], v[60:63], v[16:19], v[188:191]\n\t" \
  "v_pk_max_i16 v168, v168, v6\n\t" \
  "v_pk_max_i16 v169, v169, v6\n\t" \
  "v_mfma_f32_16x16x32_bf16 v[176:179], v[64:67], v[20:23], v[176:179]\n\t" \
  "ds_write_b64 v9, v[168:169] offset:8192\n\t" \
  "s_waitcnt lgkmcnt(5)\n\t" \
  "v_mfma_f32_16x16x32_bf16 v[176:179], v[80:83], v[24:27], v[176:179]\n\t" \
  "ds_read_b128 v[16:19], v5 offset:6144\n\t" \
  "v_cvt_pk_bf16_f32 v172, v172, v173\n\t" \
  "v_cvt_pk_bf16_f32 v173, v174, v175\n\t" \
  "v_mfma_f32_16x16x32_bf16 v[180:183], v[68:71], v[20:23], v[180:183]\n\t" \
  "v_pk_max_i16 v172, v172, v6\n\t" \
  "v_pk_max_i16 v173, v173, v6\n\t" \
  "v_mfma_f32_16x16x32_bf16 v[180:183], v[84:87], v[24:27], v[180:183]\n\t" \
  "ds_write_b64 v10, v[172:173] offset:8192\n\t" \
  "v_mfma_f32_16x16x32_bf16 v[184:187], v[72:75], v[20:23], v[184:187]\n\t" \
  "ds_read_b128 v[160:163], %[er] offset:8192\n\t" \
  "ds_read_b128 v[164:167], %[er] offset:9216\n\t" \
  "v_mfma_f32_16x16x32_bf16 v[184:187], v[88:91], v[24:27], v[184:187]\n\t" \
  "v_mfma_f32_16x16x32_bf16 v[188:191], v[76:79], v[20:23], v[188:191]\n\t" \
  "ds_read_b128 v[20:23], v5 offset:16384\n\t" \
  "v_mfma_f32_16x16x32_bf16 v[188:191], v[92:95], v[24:27], v[188:191]\n\t" \
  "s_waitcnt lgkmcnt(9)\n\t" \
  "v_mfma_f32_16x16x32_bf16 v[192:195], v[32:35], v[28:31], v[192:195]\n\t" \
  "v_cvt_pk_bf16_f32 v176, v176, v177\n\t" \
  "v_cvt_pk_bf16_f32 v177, v178, v179\n\t" \
  "s_waitcnt lgkmcnt(6)\n\t" \
  "v_mfma_f32_16x16x32_bf16 v[192:195], v[48:51], v[12:15], v[192:195]\n\t" \
  "ds_read_b128 v[24:27], v11 offset:7168\n\t" \
  "v_pk_max_i16 v176, v176, v6\n\t" \
  "v_pk_max_i16 v177, v177, v6\n\t" \
  "v_mfma_f32_16x16x32_bf16 v[196:199], v[36:39], v[28:31], v[196:199]\n\t" \
  "ds_write_b64 v7, v[176:177] offset:10240\n\t" \
  "v_mfma_f32_16x16x32_bf16 v[196:199], v[52:55], v[12:15], v[196:199]\n\t" \
  "v_cvt_pk_bf16_f32 v180, v180, v181\n\t" \
  "v_cvt_pk_bf16_f32 v181, v182, v183\n\t" \
  "s_waitcnt lgkmcnt(4)\n\t" \
  "buffer_store_dwordx4 v[160:163], %[vo], %[out], %[so] offen" OUT_POL "\n\t" \
  "s_add_u32 %[so], %[so], 32768\n\t" \
  "s_and_b32 %[so], %[so], 0xffffff\n\t" \
  "v_mfma_f32_16x16x32_bf16 v[200:203], v[40:43], v[28:31], v[200:203]\n\t" \
  "v_pk_max_i16 v180, v180, v6\n\t" \
  "v_pk_max_i16 v181, v181, v6\n\t" \
  "v_mfma_f32_16x16x32_bf16 v[200:203], v[56:59], v[12:15], v[200:203]\n\t" \
  "ds_write_b64 v8, v[180:181] offset:10240\n\t" \
  "s_waitcnt lgkmcnt(4)\n\t" \
  "buffer_store_dwordx4 v[164:167], %[vo], %[out], %[so] offen" OUT_POL "\n\t" \
  "s_add_u32 %[so], %[so], 32768\n\t" \
  "s_and_b32 %[so], %[so], 0xffffff\n\t" \
  "v_mfma_f32_16x16x32_bf16 v[204:207], v[44:47], v[28:31], v[204:207]\n\t" \
  "ds_read_b128 v[28:31], v11 offset:17408\n\t" \
  "v_cvt_pk_bf16_f32 v184, v184, v185\n\t" \
  "v_cvt_pk_bf16_f32 v185, v186, v187\n\t" \
  "v_mfma_f32_16x16x32_bf16 v[204:207], v[60:63], v[12:15], v[204:207]\n\t" \
  "v_pk_max_i16 v184, v184, v6\n\t" \
  "v_pk_max_i16 v185, v185, v6\n\t" \
  "v_mfma_f32_16x16x32_bf16 v[192:195], v[64:67], v[16:19], v[192:195]\n\t" \
  "ds_write_b64 v9, v[184:185] offset:10240\n\t" \
  "s_waitcnt lgkmcnt(5)\n\t" \
  "v_mfma_f32_16x16x32_bf16 v[192:195], v[80:83], v[20:23], v[192:195]\n\t" \
  "ds_read_b128 v[12:15], v5 offset:7168\n\t" \
  "v_cvt_pk_bf16_f32 v188, v188, v189\n\t" \
  "v_cvt_pk_bf16_f32 v189, v190, v191\n\t" \
  "v_mfma_f32_16x16x32_bf16 v[196:199], v[68:71], v[16:19], v[196:199]\n\t" \
  "v_pk_max_i16 v188, v188, v6\n\t" \
  "v_pk_max_i16 v189, v189, v6\n\t" \
  "v_mfma_f32_16x16x32_bf16 v[196:199], v[84:87], v[20:23], v[196:199]\n\t" \
  "ds_write_b64 v10, v[188:189] offset:10240\n\t" \
  "v_mfma_f32_16x16x32_bf16 v[200:203], v[72:75], v[16:19], v[200:203]\n\t" \
  "ds_read_b128 v[176:179], %[er] offset:10240\n\t" \
  "ds_read_b128 v[180:183], %[er] offset:11264\n\t" \
  "v_mfma_f32_16x16x32_bf16 v[200:203], v[88:91], v[20:23], v[200:203]\n\t" \
  "v_mfma_f32_16x16x32_bf16 v[204:207], v[76:79], v[16:19], v[204:207]\n\t" \
  "ds_read_b128 v[16:19], v5 offset:17408\n\t" \
  "v_mfma_f32_16x16x32_bf16 v[204:207], v[92:95], v[20:23], v[204:207]\n\t" \
  "s_waitcnt lgkmcnt(9)\n\t" \
  "v_mfma_f32_16x16x32_bf16 v[208:211], v[32:35], v[24:27], v[208:211]\n\t" \
  "v_cvt_pk_bf16_f32 v192, v192, v193\n\t" \
  "v_cvt_pk_bf16_f32 v193, v194, v195\n\t" \
  "s_waitcnt lgkmcnt(6)\n\t" \
  "v_mfma_f32_16x16x32_bf16 v[208:211], v[48:51], v[28:31], v[208:211]\n\t" \
  "ds_read_b128 v[20:23], v11 offset:8192\n\t" \
  "v_pk_max_i16 v192, v192, v6\n\t" \
  "v_pk_max_i16 v193, v193, v6\n\t" \
  "v_mfma_f32_16x16x32_bf16 v[212:215], v[36:39], v[24:27], v[212:215]\n\t" \
  "ds_write_b64 v7, v[192:193] offset:12288\n\t" \
  "v_mfma_f32_16x16x32_bf16 v[212:215], v[52:55], v[28:31], v[212:215]\n\t" \
  "v_cvt_pk_bf16_f32 v196, v196, v197\n\t" \
  "v_cvt_pk_bf16_f32 v197, v198, v199\n\t" \
  "s_waitcnt lgkmcnt(4)\n\t" \
  "buffer_store_dwordx4 v[176:179], %[vo], %[out], %[so] offen" OUT_POL "\n\t" \
  "s_add_u32 %[so], %[so], 32768\n\t" \
  "s_and_b32 %[so], %[so], 0xffffff\n\t" \
  "v_mfma_f32_16x16x32_bf16 v[216:219], v[40:43], v[24:27], v[216:219]\n\t" \
  "v_pk_max_i16 v196, v196, v6\n\t" \
  "v_pk_max_i16 v197, v197, v6\n\t" \
  "v_mfma_f32_16x16x32_bf16 v[216:219], v[56:59], v[28:31], v[216:219]\n\t" \
  "ds_write_b64 v8, v[196:197] offset:12288\n\t" \
  "s_waitcnt lgkmcnt(4)\n\t" \
  "buffer_store_dwordx4 v[180:183], %[vo], %[out], %[so] offen" OUT_POL "\n\t" \
  "s_add_u32 %[so], %[so], 32768\n\t" \
  "s_and_b32 %[so], %[so], 0xffffff\n\t" \
  "v_mfma_f32_16x16x32_bf16 v[220:223], v[44:47], v[24:27], v[220:223]\n\t" \
  "ds_read_b128 v[24:27], v11 offset:18432\n\t" \
  "v_cvt_pk_bf16_f32 v200, v200, v201\n\t" \
  "v_cvt_pk_bf16_f32 v201, v202, v203\n\t" \
  "v_mfma_f32_16x16x32_bf16 v[220:223], v[60:63], v[28:31], v[220:223]\n\t" \
  "v_pk_max_i16 v200, v200, v6\n\t" \
  "v_pk_max_i16 v201, v201, v6\n\t" \
  "v_mfma_f32_16x16x32_bf16 v[208:211], v[64:67], v[12:15], v[208:211]\n\t" \
  "ds_write_b64 v9, v[200:201] offset:12288\n\t" \
  "s_waitcnt lgkmcnt(5)\n\t" \
  "v_mfma_f32_16x16x32_bf16 v[208:211], v[80:83], v[16:19], v[208:211]\n\t" \
  "ds_read_b128 v[28:31], v5 offset:8192\n\t" \
  "v_cvt_pk_bf16_f32 v204, v204, v205\n\t" \
  "v_cvt_pk_bf16_f32 v205, v206, v207\n\t" \
  "v_mfma_f32_16x16x32_bf16 v[212:215], v[68:71], v[12:15], v[212:215]\n\t" \
  "v_pk_max_i16 v204, v204, v6\n\t" \
  "v_pk_max_i16 v205, v205, v6\n\t" \
  "v_mfma_f32_16x16x32_bf16 v[212:215], v[84:87], v[16:19], v[212:215]\n\t" \
  "ds_write_b64 v10, v[204:205] offset:12288\n\t" \
  "v_mfma_f32_16x16x32_bf16 v[216:219], v[72:75], v[12:15], v[216:219]\n\t" \
  "ds_read_b128 v[192:195], %[er] offset:12288\n\t" \
  "ds_read_b128 v[196:199], %[er] offset:13312\n\t" \
  "v_mfma_f32_16x16x32_bf16 v[216:219], v[88:91], v[16:19], v[216:219]\n\t" \
  "v_mfma_f32_16x16x32_bf16 v[220:223], v[76:79], v[12:15], v[220:223]\n\t" \
  "ds_read_b128 v[12:15], v5 offset:18432\n\t" \
  "v_mfma_f32_16x16x32_bf16 v[220:223], v[92:95], v[16:19], v[220:223]\n\t" \
  "s_waitcnt lgkmcnt(9)\n\t" \
  "v_mfma_f32_16x16x32_bf16 v[224:227], v[32:35], v[20:23], v[224:227]\n\t" \
  "v_cvt_pk_bf16_f32 v208, v208, v209\n\t" \
  "v_cvt_pk_bf16_f32 v209, v210, v211\n\t" \
  "s_waitcnt lgkmcnt(6)\n\t" \
  "v_mfma_f32_16x16x32_bf16 v[224:227], v[48:51], v[24:27], v[224:227]\n\t" \
  "ds_read_b128 v[16:19], v11 offset:9216\n\t" \
  "v_pk_max_i16 v208, v208, v6\n\t" \
  "v_pk_max_i16 v209, v209, v6\n\t" \
  "v_mfma_f32_16x16x32_bf16 v[228:231], v[36:39], v[20:23], v[228:231]\n\t" \
  "ds_write_b64 v7, v[208:209] offset:14336\n\t" \
  "v_mfma_f32_16x16x32_bf16 v[228:231], v[52:55], v[24:27], v[228:231]\n\t" \
  "v_cvt_pk_bf16_f32 v212, v212, v213\n\t" \
  "v_cvt_pk_bf16_f32 v213, v214, v215\n\t" \
  "s_waitcnt lgkmcnt(4)\n\t" \
  "buffer_store_dwordx4 v[192:195], %[vo], %[out], %[so] offen" OUT_POL "\n\t" \
  "s_add_u32 %[so], %[so], 32768\n\t" \
  "s_and_b32 %[so], %[so], 0xffffff\n\t" \
  "v_mfma_f32_16x16x32_bf16 v[232:235], v[40:43], v[20:23], v[232:235]\n\t" \
  "v_pk_max_i16 v212, v212, v6\n\t" \
  "v_pk_max_i16 v213, v213, v6\n\t" \
  "v_mfma_f32_16x16x32_bf16 v[232:235], v[56:59], v[24:27], v[232:235]\n\t" \
  "ds_write_b64 v8, v[212:213] offset:14336\n\t" \
  "s_waitcnt lgkmcnt(4)\n\t" \
  "buffer_store_dwordx4 v[196:199], %[vo], %[out], %[so] offen" OUT_POL "\n\t" \
  "s_add_u32 %[so], %[so], 32768\n\t" \
  "s_and_b32 %[so], %[so], 0xffffff\n\t" \
  "v_mfma_f32_16x16x32_bf16 v[236:239], v[44:47], v[20:23], v[236:239]\n\t" \
  "ds_read_b128 v[20:23], v11 offset:19456\n\t" \
  "v_cvt_pk_bf16_f32 v216, v216, v217\n\t" \
  "v_cvt_pk_bf16_f32 v217, v218, v219\n\t" \
  "v_mfma_f32_16x16x32_bf16 v[236:239], v[60:63], v[24:27], v[236:239]\n\t" \
  "v_pk_max_i16 v216, v216, v6\n\t" \
  "v_pk_max_i16 v217, v217, v6\n\t" \
  "v_mfma_f32_16x16x32_bf16 v[224:227], v[64:67], v[28:31], v[224:227]\n\t" \
  "ds_write_b64 v9, v[216:217] offset:14336\n\t" \
  "s_waitcnt lgkmcnt(5)\n\t" \
  "v_mfma_f32_16x16x32_bf16 v[224:227], v[80:83], v[12:15], v[224:227]\n\t" \
  "ds_read_b128 v[24:27], v5 offset:9216\n\t" \
  "v_cvt_pk_bf16_f32 v220, v220, v221\n\t" \
  "v_cvt_pk_bf16_f32 v221, v222, v223\n\t" \
  "v_mfma_f32_16x16x32_bf16 v[228:231], v[68:71], v[28:31], v[228:231]\n\t" \
  "v_pk_max_i16 v220, v220, v6\n\t" \
  "v_pk_max_i16 v221, v221, v6\n\t" \
  "v_mfma_f32_16x16x32_bf16 v[228:231], v[84:87], v[12:15], v[228:231]\n\t" \
  "ds_write_b64 v10, v[220:221] offset:14336\n\t" \
  "v_mfma_f32_16x16x32_bf16 v[232:235], v[72:75], v[28:31], v[232:235]\n\t" \
  "ds_read_b128 v[208:211], %[er] offset:14336\n\t" \
  "ds_read_b128 v[212:215], %[er] offset:15360\n\t" \
  "v_mfma_f32_16x16x32_bf16 v[232:235], v[88:91], v[12:15], v[232:235]\n\t" \
  "v_mfma_f32_16x16x32_bf16 v[236:239], v[76:79], v[28:31], v[236:239]\n\t" \
  "ds_read_b128 v[28:31], v5 offset:19456\n\t" \
  "v_mfma_f32_16x16x32_bf16 v[236:239], v[92:95], v[12:15], v[236:239]\n\t" \
  "s_waitcnt lgkmcnt(9)\n\t" \
  "v_mfma_f32_16x16x32_bf16 v[240:243], v[32:35], v[16:19], v[240:243]\n\t" \
  "v_cvt_pk_bf16_f32 v224, v224, v225\n\t" \
  "v_cvt_pk_bf16_f32 v225, v226, v227\n\t" \
  "s_waitcnt lgkmcnt(6)\n\t" \
  "v_mfma_f32_16x16x32_bf16 v[240:243], v[48:51], v[20:23], v[240:243]\n\t" \
  "v_pk_max_i16 v224, v224, v6\n\t" \
  "v_pk_max_i16 v225, v225, v6\n\t" \
  "v_mfma_f32_16x16x32_bf16 v[244:247], v[36:39], v[16:19], v[244:247]\n\t" \
  "ds_write_b64 v7, v[224:225] offset:16384\n\t" \
  "v_mfma_f32_16x16x32_bf16 v[244:247], v[52:55], v[20:23], v[244:247]\n\t" \
  "v_cvt_pk_bf16_f32 v228, v228, v229\n\t" \
  "v_cvt_pk_bf16_f32 v229, v230, v231\n\t" \
  "s_waitcnt lgkmcnt(3)\n\t" \
  "buffer_store_dwordx4 v[208:211], %[vo], %[out], %[so] offen" OUT_POL "\n\t" \
  "s_add_u32 %[so], %[so], 32768\n\t" \
  "s_and_b32 %[so], %[so], 0xffffff\n\t" \
  "v_mfma_f32_16x16x32_bf16 v[248:251], v[40:43], v[16:19], v[248:251]\n\t" \
  "v_pk_max_i16 v228, v228, v6\n\t" \
  "v_pk_max_i16 v229, v229, v6\n\t" \
  "v_mfma_f32_16x16x32_bf16 v[248:251], v[56:59], v[20:23], v[248:251]\n\t" \
  "ds_write_b64 v8, v[228:229] offset:16384\n\t" \
  "s_waitcnt lgkmcnt(3)\n\t" \
  "buffer_store_dwordx4 v[212:215], %[vo], %[out], %[so] offen" OUT_POL "\n\t" \
  "s_add_u32 %[so], %[so], 32768\n\t" \
  "s_and_b32 %[so], %[so], 0xffffff\n\t" \
  "v_mfma_f32_16x16x32_bf16 v[252:255], v[44:47], v[16:19], v[252:255]\n\t" \
  "v_cvt_pk_bf16_f32 v232, v232, v233\n\t" \
  "v_cvt_pk_bf16_f32 v233, v234, v235\n\t" \
  "v_mfma_f32_16x16x32_bf16 v[252:255], v[60:63], v[20:23], v[252:255]\n\t" \
  "v_pk_max_i16 v232, v232, v6\n\t" \
  "v_pk_max_i16 v233, v233, v6\n\t" \
  "v_mfma_f32_16x16x32_bf16 v[240:243], v[64:67], v[24:27], v[240:243]\n\t" \
  "ds_write_b64 v9, v[232:233] offset:16384\n\t" \
  "s_waitcnt lgkmcnt(3)\n\t" \
  "v_mfma_f32_16x16x32_bf16 v[240:243], v[80:83], v[28:31], v[240:243]\n\t" \
  "v_cvt_pk_bf16_f32 v236, v236, v237\n\t" \
  "v_cvt_pk_bf16_f32 v237, v238, v239\n\t" \
  "v_mfma_f32_16x16x32_bf16 v[244:247], v[68:71], v[24:27], v[244:247]\n\t" \
  "v_pk_max_i16 v236, v236, v6\n\t" \
  "v_pk_max_i16 v237, v237, v6\n\t" \
  "v_mfma_f32_16x16x32_bf16 v[244:247], v[84:87], v[28:31], v[244:247]\n\t" \
  "ds_write_b64 v10, v[236:237] offset:16384\n\t" \
  "v_mfma_f32_16x16x32_bf16 v[248:251], v[72:75], v[24:27], v[248:251]\n\t" \
  "ds_read_b128 v[224:227], %[er] offset:16384\n\t" \
  "ds_read_b128 v[228:231], %[er] offset:17408\n\t" \
  "v_mfma_f32_16x16x32_bf16 v[248:251], v[88:91], v[28:31], v[248:251]\n\t" \
  "v_mfma_f32_16x16x32_bf16 v[252:255], v[76:79], v[24:27], v[252:255]\n\t" \
  "v_mfma_f32_16x16x32_bf16 v[252:255], v[92:95], v[28:31], v[252:255]\n\t" \
  "s_nop 3\n\t" \
  "v_cvt_pk_bf16_f32 v240, v240, v241\n\t" \
  "v_cvt_pk_bf16_f32 v241, v242, v243\n\t" \
  "v_pk_max_i16 v240, v240, v6\n\t" \
  "v_pk_max_i16 v241, v241, v6\n\t" \
  "ds_write_b64 v7, v[240:241] offset:18432\n\t" \
  "v_cvt_pk_bf16_f32 v244, v244, v245\n\t" \
  "v_cvt_pk_bf16_f32 v245, v246, v247\n\t" \
  "v_pk_max_i16 v244, v244, v6\n\t" \
  "v_pk_max_i16 v245, v245, v6\n\t" \
  "ds_write_b64 v8, v[244:245] offset:18432\n\t" \
  "s_waitcnt lgkmcnt(3)\n\t" \
  "buffer_store_dwordx4 v[224:227], %[vo], %[out], %[so] offen" OUT_POL "\n\t" \
  "s_add_u32 %[so], %[so], 32768\n\t" \
  "s_and_b32 %[so], %[so], 0xffffff\n\t" \
  "s_waitcnt lgkmcnt(2)\n\t" \
  "buffer_store_dwordx4 v[228:231], %[vo], %[out], %[so] offen" OUT_POL "\n\t" \
  "s_add_u32 %[so], %[so], 32768\n\t" \
  "s_and_b32 %[so], %[so], 0xffffff\n\t" \
  "v_cvt_pk_bf16_f32 v248, v248, v249\n\t" \
  "v_cvt_pk_bf16_f32 v249, v250, v251\n\t" \
  "v_pk_max_i16 v248, v248, v6\n\t" \
  "v_pk_max_i16 v249, v249, v6\n\t" \
  "ds_write_b64 v9, v[248:249] offset:18432\n\t" \
  "s_nop 3\n\t" \
  "v_cvt_pk_bf16_f32 v252, v252, v253\n\t" \
  "v_cvt_pk_bf16_f32 v253, v254, v255\n\t" \
  "v_pk_max_i16 v252, v252, v6\n\t" \
  "v_pk_max_i16 v253, v253, v6\n\t" \
  "ds_write_b64 v10, v[252:253] offset:18432\n\t" \
  "ds_read_b128 v[240:243], %[er] offset:18432\n\t" \
  "ds_read_b128 v[244:247], %[er] offset:19456\n\t" \
  "s_waitcnt lgkmcnt(1)\n\t" \
  "buffer_store_dwordx4 v[240:243], %[vo], %[out], %[so] offen" OUT_POL "\n\t" \
  "s_add_u32 %[so], %[so], 32768\n\t" \
  "s_and_b32 %[so], %[so], 0xffffff\n\t" \
  "s_waitcnt lgkmcnt(0)\n\t" \
  "buffer_store_dwordx4 v[244:247], %[vo], %[out], %[so] offen" OUT_POL "\n\t" \
  "s_add_u32 %[so], %[so], 32768\n\t" \
  "s_and_b32 %[so], %[so], 0xffffff\n\t" \
  "s_setprio 0\n\t" \
  ""
#define XDMA_ASM \
  "s_load_dwordx8 %0, %5, 0x0 glc\n\t" \
  "s_mov_b32 m0, %1\n\t" \
  "s_nop 0\n\t" \
  "buffer_load_dwordx4 %3, %4, %2 offen offset:0 sc0 sc1 lds\n\t" \
  "buffer_load_dwordx4 %3, %4, %2 offen offset:1024 sc0 sc1 lds\n\t" \
  "buffer_load_dwordx4 %3, %4, %2 offen offset:2048 sc0 sc1 lds\n\t" \
  "buffer_load_dwordx4 %3, %4, %2 offen offset:3072 sc0 sc1 lds\n\t" \
  "s_add_u32 %1, %1, 4096\n\t" \
  "s_add_u32 %2, %2, 4096\n\t" \
  "s_mov_b32 m0, %1\n\t" \
  "s_nop 0\n\t" \
  "buffer_load_dwordx4 %3, %4, %2 offen offset:0 sc0 sc1 lds\n\t" \
  "buffer_load_dwordx4 %3, %4, %2 offen offset:1024 sc0 sc1 lds\n\t" \
  "buffer_load_dwordx4 %3, %4, %2 offen offset:2048 sc0 sc1 lds\n\t" \
  "buffer_load_dwordx4 %3, %4, %2 offen offset:3072 sc0 sc1 lds\n\t" \
  "s_add_u32 %1, %1, 4096\n\t" \
  "s_add_u32 %2, %2, 4096\n\t" \
  "s_mov_b32 m0, %1\n\t" \
  "s_nop 0\n\t" \
  "buffer_load_dwordx4 %3, %4, %2 offen offset:0 sc0 sc1 lds\n\t" \
  "buffer_load_dwordx4 %6, %4, %2 offen offset:1024 sc0 sc1 lds\n\t" \
  "s_add_u32 %1, %1, 2048\n\t" \
  "s_add_u32 %2, %2, 253952\n\t" \
  "s_mov_b32 m0, %1\n\t" \
  "s_nop 0\n\t" \
  "buffer_load_dwordx4 %3, %4, %2 offen offset:0 sc0 sc1 lds\n\t" \
  "buffer_load_dwordx4 %3, %4, %2 offen offset:1024 sc0 sc1 lds\n\t" \
  "buffer_load_dwordx4 %3, %4, %2 offen offset:2048 sc0 sc1 lds\n\t" \
  "buffer_load_dwordx4 %3, %4, %2 offen offset:3072 sc0 sc1 lds\n\t" \
  "s_add_u32 %1, %1, 4096\n\t" \
  "s_add_u32 %2, %2, 4096\n\t" \
  "s_mov_b32 m0, %1\n\t" \
  "s_nop 0\n\t" \
  "buffer_load_dwordx4 %3, %4, %2 offen offset:0 sc0 sc1 lds\n\t" \
  "buffer_load_dwordx4 %3, %4, %2 offen offset:1024 sc0 sc1 lds\n\t" \
  "buffer_load_dwordx4 %3, %4, %2 offen offset:2048 sc0 sc1 lds\n\t" \
  "buffer_load_dwordx4 %3, %4, %2 offen offset:3072 sc0 sc1 lds\n\t" \
  "s_add_u32 %1, %1, 4096\n\t" \
  "s_add_u32 %2, %2, 4096\n\t" \
  "s_mov_b32 m0, %1\n\t" \
  "s_nop 0\n\t" \
  "buffer_load_dwordx4 %3, %4, %2 offen offset:0 sc0 sc1 lds\n\t" \
  "buffer_load_dwordx4 %6, %4, %2 offen offset:1024 sc0 sc1 lds\n\t" \
  ""
#define CONS_CLOBBERS "v5", "v6", "v8", "v9", "v10", "v12", "v13", "v14", "v15", "v16", "v17", "v18", "v19", "v20", "v21", "v22", "v23", "v24", "v25", "v26", "v27", "v28", "v29", "v30", "v31", "v32", "v33", "v34", "v35", "v36", "v37", "v38", "v39", "v40", "v41", "v42", "v43", "v44", "v45", "v46", "v47", "v48", "v49", "v50", "v51", "v52", "v53", "v54", "v55", "v56", "v57", "v58", "v59", "v60", "v61", "v62", "v63", "v64", "v65", "v66", "v67", "v68", "v69", "v70", "v71", "v72", "v73", "v74", "v75", "v76", "v77", "v78", "v79", "v80", "v81", "v82", "v83", "v84", "v85", "v86", "v87", "v88", "v89", "v90", "v91", "v92", "v93", "v94", "v95", "v96", "v97", "v98", "v99", "v100", "v101", "v102", "v103", "v104", "v105", "v106", "v107", "v108", "v109", "v110", "v111", "v112", "v113", "v114", "v115", "v116", "v117", "v118", "v119", "v120", "v121", "v122", "v123", "v124", "v125", "v126", "v127", "v128", "v129", "v130", "v131", "v132", "v133", "v134", "v135", "v136", "v137", "v138", "v139", "v140", "v141", "v142", "v143", "v144", "v145", "v146", "v147", "v148", "v149", "v150", "v151", "v152", "v153", "v154", "v155", "v156", "v157", "v158", "v159", "v160", "v161", "v162", "v163", "v164", "v165", "v166", "v167", "v168", "v169", "v170", "v171", "v172", "v173", "v174", "v175", "v176", "v177", "v178", "v179", "v180", "v181", "v182", "v183", "v184", "v185", "v186", "v187", "v188", "v189", "v190", "v191", "v192", "v193", "v194", "v195", "v196", "v197", "v198", "v199", "v200", "v201", "v202", "v203", "v204", "v205", "v206", "v207", "v208", "v209", "v210", "v211", "v212", "v213", "v214", "v215", "v216", "v217", "v218", "v219", "v220", "v221", "v222", "v223", "v224", "v225", "v226", "v227", "v228", "v229", "v230", "v231", "v232", "v233", "v234", "v235", "v236", "v237", "v238", "v239", "v240", "v241", "v242", "v243", "v244", "v245", "v246", "v247", "v248", "v249", "v250", "v251", "v252", "v253", "v254", "v255"


typedef unsigned u32x4 __attribute__((ext_vector_type(4)));
typedef unsigned u32x2 __attribute__((ext_vector_type(2)));
typedef unsigned u32x8 __attribute__((ext_vector_type(8)));
typedef unsigned short u16;
typedef unsigned short u16x2 __attribute__((ext_vector_type(2)));
typedef unsigned long long u64;
#define LAS __attribute__((address_space(3)))
#define GAS __attribute__((address_space(1)))

namespace {
constexpr int NR = 8;
constexpr int M = 4096, K = 4096, N = 2048;
constexpr int NT = 64;
constexpr int MB = 10;
constexpr int N9 = 14;
constexpr int NRT = N9 + (256 - 9 * N9) / 10;
constexpr int NG = NRT * 8, NC = 256 - NG;
constexpr int XB = MB * 2048, STAGE = XB + 32768;
constexpr int LDS_MISC = 3 * STAGE;
constexpr int LDS_BYTES = 3 * STAGE + 64 + 256;
constexpr int T0 = 1, TB = 2;
constexpr int LEAD = 2;
constexpr int T_EARLY = 3, T_FAST = 62;
constexpr int PPP = 40;
constexpr int WPF0 = 3, WPFGAP = 3;
static_assert(NRT == 27 && PPP * 7 <= NC * 8 && TB * 112 <= NC * 8 && TB >= T0 && TB <= T0 + LEAD && 8 * 4096 <= STAGE - XB, "roles");
constexpr unsigned HDR_RAW = 127u, HDR_NONE = 121u, HDR_EXC = 128u, EXC_CAP = 15u;
constexpr unsigned HDR_V = 248u;
constexpr size_t OFF_CNT = 0;
constexpr size_t OFF_TMO = 16384;
constexpr size_t OFF_TK = 20480;
constexpr size_t OFF_CLAIM = 24576;
constexpr size_t OFF_PFLAG = 32768;
constexpr size_t OFF_XFLAG = 262144;
constexpr size_t OFF_EXC = 786432;
constexpr size_t OFF_P = 1u << 20;
constexpr size_t P_BYTES = 1024 * 3072;
constexpr size_t OFF_XW = 8u << 20;
constexpr size_t XW_BYTES = (size_t)M * K * 2;
constexpr size_t WS_NEED = OFF_XW + 2 * XW_BYTES;
constexpr u64 DEADLINE = 50000000ull;

struct Params { const char* x; char* self; int rank; int pad; const char* w; u16* out; char* ws[NR]; };

#define BAR() do { asm volatile("" ::: "memory"); __builtin_amdgcn_s_barrier(); asm volatile("" ::: "memory"); } while (0)
template <int N_> __device__ __forceinline__ void waitv() { asm volatile("s_waitcnt vmcnt(%0)" :: "i"(N_) : "memory"); }
__device__ __forceinline__ void st16_sys(const char* base, unsigned voff, u32x4 v) {
  asm volatile("global_store_dwordx4 %0, %1, %2 sc0 sc1\n\ts_nop 1" :: "v"(voff), "v"(v), "s"(base) : "memory");
}
__device__ __forceinline__ void st_flag(const char* base, unsigned off, unsigned v) {
  asm volatile("global_store_dword %0, %1, %2 sc0 sc1" :: "v"(off), "v"(v), "s"(base) : "memory");
}
__device__ __forceinline__ unsigned ld_agent(const char* p) { return __hip_atomic_load((const GAS unsigned*)p, __ATOMIC_RELAXED, __HIP_MEMORY_SCOPE_AGENT); }
__device__ __forceinline__ unsigned ld_sys(const char* p) { return __hip_atomic_load((const GAS unsigned*)p, __ATOMIC_RELAXED, __HIP_MEMORY_SCOPE_SYSTEM); }
__device__ __forceinline__ void st_agent(char* p, unsigned v) { __hip_atomic_store((GAS unsigned*)p, v, __ATOMIC_RELAXED, __HIP_MEMORY_SCOPE_AGENT); }
__device__ __forceinline__ unsigned rfl(unsigned v) { return __builtin_amdgcn_readfirstlane(v); }
#define DPP0(v, ctrl, rows) ((unsigned)__builtin_amdgcn_update_dpp(0u, v, ctrl, rows, 0xf, false))
__device__ __forceinline__ unsigned wave_scan(unsigned v) {
  v += DPP0(v, 0x111, 0xf); v += DPP0(v, 0x112, 0xf); v += DPP0(v, 0x114, 0xf); v += DPP0(v, 0x118, 0xf);
  v += DPP0(v, 0x142, 0xa); v += DPP0(v, 0x143, 0xc);
  return v;
}
__device__ __forceinline__ unsigned wave_max(unsigned v) {
  v = max(v, DPP0(v, 0x111, 0xf)); v = max(v, DPP0(v, 0x112, 0xf)); v = max(v, DPP0(v, 0x114, 0xf)); v = max(v, DPP0(v, 0x118, 0xf));
  v = max(v, DPP0(v, 0x142, 0xa)); v = max(v, DPP0(v, 0x143, 0xc));
  return __builtin_amdgcn_readlane(v, 63);
}
__host__ __device__ constexpr unsigned tile_start(unsigned ro) {
  return ro < 14u ? 33u * (ro >> 1) - 5u * ((ro >> 1) & 1u) + 9u * (ro & 1u)
       : ro < 24u ? 66u * ((ro - 14u) / 3u) + ((ro - 14u) % 3u ? 36u + 10u * ((ro - 14u) % 3u) : 18u) : 226u + 10u * (ro - 24u);
}
__host__ __device__ constexpr unsigned tile_of(unsigned q) {
  return q >= 226u ? 24u + (q - 226u) / 10u
       : q % 66u < 18u ? 4u * (q / 66u) + (q % 66u) / 9u : q % 66u < 28u ? 14u + 3u * (q / 66u)
       : q % 66u < 46u ? 4u * (q / 66u) + 2u + (q % 66u - 28u) / 9u : 15u + 3u * (q / 66u) + (q % 66u - 46u) / 10u;
}
constexpr unsigned FL_TILE = NRT * 128;
__device__ __forceinline__ unsigned flag_off(unsigned q) {
  const unsigned ro = tile_of(q);
  return (unsigned)OFF_XFLAG + ro * 128u + ((q >> 1) - (tile_start(ro) >> 1)) * 4u;
}
__device__ __forceinline__ unsigned bfi(unsigned m, unsigned a, unsigned b) { return (m & a) | (~m & b); }
__device__ __forceinline__ unsigned rot4(unsigned e4) { return bfi(0x80808080u, e4 << 7, e4 >> 1); }
}

__global__ void __launch_bounds__(512, 2) ag_gemm_kernel(Params P) {
  extern __shared__ __attribute__((aligned(16))) char smem[];
  const int b = blockIdx.x;
  const int wid = rfl(threadIdx.x >> 6), lane = threadIdx.x & 63;
  const int rank = P.rank;
  char* const ws = P.self;
  const char* const px = P.x;
  asm volatile("" :: "s"(px), "s"(ws), "s"(rank));
  const unsigned lds0 = (unsigned)(size_t)(LAS char*)smem;
  volatile LAS unsigned* misc = (volatile LAS unsigned*)(smem + LDS_MISC);
  const bool is_gemm = b < NG;
  unsigned call;
  bool pushed = false;
  const int gran = wid < 4 ? b + T0 * 4 : (2 * NG - 1 - b < 256 ? (2 * NG - 1 - b + T0 * 4) & 255 : 256);
  const bool packs = is_gemm && gran < 256;
  const int task = (b - NG) + NC * wid;
  if (is_gemm && wid == 7) { if (lane < 8) misc[8 + lane] = 0u; asm volatile("s_waitcnt lgkmcnt(0)" ::: "memory"); }
  auto claim = [&](unsigned call) {
    unsigned want;
    asm volatile("s_getreg_b32 %0, hwreg(HW_REG_XCC_ID, 0, 4)" : "=s"(want));
    want &= 7u;
#pragma unroll 1
    for (int i = 0; i < 8; ++i, want = (want + 1u) & 7u) {
      unsigned old = 0;
      if (lane == 0) old = __hip_atomic_exchange((GAS unsigned*)(ws + OFF_CLAIM + (b >> 3) * 128 + want * 4), call, __ATOMIC_RELAXED, __HIP_MEMORY_SCOPE_AGENT);
      if (rfl(old) != call) break;
    }
    if (lane == 0) misc[9] = want + 1u;
  };
  if (packs) {
    const int u = 4 * gran + (wid & 3), t = u >> 4, pair = u & 15;
    const char* src = px + (size_t)(32 * pair + (lane & 15)) * (K * 2) + t * 128 + (lane >> 4) * 16;
    char* const peer_ws = P.ws[lane & 7];
    u32x4 d[4];
    d[0] = *(const u32x4*)(src); d[1] = *(const u32x4*)(src + 16 * K * 2);
    d[2] = *(const u32x4*)(src + 64); d[3] = *(const u32x4*)(src + 16 * K * 2 + 64);
    const unsigned cnt0 = ld_agent(ws + OFF_CNT + b * 64);
    __builtin_amdgcn_s_barrier();
    call = rfl(cnt0) + 1u;
    if (wid == 7) claim(call);
    const unsigned par = call & 1u;
    const char* xw = ws + OFF_XW + par * XW_BYTES + ((size_t)(2 * t) * 256 + 224 + 2 * pair) * 1024;
    st16_sys(xw, lane * 16, d[0]); st16_sys(xw, lane * 16 + 1024, d[1]);
    st16_sys(xw, lane * 16 + 262144, d[2]); st16_sys(xw, lane * 16 + 262144 + 1024, d[3]);
    unsigned hdr = t < T0 ? HDR_RAW : 0u;
    const char* pp = ws + OFF_P + par * P_BYTES + (size_t)u * 3072;
    if (t >= T_EARLY && t < T_FAST) {
      LAS char* sc = (LAS char*)smem + STAGE + XB + wid * 4096;
      *(LAS u32x4*)(sc + lane * 16) = (u32x4){0x04040404u, 0x04040404u, 0x04040404u, 0x04040404u};
      unsigned e16[16], sum = 0;
#pragma unroll
      for (int i = 0; i < 16; ++i) { e16[i] = (d[i >> 2][i & 3] >> 7) & 0x00ff00ffu; sum = __builtin_amdgcn_sad_u8(e16[i], 0u, sum); }
      sum = __builtin_amdgcn_readlane(wave_scan(sum), 63);
      const unsigned top = min(max((sum + 8008u) >> 11, 15u), 255u);
      const u16x2 top2 = {(u16)top, (u16)top}, seven = {7, 7};
      unsigned x4[8], c4[8];
      u32x4 sm[2];
#pragma unroll
      for (int g = 0; g < 8; ++g) {
        const u16x2 xa = top2 - __builtin_bit_cast(u16x2, e16[2 * g]), xb = top2 - __builtin_bit_cast(u16x2, e16[2 * g + 1]);
        x4[g] = __builtin_amdgcn_perm(__builtin_bit_cast(unsigned, xb), __builtin_bit_cast(unsigned, xa), 0x06040200u);
        const unsigned u4 = __builtin_amdgcn_perm(__builtin_bit_cast(unsigned, __builtin_elementwise_min(xb, seven)),
                                                  __builtin_bit_cast(unsigned, __builtin_elementwise_min(xa, seven)), 0x06040200u);
        c4[g] = __builtin_amdgcn_perm(0x00000003u, 0x02010000u, u4);
        x4[g] += __builtin_amdgcn_perm(0u, 0x00000300u, u4);
        const unsigned da = d[g >> 1][2 * (g & 1)], db = d[g >> 1][2 * (g & 1) + 1];
        sm[g >> 2][g & 3] = bfi(0x80808080u, __builtin_amdgcn_perm(db, da, 0x07050301u), __builtin_amdgcn_perm(db, da, 0x06040200u));
      }
      const u32x2 A = {c4[0] | (c4[1] << 2) | (c4[2] << 4) | (c4[3] << 6), c4[4] | (c4[5] << 2) | (c4[6] << 4) | (c4[7] << 6)};
      const unsigned cnt = 32u - __builtin_popcount((A[0] | (A[0] >> 1)) & 0x55555555u) - __builtin_popcount((A[1] | (A[1] >> 1)) & 0x55555555u);
      const unsigned inc = wave_scan(cnt), total = __builtin_amdgcn_readlane(inc, 63);
      if (total <= 1024u) {
        unsigned off = inc - cnt;
#pragma unroll
        for (int g = 0; g < 8; ++g)
#pragma unroll
          for (int bb = 0; bb < 4; ++bb) {
            const bool esc = ((c4[g] >> (8 * bb)) & 3u) == 0u;
            *(LAS unsigned char*)(sc + (esc ? off : 1024u + lane)) = (unsigned char)(x4[g] >> (8 * bb));
            off += esc ? 1u : 0u;
          }
        *(LAS u32x2*)(sc + 2048 + lane * 8) = A;
        const unsigned na = (total + 31u) >> 5, eb = 516u + 12u * na;
        const u32x4 y = *(const LAS u32x4*)(sc + lane * 16);
        unsigned exm[4], pa = 0, pb = 0;
#pragma unroll
        for (int r = 0; r < 4; ++r) {
          const unsigned tt = (y[r] | 0x80808080u) - 0x04040404u;
          const unsigned e = ((tt ^ 0x80808080u) | (y[r] & 0x80808080u)) & 0xf8f8f8f8u;
          exm[r] = ((e >> 1) + 0x7c7c7c7cu) & 0x80808080u;
          pa |= (tt & 0x03030303u) << (2 * r);
          pb |= ((tt >> 2) & 0x01010101u) << r;
        }
        if ((unsigned)lane < 2u * na) {
          *(LAS unsigned*)(sc + 2048 + 516 + (lane >> 1) * 12 + (lane & 1) * 4) = pa;
          *(LAS u16*)(sc + 2048 + 524 + (lane >> 1) * 12 + (lane & 1) * 2) = (u16)(pb | (pb >> 12));
        }
        const unsigned ec = __builtin_popcount(exm[0]) + __builtin_popcount(exm[1]) + __builtin_popcount(exm[2]) + __builtin_popcount(exm[3]);
        const unsigned ei = wave_scan(ec), nexc = __builtin_amdgcn_readlane(ei, 63), sz = eb + 4u * nexc;
        if (sz <= 960u && nexc <= 64u) {
          unsigned eo = ((sz + 63u) & ~63u) - 4u - 4u * (ei - ec);
          unsigned bm = (exm[0] >> 7) | (exm[1] >> 6) | (exm[2] >> 5) | (exm[3] >> 4);
          while (bm) {
            const unsigned bp = __builtin_ctz(bm); bm &= bm - 1u;
            const unsigned ix = lane * 16 + (bp & 3u) * 4 + (bp >> 3);
            *(LAS unsigned*)(sc + 2048 + eo) = (ix << 8) | *(const LAS unsigned char*)(sc + ix);
            eo -= 4u;
          }
          if (lane == 0) *(LAS unsigned*)(sc + 2048 + 512) = top | (nexc << 8);
          const u32x4 oi = *(const LAS u32x4*)(sc + 2048 + lane * 16);
          st16_sys(pp, lane * 16, sm[0]); st16_sys(pp, lane * 16 + 1024, sm[1]); st16_sys(pp, lane * 16 + 2048, oi);
          hdr = HDR_V + ((sz - 512u + 63u) >> 6);
        }
      }
    }
    if (hdr == 0u) {
      u16x2 mx = {0, 0};
#pragma unroll
      for (int i = 0; i < 16; ++i) mx = __builtin_elementwise_max(mx, __builtin_bit_cast(u16x2, d[i >> 2][i & 3] & 0x7f007f00u));
      const unsigned top = wave_max(max((unsigned)mx[0], (unsigned)mx[1])) >> 8;
      const unsigned base = top > 7u ? top - 7u : 0u;
      const unsigned brep = base * 0x01010101u;
      u32x4 lo[2], nib; unsigned hi[8], tt[8], exc = 0;
#pragma unroll
      for (int i = 0; i < 8; ++i) {
        const unsigned a = d[i >> 1][2 * (i & 1)], c = d[i >> 1][2 * (i & 1) + 1];
        lo[i >> 2][i & 3] = __builtin_amdgcn_perm(c, a, 0x06040200u);
        hi[i] = __builtin_amdgcn_perm(c, a, 0x07050301u);
        tt[i] = (hi[i] | 0x80808080u) - brep;
        exc |= ~tt[i];
      }
      exc &= 0x80808080u;
#pragma unroll
      for (int i = 0; i < 4; ++i)
        nib[i] = (tt[2 * i] & 0x07070707u) | ((hi[2 * i] >> 4) & 0x08080808u) | ((tt[2 * i + 1] & 0x07070707u) << 4) | (hi[2 * i + 1] & 0x80808080u);
      st16_sys(pp, lane * 16, lo[0]); st16_sys(pp, lane * 16 + 1024, lo[1]); st16_sys(pp, lane * 16 + 2048, nib);
      hdr = base;
      if (__builtin_amdgcn_ballot_w64(exc != 0u) != 0ull) {
        unsigned em = 0;
#pragma unroll
        for (int i = 0; i < 8; ++i) {
          const unsigned z = ~tt[i] & 0x80808080u;
          em |= (((z >> 7) & 1u) | ((z >> 14) & 2u) | ((z >> 21) & 4u) | ((z >> 28) & 8u)) << (4 * i);
        }
        const unsigned cnt = __builtin_popcount(em);
        unsigned pre = wave_scan(cnt);
        const unsigned total = __builtin_amdgcn_readlane(pre, 63);
        pre -= cnt;
        if (total > EXC_CAP) hdr = HDR_RAW;
        else {
          hdr = base | HDR_EXC;
          char* ep = ws + OFF_EXC + ((size_t)par * 1024 + u) * 64;
          if (lane == 0) __hip_atomic_store((GAS unsigned*)ep, total, __ATOMIC_RELAXED, __HIP_MEMORY_SCOPE_SYSTEM);
          while (em) {
            const unsigned e = __builtin_ctz(em); em &= em - 1u;
            unsigned h = 0;
#pragma unroll
            for (int i = 0; i < 8; ++i) h = (e >> 2) == (unsigned)i ? hi[i] : h;
            h = (h >> (8 * (e & 3u))) & 0xffu;
            ++pre;
            __hip_atomic_store((GAS unsigned*)(ep + pre * 4), ((unsigned)lane << 16) | (e << 8) | h, __ATOMIC_RELAXED, __HIP_MEMORY_SCOPE_SYSTEM);
          }
        }
      }
    }
    waitv<0>();
    unsigned arrived = 0;
    if (lane == 0) {
      misc[wid] = hdr;
      arrived = __hip_atomic_fetch_add((LAS unsigned*)misc + 12 + (wid >> 2), 1u, __ATOMIC_ACQ_REL, __HIP_MEMORY_SCOPE_WORKGROUP);
    }
    if (rfl(arrived) == 3u) {
      const int w0 = wid & 4;
      const unsigned hv = misc[w0] | (misc[w0 + 1] << 8) | (misc[w0 + 2] << 16) | (misc[w0 + 3] << 24);
      if (lane < NR && lane != rank) {
        char* pf = peer_ws + OFF_PFLAG + (((size_t)par * NR + rank) * 256 + gran) * 8;
        __hip_atomic_store((GAS u64*)pf, ((u64)call << 32) | hv, __ATOMIC_RELAXED, __HIP_MEMORY_SCOPE_SYSTEM);
      }
    }
    st_flag(ws, flag_off(224 + 2 * pair + (lane & 1)) + t * FL_TILE, call);
  } else if (!is_gemm && task < TB * 112 && (task < T0 * 112 || tile_of((6 - task % 7) * 32 + 2 * ((task / 7) & 15)) >= (unsigned)N9)) {
    const int u = task / 7, t = u >> 4, pair = u & 15;
    const char* src = px + (size_t)(32 * pair + (lane & 15)) * (K * 2) + t * 128 + (lane >> 4) * 16;
    const u32x4 d0 = *(const u32x4*)(src), d1 = *(const u32x4*)(src + 16 * K * 2);
    const u32x4 d2 = *(const u32x4*)(src + 64), d3 = *(const u32x4*)(src + 16 * K * 2 + 64);
    call = rfl(ld_agent(ws + OFF_CNT + b * 64)) + 1u;
    const int sl = 6 - task % 7;
    const char* dws = P.ws[(rank + 1 + task % 7) & 7];
    const char* xw = dws + OFF_XW + (call & 1u) * XW_BYTES + ((size_t)(2 * t) * 256 + sl * 32 + 2 * pair) * 1024;
    st16_sys(xw, lane * 16, d0); st16_sys(xw, lane * 16 + 1024, d1);
    st16_sys(xw, lane * 16 + 262144, d2); st16_sys(xw, lane * 16 + 262144 + 1024, d3);
    unsigned fo = flag_off(sl * 32 + 2 * pair + (lane & 1)) + t * FL_TILE;
    asm volatile("" : "+v"(fo));
    BAR();
    waitv<0>();
    st_flag(dws, fo, call);
    pushed = true;
  } else {
    const unsigned cnt0 = ld_agent(ws + OFF_CNT + b * 64);
    if (is_gemm) __builtin_amdgcn_s_barrier();
    call = rfl(cnt0) + 1u;
    if (is_gemm && wid == 7) claim(call);
    asm volatile("" :: "s"(call) : "memory");
  }
  const unsigned par = call & 1u;
  char* const tmo = ws + OFF_TMO;

  if (!is_gemm) {
    if (!pushed) BAR();
    if (wid == 7 && lane == 0) st_agent(ws + OFF_CNT + b * 64, call);
    if (wid == 0) return;
    static_assert(PPP == NC, "seven puller waves per CU");
    const int pw = 7 * (b - NG) + (7 - wid);
    const int sl = pw / PPP, src = (rank + 1 + sl) & 7;
    const int wi = __builtin_popcountll(__builtin_amdgcn_ballot_w64(lane < PPP && ((sl * PPP + lane) % 7) * 64 + lane < (7 - wid) * 64 + pw % PPP));
    const char* pws = P.ws[src];
    const char* pfl = ws + OFF_PFLAG + ((size_t)par * NR + src) * 2048;
    const unsigned l16 = lane * 16;
    const unsigned xw0 = (unsigned)(OFF_XW + par * XW_BYTES) + sl * 32768u, p0 = (unsigned)(OFF_P + par * P_BYTES);
    const unsigned dummy = (unsigned)OFF_TMO + 128u + (unsigned)pw * 4u;
    const unsigned ftab = flag_off(sl * 32 + (lane & 31));
    auto flag = [&](unsigned u) {
      u64 sv;
      asm volatile("s_mov_b64 %0, exec\n\ts_lshl_b64 exec, 3, %4\n\tglobal_store_dword %1, %2, %3 sc0 sc1\n\ts_mov_b64 exec, %0"
                   : "=&s"(sv) : "v"(ftab + (u >> 4) * FL_TILE), "v"(call), "s"(ws), "s"((u & 15u) * 2u) : "scc", "memory");
    };
    auto xoff = [&](unsigned u) -> unsigned { return xw0 + ((u >> 4) << 19) + ((u & 15u) << 11); };
    auto pick = [&](u64 g, unsigned u) -> unsigned { return (unsigned)(g >> 32) == call ? ((unsigned)g >> (8u * (u & 3u))) & 0xffu : HDR_NONE; };
    auto wait_hdr = [&](unsigned u) -> unsigned {
      const u64 t0 = __builtin_amdgcn_s_memrealtime();
      for (unsigned it = 1;; ++it) {
        u64 gs;
        asm volatile("s_load_dwordx2 %0, %1, 0x0 glc\n\ts_waitcnt lgkmcnt(0)" : "=s"(gs) : "s"(pfl + (u >> 2) * 8) : "memory");
        unsigned h = pick(gs, u);
        if (h != HDR_NONE) return h;
        if ((it & 15u) == 0u) {
          const u64 gv = __hip_atomic_load((const GAS u64*)(pfl + (u >> 2) * 8), __ATOMIC_RELAXED, __HIP_MEMORY_SCOPE_SYSTEM);
          h = rfl(pick(gv, u));
          if (h != HDR_NONE) return h;
          if (rfl(ld_agent(tmo)) == call || __builtin_amdgcn_s_memrealtime() - t0 > DEADLINE) { st_agent(tmo, call); return HDR_NONE; }
        }
        __builtin_amdgcn_s_sleep(4);
      }
    };
    const unsigned slot = lds0 + wid * 8192;
    auto issue = [&](unsigned u, unsigned hdr) {
      unsigned va = l16 + (p0 + u * 3072u), vb = min(l16, 496u + 64u * (hdr - HDR_V)) + (p0 + 2048u + u * 3072u);
      if (__builtin_expect(hdr <= HDR_V, 0)) {
        const bool raw = hdr == HDR_RAW;
        if (raw) va = l16 + xoff(u) + (7 - sl) * 32768u;
        vb = va + (raw ? 262144u : 2048u);
        if (raw || (hdr & HDR_EXC)) {
          const unsigned vc = raw ? vb + 1024u : (l16 & 48u) + ((unsigned)OFF_EXC + (par * 1024u + u) * 64u);
          asm volatile("s_mov_b32 m0, %2\n\ts_nop 0\n\tglobal_load_lds_dwordx4 %0, %1 sc0 sc1" :: "v"(vc), "s"(pws), "s"(slot + 3072u) : "memory");
        }
      }
      asm volatile("s_mov_b32 m0, %3\n\ts_nop 0\n\t"
                   "global_load_lds_dwordx4 %0, %2 sc0 sc1\n\t"
                   "global_load_lds_dwordx4 %0, %2 offset:1024 sc0 sc1\n\t"
                   "s_add_u32 m0, m0, 2048\n\ts_nop 0\n\t"
                   "global_load_lds_dwordx4 %1, %2 sc0 sc1"
                   :: "v"(va), "v"(vb), "s"(pws), "s"(slot) : "scc", "memory");
    };
    auto finish = [&](unsigned u, unsigned hdr, const char* gnext) {
      auto fetch = [&]() { asm volatile("s_load_dwordx2 s[96:97], %0, 0x0 glc" :: "s"(gnext) : "s96", "s97", "memory"); };
      const LAS char* slp = (const LAS char*)(size_t)slot;
      u32x4 o[4];
      if (__builtin_expect(hdr > HDR_V, 1)) {
        LAS char* bs = (LAS char*)(size_t)(slot + 4096u);
        const unsigned hl = *(const LAS unsigned*)(slp + 2560);
        const unsigned w0 = *(const LAS unsigned*)(slp + 2564 + (lane >> 1) * 12 + (lane & 1) * 4);
        const unsigned h16 = *(const LAS u16*)(slp + 2572 + (lane >> 1) * 12 + (lane & 1) * 2);
        const unsigned en = *(const LAS unsigned*)(slp + 2556 + 64u * (hdr - HDR_V) - lane * 4);
        const u32x2 A = *(const LAS u32x2*)(slp + 2048 + lane * 8);
        const u32x4 sm0 = *(const LAS u32x4*)(slp + l16), sm1 = *(const LAS u32x4*)(slp + 1024 + l16);
        const unsigned E[2] = {~(A[0] | (A[0] >> 1)) & 0x55555555u, ~(A[1] | (A[1] >> 1)) & 0x55555555u};
        const unsigned cnt = __builtin_popcount(E[0]) + __builtin_popcount(E[1]);
        unsigned off = ((wave_scan(cnt) - cnt) & 1023u) + (slot + 4096u);
        const unsigned h = rfl(hl);
        const unsigned top = h & 0xffu, trep = top * 0x01010101u;
        const unsigned t0 = rot4(trep - 0x07060501u), t1 = rot4(trep - 0x0b0a0908u);
        const unsigned pb = h16 | (h16 << 12);
        u32x4 ex;
#pragma unroll
        for (int r = 0; r < 4; ++r)
          ex[r] = __builtin_amdgcn_perm(t1, t0, ((w0 >> (2 * r)) & 0x03030303u) | ((r < 2 ? pb << (2 - r) : pb >> (r - 2)) & 0x04040404u));
        *(LAS u32x4*)(bs + lane * 16) = ex;
        *(LAS unsigned char*)(bs + ((unsigned)lane < ((h >> 8) & 0xffu) ? (en >> 8) & 1023u : 1088u + lane)) = (unsigned char)rot4((top - en) & 0xffu);
        unsigned wl[8], wh[8], wo[8];
#pragma unroll
        for (int g = 0; g < 8; ++g) {
          const LAS unsigned* wp = (const LAS unsigned*)(size_t)(off & ~3u);
          wl[g] = wp[0]; wh[g] = wp[1]; wo[g] = off;
          off += __builtin_popcount(E[g >> 2] & (0x01010101u << (2 * (g & 3))));
        }
        __builtin_amdgcn_sched_barrier(0);
        unsigned Sq[4];
#pragma unroll
        for (int dd = 0; dd < 2; ++dd) {
          const unsigned Nz = E[dd] ^ 0x55555555u;
          const unsigned Pf = ((E[dd] + (E[dd] << 8)) + (E[dd] << 16)) << 8;
          const unsigned S = A[dd] | (Pf & (E[dd] | (E[dd] << 1)));
          Sq[2 * dd] = (S & 0x33333333u) | ((Nz & 0x11111111u) << 2);
          Sq[2 * dd + 1] = ((S >> 2) & 0x33333333u) | (Nz & 0x44444444u);
        }
        asm volatile("" : "+v"(wl[0]), "+v"(wl[1]), "+v"(wl[2]), "+v"(wl[3]), "+v"(wl[4]), "+v"(wl[5]), "+v"(wl[6]), "+v"(wl[7]),
                          "+v"(Sq[0]), "+v"(Sq[1]), "+v"(Sq[2]), "+v"(Sq[3]) :: "memory");
        fetch();
        __builtin_amdgcn_sched_barrier(0);
        const unsigned tab = rot4(trep - 0x04030200u);
        unsigned sel[8], r4[8], hi[8], lo[8];
#pragma unroll
        for (int g = 0; g < 8; ++g) { r4[g] = __builtin_amdgcn_alignbyte(wh[g], wl[g], wo[g]); sel[g] = Sq[2 * (g >> 2) + (g & 1)] >> (4 * ((g >> 1) & 1)); }
        __builtin_amdgcn_sched_barrier(0);
#pragma unroll
        for (int g = 0; g < 8; ++g) sel[g] &= 0x07070707u;
        __builtin_amdgcn_sched_barrier(0);
#pragma unroll
        for (int g = 0; g < 8; ++g) r4[g] = __builtin_amdgcn_perm(tab, r4[g], sel[g]);
        __builtin_amdgcn_sched_barrier(0);
#pragma unroll
        for (int g = 0; g < 8; ++g) { const unsigned s4 = g < 4 ? sm0[g & 3] : sm1[g & 3]; hi[g] = bfi(0x80808080u, s4, r4[g]); lo[g] = bfi(0x80808080u, r4[g], s4); }
        __builtin_amdgcn_sched_barrier(0);
#pragma unroll
        for (int g = 0; g < 8; ++g) {
          o[g >> 1][2 * (g & 1)] = __builtin_amdgcn_perm(hi[g], lo[g], 0x05010400u);
          o[g >> 1][2 * (g & 1) + 1] = __builtin_amdgcn_perm(hi[g], lo[g], 0x07030602u);
        }
      } else {
        u32x4 v[4];
        const LAS u32x4* s4 = (const LAS u32x4*)(slp + l16);
        v[0] = s4[0]; v[1] = s4[64]; v[2] = s4[128]; v[3] = s4[192];
        fetch();
        if (hdr == HDR_RAW) { o[0] = v[0]; o[1] = v[1]; o[2] = v[2]; o[3] = v[3]; }
        else {
          const unsigned brep = (hdr & 127u) * 0x01010101u;
#pragma unroll
          for (int i = 0; i < 4; ++i) {
            const unsigned nb = v[2][i];
            const unsigned h0 = ((nb & 0x07070707u) + brep) | ((nb << 4) & 0x80808080u);
            const unsigned h1 = (((nb >> 4) & 0x07070707u) + brep) | (nb & 0x80808080u);
            const unsigned l0 = v[i >> 1][2 * (i & 1)], l1 = v[i >> 1][2 * (i & 1) + 1];
            o[i][0] = __builtin_amdgcn_perm(h0, l0, 0x05010400u); o[i][1] = __builtin_amdgcn_perm(h0, l0, 0x07030602u);
            o[i][2] = __builtin_amdgcn_perm(h1, l1, 0x05010400u); o[i][3] = __builtin_amdgcn_perm(h1, l1, 0x07030602u);
          }
          if (hdr & HDR_EXC) {
            const unsigned cnt = min((unsigned)__builtin_amdgcn_readlane(v[3][0], 0), EXC_CAP);
            for (unsigned q = 1; q <= cnt; ++q) {
              const unsigned w = rfl(*(const LAS unsigned*)(slp + 3072 + q * 4));
              const unsigned e = (w >> 8) & 31u, sh = 8u + 16u * (e & 1u), dw = e >> 1;
              const bool own = (unsigned)lane == (w >> 16);
#pragma unroll
              for (int i = 0; i < 16; ++i) {
                const unsigned cur = o[i >> 2][i & 3];
                o[i >> 2][i & 3] = (own && dw == (unsigned)i) ? ((cur & ~(0xffu << sh)) | ((w & 0xffu) << sh)) : cur;
              }
            }
          }
        }
      }
      const unsigned va = l16 + xoff(u), vb = va + 262144u;
      asm volatile("global_store_dwordx4 %0, %2, %6 sc0 sc1\n\t"
                   "global_store_dwordx4 %0, %3, %6 offset:1024 sc0 sc1\n\t"
                   "global_store_dwordx4 %1, %4, %6 sc0 sc1\n\t"
                   "global_store_dwordx4 %1, %5, %6 offset:1024 sc0 sc1\n\ts_nop 1"
                   :: "v"(va), "v"(vb), "v"(o[0]), "v"(o[1]), "v"(o[2]), "v"(o[3]), "s"(ws) : "memory");
    };
    const unsigned tk = (unsigned)OFF_TK + src * 128u;
    auto ticket = [&]() {
      u64 sv;
      asm volatile("s_mov_b64 %0, exec\n\ts_mov_b64 exec, 1\n\tglobal_atomic_add v255, %1, %2, %3 sc0\n\ts_mov_b64 exec, %0"
                   : "=&s"(sv) : "v"(tk), "v"(1u), "s"(ws) : "v255", "memory");
    };
    constexpr unsigned NMID = (NT - T0 - LEAD) * 16, NA = T0 + LEAD - TB;
    const unsigned bmask = (unsigned)__builtin_amdgcn_ballot_w64(lane < 16 && tile_of(sl * 32 + 2 * lane) >= (unsigned)N9);
    const unsigned nbig = __builtin_popcount(bmask);
    const unsigned tk0 = (call - 1u) * ((NT - T0) * 16u - (TB - T0) * nbig);
    auto nth = [&](unsigned m, unsigned k) -> unsigned { for (; k; --k) m &= m - 1u; return __builtin_ctz(m); };
    auto order = [&](unsigned i) -> unsigned {
      const unsigned j = i - NA * nbig;
      if (__builtin_expect(j < NMID, 1)) return T0 * 16 + j + ((bmask >> (j & 15u)) & 1u) * (LEAD * 16u);
      if (i < NA * nbig) return (TB + i / nbig) * 16u + nth(bmask, i % nbig);
      const unsigned k = j - NMID, ns = 16u - nbig;
      return k < LEAD * ns ? (NT - LEAD + k / ns) * 16u + nth(bmask ^ 0xffffu, k % ns) : 1024u;
    };
    unsigned uc = order(wi);
    unsigned hc = wait_hdr(uc);
    if (hc == HDR_NONE) return;
    for (int i = 8; i < wi; i += 4) __builtin_amdgcn_s_sleep(4);
    ticket();
    issue(uc, hc);
    st_flag(ws, dummy, call);
    for (;;) {
      unsigned tv;
      asm volatile("s_waitcnt vmcnt(1)\n\tv_readfirstlane_b32 %0, v255" : "=s"(tv) :: "memory");
      const unsigned un = order(min(tv - tk0, 1024u) + PPP);
      finish(uc, hc, pfl + (min(un, 1023u) >> 2) * 8);
      u64 gs;
      asm volatile("s_waitcnt lgkmcnt(0)\n\ts_mov_b64 %0, s[96:97]" : "=s"(gs) :: "memory");
      if (un >= 1024u) { waitv<0>(); flag(uc); return; }
      hc = pick(gs, un);
      if (__builtin_expect(hc == HDR_NONE, 0)) {
        waitv<0>(); flag(uc);
        hc = wait_hdr(un);
        if (hc == HDR_NONE) return;
      }
      ticket();
      issue(un, hc);
      waitv<4>();
      flag(uc);
      uc = un;
    }
  }

  unsigned slot1;
  while ((slot1 = rfl(misc[9])) == 0u) __builtin_amdgcn_s_sleep(1);
  const int xcd = (int)slot1 - 1;
  const int xh = (xcd >> 1) & 1, ix = b >> 3;
  const int lin = (xcd & 1) ? (ix < 24 ? ((ix >> 2) * 2 + xh) * 4 + (ix & 3) : 48 + xh * 4 + (ix - 24))
                            : (ix < 20 ? 56 + ((ix >> 2) * 2 + xh) * 4 + (ix & 3) : ix < 26 ? 96 + xh * 6 + (ix - 20) : 51 + xh * 4);
  const int ro = lin >> 2, ct = (xcd & 4) + (lin & 3);
  const bool big = ro >= N9;
  const int rb0 = (int)tile_start(ro);

  if (wid == 4 || wid == 5) {
    const int h = wid - 4;
    const __amdgpu_buffer_rsrc_t rs = __builtin_amdgcn_make_buffer_rsrc((void*)P.w, 0, K * N * 2, 0x00020000);
    unsigned voff[4];
#pragma unroll
    for (int pt = 0; pt < 4; ++pt) {
      const unsigned fk = (lane >> 5) | ((pt & 1) << 1) | ((pt >> 1) << 2);
      voff[pt] = (lane >> 5) * 4096 + (((lane & 31) ^ (fk << 1)) * 16) + ct * 512;
    }
    auto issue = [&](int j) {
      const unsigned lds = lds0 + ((j + 3) % 3) * STAGE + XB + h * 16384;
      const unsigned soff = (unsigned)(64 * j + 64 + 32 * h) * 4096u;
#pragma unroll
      for (int i = 0; i < 16; ++i)
        asm volatile("s_mov_b32 m0, %0\n\ts_nop 0\n\tbuffer_load_dwordx4 %1, %2, %3 offen lds"
                     :: "s"(lds + i * 1024), "v"(voff[(i & 1) | (((i >> 2) & 1) << 1)]), "s"(rs), "s"(soff + i * 8192) : "memory");
    };
    const bool early = big && b >= 64;
    if (early) { issue(-1); issue(0); }
    while (rfl(misc[12]) < 4u || (packs && rfl(misc[13]) < 4u)) __builtin_amdgcn_s_sleep(2);
    if (!early) { issue(-1); issue(0); }
    issue(1);
    waitv<32>();
    BAR();
    waitv<16>();
    BAR();
    for (int j = 0; j < NT - 2; ++j) {
      if (j + 3 < NT) { issue(j + 2); waitv<16>(); } else waitv<0>();
      BAR();
    }
    return;
  }
  if (wid == 7) {
    const __amdgpu_buffer_rsrc_t rs = __builtin_amdgcn_make_buffer_rsrc((void*)P.w, 0, K * N * 2, 0x00020000);
    auto touch = [&](int i) {
      asm volatile("s_mov_b32 m0, %0\n\ts_nop 0\n\tbuffer_load_dword %1, %2, %3 offen lds"
                   :: "s"(lds0 + LDS_MISC + 64), "v"(lane * 128), "s"(rs), "s"((unsigned)(i * NG + b) * 8192u) : "memory");
    };
    while (rfl(misc[12]) < 4u) __builtin_amdgcn_s_sleep(2);
#pragma unroll
    for (int i = 0; i < WPF0; ++i) touch(i);
    if (big) return;
    BAR();
    constexpr int NS = N9 * 8;
    static_assert(((K * N * 2 / 8192 - WPF0 * NG + 2 * NS - 1) / (2 * NS)) * WPFGAP <= NT - 1, "touch rounds");
    const unsigned s0 = (unsigned)(WPF0 * NG + (xcd >> 2) * (N9 * 4) + lin);
    for (int i = 0; WPF0 * NG + i * NS < K * N * 2 / 8192; i += 2) {
#pragma unroll
      for (int e = 0; e < WPFGAP; ++e) BAR();
#pragma unroll
      for (int e = 0; e < 2; ++e)
        asm volatile("s_mov_b32 m0, %0\n\ts_nop 0\n\tbuffer_load_dword %1, %2, %3 offen lds"
                     :: "s"(lds0 + LDS_MISC + 64), "v"(lane * 128), "s"(rs), "s"(min(s0 + (i + e) * NS, (unsigned)(K * N * 2 / 8192 - 1)) * 8192u) : "memory");
    }
    return;
  }
  if (wid == 6) {
    const __amdgpu_buffer_rsrc_t rs = __builtin_amdgcn_make_buffer_rsrc((void*)(ws + OFF_XW + par * XW_BYTES), 0, (unsigned)XW_BYTES, 0x00020000);
    const unsigned voff = lane * 16;
    const unsigned voff2 = big ? voff : 0x80000000u;
    const char* fls = ws + OFF_XFLAG + ro * 128;
    bool dead = false;
    auto vpoll = [&](int j) -> bool { return __builtin_amdgcn_ballot_w64((int)(ld_sys(fls + min(lane & 7, 4) * 4 + j * FL_TILE) - call) < 0) == 0ull; };
    auto allset = [&](const u32x8& v) -> bool { return (int)((v[0] - call) | (v[1] - call) | (v[2] - call) | (v[3] - call) | (v[4] - call)) >= 0; };
    auto spoll = [&](int j) -> bool {
      u32x8 v;
      asm volatile("s_load_dwordx8 %0, %1, 0x0 glc\n\ts_waitcnt lgkmcnt(0)" : "=s"(v) : "s"(fls + j * FL_TILE) : "memory");
      return allset(v);
    };
    auto issue = [&](int j) -> bool {
      unsigned lds = lds0 + (j % 3) * STAGE;
      unsigned soff = (unsigned)(2 * j * 256 + rb0) * 1024u;
      u32x8 v;
      asm volatile(XDMA_ASM "s_waitcnt lgkmcnt(0)"
                   : "=&s"(v), "+s"(lds), "+s"(soff) : "v"(voff), "s"(rs), "s"(fls + min(j + 1, NT - 1) * FL_TILE), "v"(voff2) : "scc", "memory");
      return allset(v);
    };
    int issued = 0;
    bool nxt = false;
    for (int j = -1; j < NT - 1; ++j) {
      if (issued <= j + 1) {
        bool late = false;
        if (!nxt && !dead) {
          late = true;
          u64 t0 = 0;
          for (unsigned it = 1;; ++it) {
            if (spoll(j + 1)) break;
            if ((it & 15u) == 0u) {
              if (it == 16u) t0 = __builtin_amdgcn_s_memrealtime();
              if (vpoll(j + 1)) break;
              if (rfl(ld_agent(tmo)) == call || __builtin_amdgcn_s_memrealtime() - t0 > DEADLINE) {
                st_agent(tmo, call); misc[8] = 0x7fc07fc0u; dead = true; break;
              }
            }
            __builtin_amdgcn_s_sleep(1);
          }
        }
        nxt = issue(j + 1) || dead; issued = j + 2;
        if (j + 2 < NT) {
          if (!nxt && !late) nxt = spoll(j + 2);
          if (nxt) { nxt = issue(j + 2) || dead; issued = j + 3; }
          else nxt = spoll(j + 2);
        }
      } else if (j + 2 < NT) {
        if (!nxt) nxt = spoll(j + 2);
        if (nxt) { nxt = issue(j + 2) || dead; issued = j + 3; }
      }
      if (j < 0) { BAR(); st_agent(ws + OFF_CNT + b * 64, call); }
      if (j == NT - 3) continue;
      if (issued == j + 3) asm volatile("s_waitcnt vmcnt(%0) lgkmcnt(0)" :: "i"(2 * MB) : "memory");
      else asm volatile("s_waitcnt vmcnt(0) lgkmcnt(0)" ::: "memory");
      BAR();
    }
    return;
  }

  {
    const int g = lane >> 4, q = (lane & 15) >> 2, p = lane & 3;
    const unsigned f = q | ((g & 1) << 2);
    unsigned xb = lds0 + lane * 16;
    unsigned wb = lds0 + 2 * STAGE + XB + (8 * g + q) * 512 + ((128 * wid + 8 * p) ^ (f << 5));
    static_assert((NT - 1) % 3 == 0 && XB + 4 * MB * 2048 <= 2 * STAGE && STAGE % 512 == 0, "epilogue staging");
    const unsigned ep = lds0 + XB + wid * (MB * 2048);
    const int jr = lane & 15;
    const unsigned ew = ep + jr * 128 + 8 * (g & 1) + (((g >> 1) ^ (jr & 7)) * 16);
    const unsigned er = ep + (lane >> 3) * 128 + (((lane & 7) ^ (lane >> 3)) * 16);
    const unsigned vo = (lane >> 3) * (N * 2) + (lane & 7) * 16;
    const __amdgpu_buffer_rsrc_t outr = __builtin_amdgcn_make_buffer_rsrc((void*)(P.out + ct * 256 + wid * 64), 0, M * N * 2 - (ct * 256 + wid * 64) * 2, 0x00020000);
    unsigned so = (unsigned)((rb0 + 32 * (rank + 1)) & 255) * (16 * N * 2);
    const unsigned badp = lds0 + LDS_MISC + 32;
    unsigned st, cnt, dl, dp, dn;
    asm volatile("s_cmp_lt_u32 %[ro], %[n9]\n\ts_cbranch_scc0 L_big_%=\n\t" CONS_ASM_9 "s_branch L_done_%=\n\tL_big_%=:\n\t" CONS_ASM_10 "L_done_%=:"
                 : [st] "=&s"(st), [cnt] "=&s"(cnt), [dl] "=&s"(dl), [dp] "=&s"(dp), [dn] "=&s"(dn), [so] "+s"(so), "+{v11}"(xb), "+{v7}"(wb)
                 : [ew] "{v1}"(ew), [er] "{v2}"(er), [vo] "{v3}"(vo), [bad] "s"(badp), [out] "s"(outr), [ro] "s"(ro), [n9] "i"(N9)
                 : CONS_CLOBBERS, "scc", "memory");
    __builtin_amdgcn_endpgm();
  }
}

extern "C" void kernel_launch(int rank, int nranks, ncclComm_t comm, hipStream_t stream,
                              void* const* d_in, const int* in_sizes, int n_in,
                              void* d_out, int out_size, const int* in_dtypes, int out_dtype,
                              void* const* d_out_all, void* const* d_ws_all, size_t ws_bytes) {
  (void)comm; (void)in_dtypes; (void)out_dtype; (void)d_out_all;
  if (nranks != NR || (unsigned)rank >= (unsigned)NR || ws_bytes < WS_NEED || n_in != 2 || in_sizes[0] != M * K / NR || in_sizes[1] != K * N ||
      out_size != M * N) { fprintf(stderr, "ag_gemm: unexpected configuration\n"); return; }
  static bool attr_set[NR] = {};
  if (!attr_set[rank]) {
    hipError_t e = hipFuncSetAttribute((const void*)ag_gemm_kernel, hipFuncAttributeMaxDynamicSharedMemorySize, LDS_BYTES);
    if (e != hipSuccess) { fprintf(stderr, "hipFuncSetAttribute: %s\n", hipGetErrorString(e)); return; }
    attr_set[rank] = true;
  }
  Params p;
  p.x = (const char*)d_in[0]; p.w = (const char*)d_in[1]; p.out = (u16*)d_out;
  const size_t base = (ws_bytes - WS_NEED) & ~(size_t)((2u << 20) - 1);
  for (int i = 0; i < NR; ++i) p.ws[i] = (char*)d_ws_all[i] + base;
  p.self = p.ws[rank]; p.rank = rank; p.pad = 0;
  ag_gemm_kernel<<<256, 512, LDS_BYTES, stream>>>(p);
}
```
